```python
import numpy as np

kernel_name = "gpu_gemm_ar_m4096_k4096_n4096_bf16_silu"


ABSMAX_RTOL = 0.008

COMPUTE_DTYPE = "bfloat16"
EPILOGUE = "silu"
IN_QUANT = None
OUT_QUANT = None
IN_DTYPES = {"x": "bfloat16", "w_mat": "bfloat16"}
WORLD_SIZE = 8

def _to_bf16(x):
    u = np.ascontiguousarray(x, dtype=np.float32).view(np.uint32)
    return ((u + 0x7FFF + ((u >> 16) & 1)) & 0xFFFF0000).view(np.float32)

def setup_inputs():

    x = _to_bf16(np.random.randn(4096, 4096).astype(np.float32))
    w_mat = _to_bf16((np.random.randn(4096, 4096) / np.sqrt(4096)).astype(np.float32))
    return {"x": x, "w_mat": w_mat}

def _epilogue(y):
    return (y / (1.0 + np.exp(-np.clip(y, -60.0, 60.0)))).astype(np.float32)

def reference(x, w_mat):
    y = (x @ w_mat).astype(np.float32)
    return _epilogue(y)
```

```cpp
#include <hip/hip_runtime.h>
#include <rccl/rccl.h>
#include <cstdio>
#include <cstdint>

typedef unsigned int u32;
typedef unsigned long long u64;
typedef __attribute__((ext_vector_type(4))) float f32x4;
typedef __attribute__((ext_vector_type(4))) unsigned int u32x4;
typedef __attribute__((ext_vector_type(2))) unsigned int u32x2;
typedef __attribute__((ext_vector_type(4))) __bf16 bf16x4;
typedef __attribute__((ext_vector_type(8))) __bf16 bf16x8;
typedef __attribute__((ext_vector_type(2))) __bf16 bf16x2;
#define LAS __attribute__((address_space(3)))
#define GAS __attribute__((address_space(1)))

constexpr size_t OFF_CTR = 0;
constexpr size_t OFF_TMO = 4096;
constexpr size_t OFF_INFLAG = 8192;
constexpr size_t OFF_PFLAG = 32768;
constexpr size_t OFF_FFLAG = 49152;
constexpr size_t OFF_DONE = 53248;
constexpr size_t OFF_DBG = 65536;
constexpr size_t OFF_XS = 1u << 20;
constexpr size_t OFF_WS = 9u << 20;
constexpr size_t OFF_PS = 17u << 20;
constexpr size_t WS_NEED = 21u << 20;

constexpr u64 DEADLINE = 100000000ull;
constexpr int NSTAGE = 4;
constexpr int STAGE_BYTES = 32768;
constexpr int LDS_CTL = NSTAGE * STAGE_BYTES;
constexpr int LDS_BYTES = LDS_CTL + 256;

struct Args {
  const char* x;
  const char* w;
  char* out[8];
  char* ws[8];
  u32 rank;
  u32 pad;
  u32 psrc[8][32];
  u32 pdst[32];
  u32 tile[8][32];
  u32 foff[128];
};

#define STAMP(i) do {} while (0)

__host__ __device__ __forceinline__ u32 grow0(u32 r, u32 lr) {
  const u32 g = r >> 2, qi = (r >> 1) & 1, kind = lr >> 3, rb = (lr >> 2) & 1, i4 = lr & 3;
  const u32 RB = g ? qi + 2 * rb : 2 * qi + rb;
  const u32 half = kind ? g : 1 - g;
  return RB * 1024 + half * 512 + i4 * 128;
}
__host__ __device__ __forceinline__ u32 gcol0(u32 r, u32 lc) {
  const u32 g = r >> 2, qj = r & 1, cb = lc >> 3, j8 = lc & 7;
  const u32 CB = g ? qj + 2 * cb : 2 * qj + cb;
  return CB * 1024 + j8 * 128;
}

__device__ __forceinline__ void ld16(u32x4& v, u32 voff, const char* sbase) {
  asm volatile("global_load_dwordx4 %0, %1, %2" : "=v"(v) : "v"(voff), "s"(sbase) : "memory");
}
__device__ __forceinline__ void ld16_sc1(u32x4& v, u32 voff, const char* sbase) {
  asm volatile("global_load_dwordx4 %0, %1, %2 sc1" : "=v"(v) : "v"(voff), "s"(sbase) : "memory");
}
__device__ __forceinline__ void st16_sys(u32 voff, u32x4 v, char* sbase) {
  asm volatile("global_store_dwordx4 %0, %1, %2 sc0 sc1\n\ts_nop 1" :: "v"(voff), "v"(v), "s"(sbase) : "memory");
}
__device__ __forceinline__ void st16_agent(u32 voff, u32x4 v, char* sbase) {
  asm volatile("global_store_dwordx4 %0, %1, %2 sc1\n\ts_nop 1" :: "v"(voff), "v"(v), "s"(sbase) : "memory");
}
template <int N> __device__ __forceinline__ void waitv(u32x4& v) {
  asm volatile("s_waitcnt vmcnt(%1)" : "+v"(v) : "n"(N) : "memory");
}
__device__ __forceinline__ bool expired(u64 t0) { return __builtin_amdgcn_s_memrealtime() - t0 > DEADLINE; }

constexpr u32 RING = 24;
constexpr u32 C_LANDED = 32, C_CONS = 33;

template <bool SC1>
__device__ __forceinline__ void glds4v(u32 v0, u32 v1, u32 v2, u32 v3, const char* sbase, u32 lds) {
  u32 keep;
#define GLDS4V(S) asm volatile( \
      "s_mov_b32 %0, m0\n\t" \
      "s_mov_b32 m0, %6\n\ts_nop 0\n\tglobal_load_lds_dwordx4 %1, %5" S "\n\t" \
      "s_add_u32 m0, m0, 0x400\n\ts_nop 0\n\tglobal_load_lds_dwordx4 %2, %5" S "\n\t" \
      "s_add_u32 m0, m0, 0x400\n\ts_nop 0\n\tglobal_load_lds_dwordx4 %3, %5" S "\n\t" \
      "s_add_u32 m0, m0, 0x400\n\ts_nop 0\n\tglobal_load_lds_dwordx4 %4, %5" S "\n\t" \
      "s_mov_b32 m0, %0" \
      : "=&s"(keep) : "v"(v0), "v"(v1), "v"(v2), "v"(v3), "s"(sbase), "s"(lds) : "memory", "scc")
  if constexpr (SC1) GLDS4V(" sc1"); else GLDS4V("");
}
__device__ __forceinline__ u32 lds_get(LAS u32* p) {
  asm volatile("" ::: "memory");
  const u32 v = __builtin_amdgcn_readfirstlane(__hip_atomic_load(p, __ATOMIC_RELAXED, __HIP_MEMORY_SCOPE_WORKGROUP));
  asm volatile("" ::: "memory");
  return v;
}
__device__ __forceinline__ void lds_set(LAS u32* p, u32 v) {
  asm volatile("" ::: "memory");
  __hip_atomic_store(p, v, __ATOMIC_RELAXED, __HIP_MEMORY_SCOPE_WORKGROUP);
  asm volatile("" ::: "memory");
}
__device__ __forceinline__ void ring_space(LAS u32* ctl, u32 gi, u32 s3) {
  if (gi >= RING && lds_get(ctl + C_CONS + s3) <= gi - RING) {
    asm volatile("s_waitcnt vmcnt(0)" ::: "memory");
    lds_set(ctl + C_LANDED, gi);
    while (lds_get(ctl + C_CONS + s3) <= gi - RING) __builtin_amdgcn_s_sleep(1);
  }
}
__device__ __forceinline__ void ring_publish(LAS u32* ctl, u32 gi) {
  asm volatile("s_waitcnt vmcnt(12)" ::: "memory");
  if (gi >= 3) __hip_atomic_fetch_max(ctl + C_LANDED, gi - 2, __ATOMIC_RELAXED, __HIP_MEMORY_SCOPE_WORKGROUP);
  asm volatile("" ::: "memory");
}
template <int W> __device__ __forceinline__ void wst16_sys(u32 voff, u32x4 v, char* sbase) {
  asm volatile("s_waitcnt vmcnt(%3)\n\tglobal_store_dwordx4 %0, %1, %2 sc0 sc1\n\ts_nop 1" :: "v"(voff), "v"(v), "s"(sbase), "n"(W) : "memory");
}

template <int NKB>
__device__ __forceinline__ void push_inputs(const Args& a, u32 p, u32 xl, u32 j, u32 wv, u32 lane, u32 seq,
                                            LAS char* shm, u32 lds0) {
  LAS u32* ctl = (LAS u32*)(shm + LDS_CTL);
  constexpr u32 NG = 32 * NKB;
  if (wv == 0) {
    u32 vx[4], vw[4];
#pragma unroll
    for (u32 s = 0; s < 4; ++s) {
      vx[s] = (8 * s + (lane >> 3)) * 1024 + (((lane & 7) ^ (4 * (s & 1) + (lane >> 4))) * 16);
      vw[s] = (4 * s + (lane >> 4)) * 8192 + (((lane & 15) ^ (((lane >> 4) | ((s >> 1) << 2)) << 1)) * 16);
    }
    u32 gi = 0, slot = 0, s3 = 0;
    for (u32 i = 0; i < 32; ++i) {
      const u32 ps = a.psrc[xl][i];
      const bool isw = ps >> 31;
      const char* sb = isw ? a.w + (ps & 0x7fffffffu) + j * 16 * 8192 : a.x + ps + j * 32 * 1024;
      for (u32 kb = 0; kb < NKB; ++kb) {
        ring_space(ctl, gi, s3);
        if (isw) { glds4v<false>(vw[0], vw[1], vw[2], vw[3], sb, lds0 + slot * 4096); sb += 64 * 8192; }
        else { glds4v<false>(vx[0], vx[1], vx[2], vx[3], sb, lds0 + slot * 4096); sb += 128; }
        ring_publish(ctl, gi);
        ++gi; slot = slot + 1 == RING ? 0 : slot + 1; s3 = s3 == 2 ? 0 : s3 + 1;
      }
    }
    asm volatile("s_waitcnt vmcnt(0)" ::: "memory");
    lds_set(ctl + C_LANDED, NG);
  } else {
    const u32 kboff = NKB == 5 ? 17 + 5 * (xl - 1) : 5 + 3 * (xl - 4);
    char* const wsp = a.ws[p];
    char* const dbase = wsp + kboff * 16384 + j * 4096;
    GAS u32* const flag = (GAS u32*)(wsp + OFF_INFLAG + xl * 32 * 64 + j * 4);
    const u32 vd = lane * 16;
    auto report = [&](u32 pi) {
      if (lane == 0) {
        const u32 old = __hip_atomic_fetch_add(ctl + pi, 1u, __ATOMIC_RELAXED, __HIP_MEMORY_SCOPE_WORKGROUP);
        if (old == NKB - 1) __hip_atomic_store(flag + pi * 16, seq, __ATOMIC_RELAXED, __HIP_MEMORY_SCOPE_SYSTEM);
      }
    };
    u32 landed = 0, prev = 32, slot = wv - 1;
    for (u32 gi = wv - 1; gi < NG; gi += 3) {
      const u32 pi = gi / NKB, kb = gi - pi * NKB;
      char* db = dbase + a.pdst[pi] + kb * 16384;
      while (landed <= gi) { landed = lds_get(ctl + C_LANDED); if (landed <= gi) __builtin_amdgcn_s_sleep(1); }
      const LAS u32x4* rs = (const LAS u32x4*)(shm + slot * 4096) + lane;
      const u32x4 d0 = rs[0], d1 = rs[64], d2 = rs[128], d3 = rs[192];
      lds_set(ctl + C_CONS + wv - 1, gi + 1);
      wst16_sys<3>(vd, d0, db); wst16_sys<3>(vd, d1, db + 1024); wst16_sys<3>(vd, d2, db + 2048); wst16_sys<3>(vd, d3, db + 3072);
      if (prev < 32) report(prev);
      prev = pi;
      slot = slot + 3 >= RING ? slot + 3 - RING : slot + 3;
    }
    asm volatile("s_waitcnt vmcnt(0)" ::: "memory");
    report(prev);
  }
}

__device__ __forceinline__ void push_finals(const Args& a, u32 rank, u32 p, u32 j, u32 wv, u32 lane, u32 seq, u64 t0,
                                            LAS char* shm, u32 lds0) {
  LAS u32* ctl = (LAS u32*)(shm + LDS_CTL);
  const u32 v0 = (lane >> 4) * 8192 + (lane & 15) * 16, v1 = v0 + 32768, v2 = v0 + 65536, v3 = v0 + 98304;
  if (wv == 0) {
    const char* src = a.out[rank] + j * 32 * 8192;
    const GAS u32* ff = (const GAS u32*)(a.ws[rank] + OFF_FFLAG);
    u32 nready = 0, slot = 0, s3 = 0;
    bool bad = false;
    for (u32 gi = 0; gi < 256; ++gi) {
      const u32 t = gi >> 1;
      if (nready <= t) {
        asm volatile("s_waitcnt vmcnt(0)" ::: "memory");
        lds_set(ctl + C_LANDED, gi);
        do {
          const u32 f0 = __hip_atomic_load(ff + lane, __ATOMIC_RELAXED, __HIP_MEMORY_SCOPE_AGENT);
          const u32 f1 = __hip_atomic_load(ff + 64 + lane, __ATOMIC_RELAXED, __HIP_MEMORY_SCOPE_AGENT);
          const u64 n0 = __ballot(f0 < seq), n1 = __ballot(f1 < seq);
          nready = n0 ? (u32)__builtin_ctzll(n0) : (n1 ? 64u + (u32)__builtin_ctzll(n1) : 128u);
          if (nready <= t) {
            if (expired(t0)) { bad = true; nready = 128; }
            __builtin_amdgcn_s_sleep(4);
          }
        } while (nready <= t);
      }
      ring_space(ctl, gi, s3);
      glds4v<true>(v0, v1, v2, v3, src + a.foff[t] + (gi & 1) * 16 * 8192, lds0 + slot * 4096);
      ring_publish(ctl, gi);
      slot = slot + 1 == RING ? 0 : slot + 1; s3 = s3 == 2 ? 0 : s3 + 1;
    }
    asm volatile("s_waitcnt vmcnt(0)" ::: "memory");
    if (bad) lds_set((LAS u32*)(shm + (255 % RING) * 4096), 0x7fc07fc0u);
    lds_set(ctl + C_LANDED, 256);
  } else {
    char* dst = a.out[p] + j * 32 * 8192;
    u32 landed = 0, slot = wv - 1;
    for (u32 gi = wv - 1; gi < 256; gi += 3) {
      char* db = dst + a.foff[gi >> 1] + (gi & 1) * 16 * 8192;
      while (landed <= gi) { landed = lds_get(ctl + C_LANDED); if (landed <= gi) __builtin_amdgcn_s_sleep(1); }
      const LAS u32x4* rs = (const LAS u32x4*)(shm + slot * 4096) + lane;
      const u32x4 d0 = rs[0], d1 = rs[64], d2 = rs[128], d3 = rs[192];
      lds_set(ctl + C_CONS + wv - 1, gi + 1);
      wst16_sys<3>(v0, d0, db); wst16_sys<3>(v1, d1, db); wst16_sys<3>(v2, d2, db); wst16_sys<3>(v3, d3, db);
      slot = slot + 3 >= RING ? slot + 3 - RING : slot + 3;
    }
  }
}

__device__ __forceinline__ void glds4(u32 voff, const char* sbase, u32 gstep, u32 lds) {
  const char* b1 = sbase + gstep; const char* b2 = b1 + gstep; const char* b3 = b2 + gstep;
  u32 keep;
  asm volatile(
      "s_mov_b32 %0, m0\n\t"
      "s_mov_b32 m0, %6\n\ts_nop 0\n\tglobal_load_lds_dwordx4 %1, %2\n\t"
      "s_add_u32 m0, m0, 0x1000\n\ts_nop 0\n\tglobal_load_lds_dwordx4 %1, %3\n\t"
      "s_add_u32 m0, m0, 0x1000\n\ts_nop 0\n\tglobal_load_lds_dwordx4 %1, %4\n\t"
      "s_add_u32 m0, m0, 0x1000\n\ts_nop 0\n\tglobal_load_lds_dwordx4 %1, %5\n\t"
      "s_mov_b32 m0, %0"
      : "=&s"(keep) : "v"(voff), "s"(sbase), "s"(b1), "s"(b2), "s"(b3), "s"(lds) : "memory", "scc");
}

__device__ __forceinline__ u32 pack2(float lo, float hi) {
  bf16x2 v = {(__bf16)lo, (__bf16)hi};
  return __builtin_bit_cast(u32, v);
}
__device__ __forceinline__ float silu(float y) {
  const float c = __builtin_fminf(__builtin_fmaxf(y, -60.f), 60.f);
  return y * __builtin_amdgcn_rcpf(1.f + __builtin_amdgcn_exp2f(c * -1.44269504088896341f));
}

__global__ void __launch_bounds__(256, 1) fused_gemm_ar_silu(const Args a) {
  __shared__ __attribute__((aligned(1024))) char shm[LDS_BYTES];
  const u32 tid = threadIdx.x, lane = tid & 63;
  const u32 wv = __builtin_amdgcn_readfirstlane(tid >> 6);
  const u32 b = blockIdx.x, xl = b & 7, slot = b >> 3;
  const u32 rank = a.rank & 7;
  const u32 peer = (rank ^ xl) & 7;
  char* const ws = a.ws[rank];
  const u64 t0 = __builtin_amdgcn_s_memrealtime();
  LAS u32* lctr = (LAS u32*)(shm + LDS_CTL);
  const u32 lds0 = (u32)(size_t)(LAS char*)shm;
  STAMP(0);

  const bool is_in = xl != 0 && slot < 4;
  const bool is_fin = xl != 0 && slot >= 4 && slot < 8;
  if (tid < 64) lctr[tid] = 0;
  asm volatile("s_waitcnt lgkmcnt(0)\n\ts_barrier" ::: "memory");

  GAS u32* ctr = (GAS u32*)(ws + OFF_CTR) + b;
  u32 seq = 0;
  if (!(is_in && wv == 0))
    seq = __builtin_amdgcn_readfirstlane(__hip_atomic_load(ctr, __ATOMIC_RELAXED, __HIP_MEMORY_SCOPE_AGENT)) + 1;
  if (wv == 3) {
    __builtin_amdgcn_fence(__ATOMIC_ACQUIRE, "");
    asm volatile("s_waitcnt vmcnt(0)" ::: "memory");
  }
  if (is_in) {
    if (xl < 4) push_inputs<5>(a, peer, xl, slot, wv, lane, seq, (LAS char*)shm, lds0);
    else push_inputs<3>(a, peer, xl, slot, wv, lane, seq, (LAS char*)shm, lds0);
    if (wv == 0) seq = __builtin_amdgcn_readfirstlane(__hip_atomic_load(ctr, __ATOMIC_RELAXED, __HIP_MEMORY_SCOPE_AGENT)) + 1;
    asm volatile("s_waitcnt lgkmcnt(0)\n\ts_barrier" ::: "memory");
    STAMP(1);
  }

  const u32 vx = (8 * wv + (lane >> 3)) * 1024 + (((lane & 7) ^ (4 * (wv & 1) + (lane >> 4))) * 16);
  const u32 vw = (4 * wv + (lane >> 4)) * 8192 + (((lane & 15) ^ (((lane >> 4) | ((wv >> 1) << 2)) << 1)) * 16);

  const u32 kind = xl < 4 ? 1u : 0u;
  const u32 blk = (rank ^ xl) & 3;
  const u32 rb = blk >> 1, cb = blk & 1;
  const u32 ord = xl == 0 ? slot : (slot < 4 ? 28 + slot : slot - 4);
  const u32 te = a.tile[kind * 4 + blk][ord];
  const u32 i4 = te & 3, j8 = (te >> 2) & 7, rpos = (te >> 5) & 31, fpos = (te >> 10) & 127;
  const u32 lr = kind * 8 + rb * 4 + i4, lc = cb * 8 + j8;
  const u32 row0 = grow0(rank, lr), col0 = gcol0(rank, lc);
  const char* xs = ws + OFF_XS + (size_t)lr * 32 * 16384 + wv * 1024;
  const char* wsg = ws + OFF_WS + (size_t)lc * 32 * 16384 + wv * 1024;
  const char* xo = a.x + (size_t)row0 * 1024;
  const char* wo = a.w + (size_t)col0 * 2;
  const u32 vl = lane * 16;
  const GAS u32x4* myflag = (const GAS u32x4*)(ws + OFF_INFLAG + ((lane & 7) * 32 + rpos) * 64);
  u32 mask = 1;
  bool bad = false;

  auto issue = [&](u32 kb) {
    const u32 l = lds0 + (kb & (NSTAGE - 1)) * STAGE_BYTES + wv * 1024;
    if (kb < 5) {
      glds4(vx, xo + kb * 128, 32 * 1024, l);
      glds4(vw, wo + (size_t)kb * 64 * 8192, 16 * 8192, l + 16384);
    } else {
      const u32 need = kb < 17 ? 4 + (kb - 5) / 3 : 1 + (kb - 17) / 5;
      while (!((mask >> need) & 1)) {
        u32x4 f;
        asm volatile("global_load_dwordx4 %0, %1, off sc0 sc1\n\ts_waitcnt vmcnt(0)" : "=&v"(f) : "v"(myflag) : "memory");
        const bool ok = f.x >= seq && f.y >= seq && f.z >= seq && f.w >= seq;
        mask = ((u32)__ballot(ok) & 0xffu) | 1u;
        if (!((mask >> need) & 1)) {
          if (expired(t0)) { bad = true; mask = 0xff; break; }
          __builtin_amdgcn_s_sleep(4);
        }
      }
      glds4(vl, xs + (size_t)kb * 16384, 4096, l);
      glds4(vl, wsg + (size_t)kb * 16384, 4096, l + 16384);
    }
  };

  const u32 wn = wv & 1, wm = wv >> 1;
  const u32 g = lane >> 4, q = (lane >> 2) & 3, pp = lane & 3;
  u32 ax[2];
#pragma unroll
  for (int ks = 0; ks < 2; ++ks) ax[ks] = (wm * 64 + (lane & 15)) * 128 + (((4 * ks + g) ^ ((lane & 15) >> 1)) * 16);
  u32 aw[4];
#pragma unroll
  for (int nt = 0; nt < 4; ++nt)
    aw[nt] = 16384 + (8 * g + q) * 256 + (((2 * (wn * 4 + nt) + (pp >> 1)) ^ ((q | ((g & 1) << 2)) << 1)) * 16) + (pp & 1) * 8;

  f32x4 acc[4][4];
#pragma unroll
  for (int i = 0; i < 4; ++i)
#pragma unroll
    for (int jj = 0; jj < 4; ++jj) acc[i][jj] = (f32x4){0.f, 0.f, 0.f, 0.f};

  issue(0); issue(1); issue(2);
  for (u32 t = 0; t < 32; ++t) {
    if (t < 30) asm volatile("s_waitcnt vmcnt(16) lgkmcnt(0)\n\ts_barrier" ::: "memory");
    else if (t == 30) asm volatile("s_waitcnt vmcnt(8) lgkmcnt(0)\n\ts_barrier" ::: "memory");
    else asm volatile("s_waitcnt vmcnt(0) lgkmcnt(0)\n\ts_barrier" ::: "memory");
    if (t + 3 < 32) issue(t + 3);
    const LAS char* st = (const LAS char*)shm + (t & (NSTAGE - 1)) * STAGE_BYTES;
#pragma unroll
    for (int ks = 0; ks < 2; ++ks) {
      bf16x8 fx[4], fw[4];
#pragma unroll
      for (int nt = 0; nt < 4; ++nt) {
        const bf16x4 lo = __builtin_amdgcn_ds_read_tr16_b64_v4bf16((LAS bf16x4*)(st + aw[nt] + ks * 8192));
        const bf16x4 hi = __builtin_amdgcn_ds_read_tr16_b64_v4bf16((LAS bf16x4*)(st + aw[nt] + ks * 8192 + 1024));
        fw[nt] = __builtin_shufflevector(lo, hi, 0, 1, 2, 3, 4, 5, 6, 7);
      }
#pragma unroll
      for (int mt = 0; mt < 4; ++mt) fx[mt] = *(const LAS bf16x8*)(st + ax[ks] + mt * 2048);
#pragma unroll
      for (int mt = 0; mt < 4; ++mt)
#pragma unroll
        for (int nt = 0; nt < 4; ++nt)
          acc[nt][mt] = __builtin_amdgcn_mfma_f32_16x16x32_bf16(fw[nt], fx[mt], acc[nt][mt], 0, 0, 0);
    }
  }
  STAMP(5);

  const u32 tix = i4 * 8 + j8;
  const float qnan = __builtin_nanf("");
  if (kind == 0) {
    char* wsp = a.ws[peer];
    const u32 pt = (rank & 3) * 32 + tix;
    char* db = wsp + OFF_PS + (size_t)pt * 32768 + wv * 8192;
#pragma unroll
    for (int mt = 0; mt < 4; ++mt)
#pragma unroll
      for (int np = 0; np < 2; ++np) {
        f32x4 u = acc[2 * np][mt], v = acc[2 * np + 1][mt];
        if (bad) u[0] = qnan;
        const u32x4 o = {pack2(u[0], u[1]), pack2(u[2], u[3]), pack2(v[0], v[1]), pack2(v[2], v[3])};
        st16_sys(vl, o, db + (mt * 2 + np) * 1024);
      }
    asm volatile("s_waitcnt vmcnt(0)\n\ts_barrier" ::: "memory");
    if (tid == 0) __hip_atomic_store((GAS u32*)(wsp + OFF_PFLAG + pt * 64), seq, __ATOMIC_RELAXED, __HIP_MEMORY_SCOPE_SYSTEM);
  } else {
    const u32 pt = blk * 32 + tix;
    const GAS u32* pf = (const GAS u32*)(ws + OFF_PFLAG + pt * 64);
    while (__builtin_amdgcn_readfirstlane(__hip_atomic_load(pf, __ATOMIC_RELAXED, __HIP_MEMORY_SCOPE_SYSTEM)) < seq) {
      if (expired(t0)) { bad = true; break; }
      __builtin_amdgcn_s_sleep(4);
    }
    STAMP(6);
    const char* sb = ws + OFF_PS + (size_t)pt * 32768 + wv * 8192;
    u32x4 pr[8];
#pragma unroll
    for (int i = 0; i < 8; ++i) ld16(pr[i], vl, sb + i * 1024);
#pragma unroll
    for (int i = 0; i < 8; ++i) waitv<0>(pr[i]);
    char* ob = a.out[rank] + (size_t)(row0 + wm * 64) * 8192 + (col0 + wn * 64) * 2;
    const u32 vo = (lane & 15) * 8192 + (g & 1) * 32 + (g >> 1) * 16;
#pragma unroll
    for (int mt = 0; mt < 4; ++mt)
#pragma unroll
      for (int np = 0; np < 2; ++np) {
        const u32x4 pv = pr[mt * 2 + np];
        f32x4 u = acc[2 * np][mt], v = acc[2 * np + 1][mt];
        u[0] += __uint_as_float(pv.x << 16); u[1] += __uint_as_float(pv.x & 0xffff0000u);
        u[2] += __uint_as_float(pv.y << 16); u[3] += __uint_as_float(pv.y & 0xffff0000u);
        v[0] += __uint_as_float(pv.z << 16); v[1] += __uint_as_float(pv.z & 0xffff0000u);
        v[2] += __uint_as_float(pv.w << 16); v[3] += __uint_as_float(pv.w & 0xffff0000u);
        if (bad) u[0] = qnan;
        const u32 a0 = pack2(silu(u[0]), silu(u[1])), a1 = pack2(silu(u[2]), silu(u[3]));
        const u32 b0 = pack2(silu(v[0]), silu(v[1])), b1 = pack2(silu(v[2]), silu(v[3]));
        const auto s0 = __builtin_amdgcn_permlane16_swap(a0, b0, false, false);
        const auto s1 = __builtin_amdgcn_permlane16_swap(a1, b1, false, false);
        const u32x4 o = {s0[0], s1[0], s0[1], s1[1]};
        st16_agent(vo, o, ob + (size_t)mt * 16 * 8192 + np * 64);
      }
    asm volatile("s_waitcnt vmcnt(0)\n\ts_barrier" ::: "memory");
    if (tid == 0) __hip_atomic_store((GAS u32*)(ws + OFF_FFLAG) + fpos, seq, __ATOMIC_RELAXED, __HIP_MEMORY_SCOPE_AGENT);
  }
  if (tid == 0) __hip_atomic_store(ctr, seq, __ATOMIC_RELAXED, __HIP_MEMORY_SCOPE_AGENT);
  STAMP(7);

  if (is_fin) {
    const u32 j = slot - 4;
    push_finals(a, rank, peer, j, wv, lane, seq, t0, (LAS char*)shm, lds0);
    asm volatile("s_waitcnt vmcnt(0)\n\ts_barrier" ::: "memory");
    STAMP(8);
    if (tid == 0) {
      __hip_atomic_store((GAS u32*)(a.ws[peer] + OFF_DONE + xl * 128) + j, seq, __ATOMIC_RELAXED, __HIP_MEMORY_SCOPE_SYSTEM);
      const GAS u32* df = (const GAS u32*)(ws + OFF_DONE + xl * 128) + j;
      while (__hip_atomic_load(df, __ATOMIC_RELAXED, __HIP_MEMORY_SCOPE_SYSTEM) < seq) {
        if (expired(t0)) {
          __hip_atomic_store((GAS u32*)(ws + OFF_TMO), seq, __ATOMIC_RELAXED, __HIP_MEMORY_SCOPE_AGENT);
          *(GAS u32*)(a.out[rank] + (size_t)grow0(rank, 8) * 8192 + gcol0(rank, 0) * 2) = 0x7fc07fc0u;
          break;
        }
        __builtin_amdgcn_s_sleep(2);
      }
    }
    STAMP(9);
  }
}

static void build_tables(Args& a) {
  int n = 0;
  u32 piece[32];
  for (int i = 0; i < 4; ++i) {
    piece[n++] = 0 + i;
    piece[n++] = 16 + i;
    piece[n++] = 4 + i;
    piece[n++] = 16 + 8 + i;
  }
  for (int i = 4; i < 8; ++i) { piece[n++] = 16 + i; piece[n++] = 16 + 8 + i; }
  for (int i = 0; i < 4; ++i) { piece[n++] = 8 + i; piece[n++] = 12 + i; }
  int pos[32];
  for (int i = 0; i < 32; ++i) pos[piece[i]] = i;
  for (int i = 0; i < 32; ++i) {
    const u32 ch = piece[i] & 15;
    a.pdst[i] = (u32)((piece[i] & 16) ? OFF_WS : OFF_XS) + ch * 32 * 16384;
    for (u32 xl = 0; xl < 8; ++xl) {
      const u32 p = a.rank ^ xl, k0 = xl < 4 ? 0 : 320;
      a.psrc[xl][i] = (piece[i] & 16) ? (0x80000000u | (k0 * 8192 + gcol0(p, ch) * 2)) : grow0(p, ch) * 1024 + k0 * 2;
    }
  }
  int key[8][32], idx[8][32];
  for (int hb = 0; hb < 8; ++hb) {
    const int kind = hb >> 2, rb = (hb >> 1) & 1, cb = hb & 1;
    for (int t = 0; t < 32; ++t) {
      const int pa = pos[kind * 8 + rb * 4 + (t & 3)], pb = pos[16 + cb * 8 + (t >> 2)];
      key[hb][t] = (pa > pb ? pa : pb) * 64 + (pa < pb ? pa : pb);
      idx[hb][t] = t;
    }
    for (int i = 1; i < 32; ++i)
      for (int j = i; j > 0 && key[hb][idx[hb][j]] < key[hb][idx[hb][j - 1]]; --j) { int s = idx[hb][j]; idx[hb][j] = idx[hb][j - 1]; idx[hb][j - 1] = s; }
  }
  int fk[128], fi[128];
  for (int blk = 0; blk < 4; ++blk)
    for (int o = 0; o < 32; ++o) {
      const bool late = o >= 28 && ((a.rank ^ blk) & 3) != 0;
      fk[blk * 32 + o] = key[4 + blk][idx[4 + blk][o]] + (late ? 100000 : 0);
      fi[blk * 32 + o] = blk * 32 + o;
    }
  for (int i = 1; i < 128; ++i)
    for (int j = i; j > 0 && fk[fi[j]] < fk[fi[j - 1]]; --j) { int s = fi[j]; fi[j] = fi[j - 1]; fi[j - 1] = s; }
  int fpos[128];
  for (int i = 0; i < 128; ++i) {
    const int blk = fi[i] >> 5, t = idx[4 + blk][fi[i] & 31];
    fpos[fi[i]] = i;
    a.foff[i] = grow0(a.rank, 8 + (blk >> 1) * 4 + (t & 3)) * 8192 + gcol0(a.rank, (blk & 1) * 8 + (t >> 2)) * 2;
  }
  for (int hb = 0; hb < 8; ++hb)
    for (int o = 0; o < 32; ++o) {
      const int t = idx[hb][o];
      a.tile[hb][o] = (t & 3) | ((t >> 2) << 2) | ((key[hb][t] >> 6) << 5) | ((hb >= 4 ? fpos[(hb - 4) * 32 + o] : 0) << 10);
    }
}

extern "C" void kernel_launch(int rank, int nranks, ncclComm_t comm, hipStream_t stream,
                              void* const* d_in, const int* in_sizes, int n_in,
                              void* d_out, int out_size, const int* in_dtypes, int out_dtype,
                              void* const* d_out_all, void* const* d_ws_all, size_t ws_bytes) {
  if (nranks != 8 || (unsigned)rank >= 8u || n_in < 2 || ws_bytes < WS_NEED || out_size != 4096 * 4096 ||
      in_sizes[0] != 4096 * 512 || in_sizes[1] != 512 * 4096) {
    fprintf(stderr, "kernel_launch: unexpected problem shape\n");
    return;
  }
  Args a;
  a.x = (const char*)d_in[0];
  a.w = (const char*)d_in[1];
  for (int p = 0; p < 8; ++p) { a.out[p] = (char*)d_out_all[p]; a.ws[p] = (char*)d_ws_all[p]; }
  a.rank = (u32)rank;
  a.pad = 0;
  build_tables(a);
  fused_gemm_ar_silu<<<256, 256, 0, stream>>>(a);
  hipError_t e = hipGetLastError();
  if (e != hipSuccess) fprintf(stderr, "launch failed: %s\n", hipGetErrorString(e));
}
```

```cpp
#include <hip/hip_runtime.h>
#include <rccl/rccl.h>
#include <cstdio>
#include <cstdint>

typedef unsigned int u32;
typedef unsigned long long u64;
typedef __attribute__((ext_vector_type(4))) float f32x4;
typedef __attribute__((ext_vector_type(4))) unsigned int u32x4;
typedef __attribute__((ext_vector_type(4))) __bf16 bf16x4;
typedef __attribute__((ext_vector_type(8))) __bf16 bf16x8;
typedef __attribute__((ext_vector_type(2))) __bf16 bf16x2;
#define LAS __attribute__((address_space(3)))
#define GAS __attribute__((address_space(1)))
#define RLX __ATOMIC_RELAXED
#define SYS __HIP_MEMORY_SCOPE_SYSTEM
#define AGT __HIP_MEMORY_SCOPE_AGENT
#define WGP __HIP_MEMORY_SCOPE_WORKGROUP

#define PULL 1

constexpr size_t OFF_CTR = 0;
constexpr size_t OFF_TMO = 4096;
constexpr size_t OFF_TICK = 5120;
constexpr size_t OFF_INFLAG = 8192;
constexpr size_t OFF_PFLAG = 16384;
constexpr size_t OFF_QUEUE = 32768;
constexpr size_t OFF_DONE = 53248;
constexpr size_t OFF_GATE = 57344;
constexpr size_t OFF_DBG = 65536;
constexpr size_t OFF_XS = 1u << 20;
constexpr size_t OFF_WS = 9u << 20;
constexpr size_t OFF_PS = 17u << 20;
constexpr size_t OFF_PO = 21u << 20;
constexpr size_t WS_NEED = 25u << 20;

constexpr u64 DEADLINE = 100000000ull;
constexpr int NSTAGE = 4;
constexpr int STAGE_BYTES = 32768;
constexpr int LDS_CTL = NSTAGE * STAGE_BYTES;
constexpr int LDS_BYTES = LDS_CTL + 2048;

#define PIECE_ORDER 0, 4, 16, 24, 1, 5, 17, 25, 2, 6, 18, 26, 8, 3, 7, 19, 27, 12, 20, 28, 21, 29, 9, 13, 22, 30, 23, 31, 10, 14, 11, 15
constexpr u32 PIECES[32] = {PIECE_ORDER};
constexpr u64 pack_pieces(int first) {
  u64 v = 0;
  for (int i = 0; i < 8; ++i) v |= (u64)PIECES[first + i] << (8 * i);
  return v;
}
__device__ __forceinline__ u32 piece_at(u32 i) {
  constexpr u64 p0 = pack_pieces(0), p1 = pack_pieces(8), p2 = pack_pieces(16), p3 = pack_pieces(24);
  const u64 v = i < 16 ? (i < 8 ? p0 : p1) : (i < 24 ? p2 : p3);
  return (u32)(v >> (8 * (i & 7))) & 31u;
}

struct Args {
  const char* x;
  const char* w;
  u32 rank;
  u32 xmap;
  char* ws[8];
  char* out[8];
  u32 tile[256];
};

#define STAMP(i) do {} while (0)

__device__ __forceinline__ u32 grow0(u32 r, u32 lr) {
  const u32 g = r >> 2, qi = (r >> 1) & 1, kind = lr >> 3, rb = ((lr >> 2) & 1) ^ qi, i4 = lr & 3;
  const u32 RB = g ? qi + 2 * rb : 2 * qi + rb;
  return RB * 1024 + (kind ^ g ^ 1) * 512 + i4 * 128;
}
__device__ __forceinline__ u32 gcol0(u32 r, u32 lc) {
  const u32 g = r >> 2, qj = r & 1, cb = (lc >> 3) ^ qj, j8 = lc & 7;
  const u32 CB = g ? qj + 2 * cb : 2 * qj + cb;
  return CB * 1024 + j8 * 128;
}

__device__ __forceinline__ void st16_sys(u32 voff, u32x4 v, char* sbase) {
  asm volatile("global_store_dwordx4 %0, %1, %2 sc0 sc1\n\ts_nop 1" :: "v"(voff), "v"(v), "s"(sbase) : "memory");
}
__device__ __forceinline__ void st16_agent(u32 voff, u32x4 v, char* sbase) {
  asm volatile("global_store_dwordx4 %0, %1, %2 sc1\n\ts_nop 1" :: "v"(voff), "v"(v), "s"(sbase) : "memory");
}
__device__ __forceinline__ bool expired(u64 t0) { return __builtin_amdgcn_s_memrealtime() - t0 > DEADLINE; }
#define WAIT_VM0() asm volatile("s_waitcnt vmcnt(0)" ::: "memory")
#define WG_BARRIER() asm volatile("s_waitcnt lgkmcnt(0)\n\ts_barrier" ::: "memory")

constexpr u32 RING = 24;
#define LDEPTH_IN 7
#define LDEPTH_FIN (PULL ? 3 : 7)
#define NST_IN 3
#define NST_FIN 3
#define LOADER_PRIO 2
#define POLL_SLEEP 2
#define START_DELAY 100
#define START_SLOPE 60
#define GATE_LEAD 3
#define GATE_PIECE_S 30
#define GATE_PIECE_C 28
constexpr u32 C_CNT = 0;
constexpr u32 C_LANDED = 32, C_TICKET = 33, C_BAD = 34, C_MASK = 35, C_NF = 36, C_NP = 37;
constexpr u32 C_FREE = 64, C_DESC = 96;
constexpr u32 C_LIST = 128;

template <int F>
__device__ __forceinline__ void glds4v(u32 v0, u32 v1, u32 v2, u32 v3, const char* sbase, u32 lds) {
#define GLDS4V(S) asm volatile( \
      "s_mov_b32 m0, %5\n\ts_nop 0\n\t" \
      "global_load_lds_dwordx4 %0, %4" S "\n\t" \
      "global_load_lds_dwordx4 %1, %4 offset:1024" S "\n\t" \
      "global_load_lds_dwordx4 %2, %4 offset:2048" S "\n\t" \
      "global_load_lds_dwordx4 %3, %4 offset:3072" S \
      :: "v"(v0), "v"(v1), "v"(v2), "v"(v3), "s"(sbase), "s"(lds) : "memory")
  if constexpr (F == 0) GLDS4V(""); else if constexpr (F == 1) GLDS4V(" sc1"); else GLDS4V(" sc0 sc1");
}
__device__ __forceinline__ u32 src_x(u32 s, u32 lane) {
  return (8 * s + (lane >> 3)) * 1024 + (((lane & 7) ^ (4 * (s & 1) + (lane >> 4))) * 16) - s * 1024;
}
__device__ __forceinline__ u32 src_w(u32 s, u32 lane) {
  return (4 * s + (lane >> 4)) * 8192 + (((lane & 15) ^ (((lane >> 4) | ((s >> 1) << 2)) << 1)) * 16) - s * 1024;
}
__device__ __forceinline__ u32 lds_get(LAS u32* p) {
  asm volatile("" ::: "memory");
  const u32 v = __builtin_amdgcn_readfirstlane(__hip_atomic_load(p, RLX, WGP));
  asm volatile("" ::: "memory");
  return v;
}
__device__ __forceinline__ void lds_set(LAS u32* p, u32 v) {
  asm volatile("" ::: "memory");
  __hip_atomic_store(p, v, RLX, WGP);
  asm volatile("" ::: "memory");
}
__device__ __forceinline__ void ring_space(LAS u32* ctl, u32 gi, u32 slot) {
  if (gi >= RING && lds_get(ctl + C_FREE + slot) <= gi - RING) {
    WAIT_VM0();
    lds_set(ctl + C_LANDED, gi);
    while (lds_get(ctl + C_FREE + slot) <= gi - RING) __builtin_amdgcn_s_sleep(1);
  }
}
template <u32 DEPTH>
__device__ __forceinline__ void ring_publish(LAS u32* ctl, u32 gi) {
  asm volatile("s_waitcnt vmcnt(%0)" :: "n"(4 * DEPTH) : "memory");
  if (gi >= DEPTH) __hip_atomic_fetch_max(ctl + C_LANDED, gi + 1 - DEPTH, RLX, WGP);
  asm volatile("" ::: "memory");
}
__device__ __forceinline__ u32 lds_add(LAS u32* p, u32 lane) {
  u32 t = 0;
  if (lane == 0) t = __hip_atomic_fetch_add(p, 1u, RLX, WGP);
  return __builtin_amdgcn_readfirstlane(t);
}
template <class Store, class Done>
__device__ __forceinline__ void ring_storer(LAS char* shm, u32 ng, u32 lane, Store store, Done done) {
  LAS u32* ctl = (LAS u32*)(shm + LDS_CTL);
  u32 landed = 0, prev = ~0u;
#pragma unroll 1
  for (;;) {
    const u32 gi = lds_add(ctl + C_TICKET, lane);
    if (gi >= ng) break;
    if (landed <= gi) {
      landed = lds_get(ctl + C_LANDED);
      if (landed <= gi) {
        if (prev != ~0u) { WAIT_VM0(); done(prev); prev = ~0u; }
        do { __builtin_amdgcn_s_sleep(1); landed = lds_get(ctl + C_LANDED); } while (landed <= gi);
      }
    }
    const u32 slot = gi % RING;
    const u32 desc = lds_get(ctl + C_DESC + slot);
    const LAS u32x4* rs = (const LAS u32x4*)(shm + slot * 4096) + lane;
    const u32x4 d0 = rs[0], d1 = rs[64], d2 = rs[128], d3 = rs[192];
    lds_set(ctl + C_FREE + slot, gi + 1);
    WAIT_VM0();
    store(gi, desc, d0, d1, d2, d3);
    if (prev != ~0u) done(prev);
    prev = desc;
  }
  WAIT_VM0();
  if (prev != ~0u) done(prev);
}

__device__ __forceinline__ void inputs_loader(const char* x, const char* w, u32 p, u32 nkb, u32 j, u32 lane,
                                              LAS u32* ctl, u32 lds0) {
  u32 vx[4], vw[4];
#pragma unroll
  for (u32 s = 0; s < 4; ++s) {
    vx[s] = src_x(s, lane);
    vw[s] = src_w(s, lane);
  }
  const u32 k0 = nkb == 5 ? 0u : 320u;
  const char* xb = x + (j * 32 * 1024 + k0 * 2);
  const char* wb = w + (j * 16 + k0) * 8192;
  u32 gi = 0, slot = 0;
#pragma unroll 1
  for (u32 i = 0; i < 32; ++i) {
    const u32 pc = piece_at(i);
    const bool isw = pc >> 4;
    const char* sb = isw ? wb + gcol0(p, pc & 15) * 2 : xb + grow0(p, pc) * 1024;
#pragma unroll 1
    for (u32 kb = 0; kb < nkb; ++kb) {
      ring_space(ctl, gi, slot);
      if (isw) { glds4v<0>(vw[0], vw[1], vw[2], vw[3], sb, lds0 + slot * 4096); sb += 64 * 8192; }
      else { glds4v<0>(vx[0], vx[1], vx[2], vx[3], sb, lds0 + slot * 4096); sb += 128; }
      lds_set(ctl + C_DESC + slot, (pc * 32 + kb) * 32 + i);
      ring_publish<LDEPTH_IN>(ctl, gi);
      ++gi; slot = slot + 1 == RING ? 0 : slot + 1;
    }
  }
  WAIT_VM0();
  lds_set(ctl + C_LANDED, gi);
}

__device__ __forceinline__ void inputs_storer(const Args& a, u32 xl, u32 nkb, u32 j, u32 lane, LAS char* shm) {
  LAS u32* ctl = (LAS u32*)(shm + LDS_CTL);
  const u32 kboff = nkb == 5 ? 12 + 5 * xl : 3 * xl - 7;
  char* const dbase = a.ws[xl] + (OFF_XS + kboff * 16384 + j * 4096);
  const u32 vd = lane * 16;
  ring_storer(shm, 32 * nkb, lane,
    [&](u32 gi, u32 desc, u32x4 d0, u32x4 d1, u32x4 d2, u32x4 d3) {
      char* db = dbase + (desc >> 5) * 16384;
      st16_sys(vd, d0, db); st16_sys(vd, d1, db + 1024); st16_sys(vd, d2, db + 2048); st16_sys(vd, d3, db + 3072);
    },
    [&](u32 desc) { if (lane == 0) __hip_atomic_fetch_add(ctl + C_CNT + (desc & 31), 1u, RLX, WGP); });
}
__device__ __forceinline__ void inputs_flagger(const Args& a, u32 xl, u32 nkb, u32 j, u32 lane, u32 seq, LAS u32* ctl) {
  GAS u32* const flag = (GAS u32*)(a.ws[xl] + OFF_INFLAG + xl * 16 + j * 4);
  GAS u32* const gate = (GAS u32*)(a.ws[0] + OFF_GATE + xl * 128) + j;
  bool open = PULL;
#pragma unroll 1
  for (u32 pi = 0; pi < 32; ++pi) {
    for (;;) {
      if (!open && lds_get(ctl + C_TICKET) >= 32 * nkb - GATE_LEAD) {
        open = true;
        if (lane == 0) __hip_atomic_store(gate, seq, RLX, AGT);
      }
      if (lds_get(ctl + C_CNT + pi) >= nkb) break;
      __builtin_amdgcn_s_sleep(1);
    }
    if (lane == 0) __hip_atomic_store(flag + pi * 64, seq, RLX, SYS);
  }
  if (!open && lane == 0) __hip_atomic_store(gate, seq, RLX, AGT);
}

__device__ __forceinline__ void move_tiles(const Args& a, u32 rank, u32 xl, u32 j, u32 wv, u32 lane, u32 seq, u64 t0,
                                           LAS char* shm, u32 lds0) {
  LAS u32* ctl = (LAS u32*)(shm + LDS_CTL);
  const u32 np_tot = xl >= 4 ? 32u : 0u;
  const u32 ng = 256 + 2 * np_tot;
  const u32 f0 = (lane >> 4) * 8192 + (lane & 15) * 16, f1 = f0 + 32768, f2 = f0 + 65536, f3 = f0 + 98304;
  const u32 p0 = lane * 16, p1 = p0 + 1024, p2 = p0 + 2048, p3 = p0 + 3072;
  const u32 sl = PULL ? xl : 0u, dl = PULL ? 0u : xl;
  if (wv == 0) {
    const char* fsrc = a.out[sl] + j * 32 * 8192;
    const char* psrc = a.ws[sl] + (OFF_PO + (xl & 3) * 32 * 32768 + j * 8192);
    const u32 prod = rank ^ sl;
    u32 nf = 0, np = 0, af = 0, ap = 0, slot = 0;
#pragma unroll 1
    for (u32 gi = 0; gi < ng; gi += 2) {
      if (np < np_tot && np >= ap) ap = lds_get(ctl + C_NP);
      if (np >= ap && nf >= af) {
        af = lds_get(ctl + C_NF);
        if (nf >= af) {
          WAIT_VM0();
          lds_set(ctl + C_LANDED, gi);
          do { __builtin_amdgcn_s_sleep(1); ap = lds_get(ctl + C_NP); af = lds_get(ctl + C_NF); } while (np >= ap && nf >= af);
        }
      }
      const bool isp = np < ap;
      const u32 id = lds_get(ctl + C_LIST + (isp ? 128 + np : nf));
      u32 off;
      if (isp) { ++np; off = (id & 31) * 32768; }
      else { ++nf; off = grow0(prod, 8 + ((id >> 6) & 1) * 4 + ((id >> 3) & 3)) * 8192 + gcol0(prod, ((id >> 5) & 1) * 8 + (id & 7)) * 2; }
#pragma unroll
      for (u32 h = 0; h < 2; ++h) {
        ring_space(ctl, gi + h, slot);
        if (isp) glds4v<PULL ? 2 : 1>(p0, p0, p0, p0, psrc + off + h * 4096, lds0 + slot * 4096);
        else glds4v<PULL ? 2 : 1>(f0, f1 - 1024, f2 - 2048, f3 - 3072, fsrc + off + h * 16 * 8192, lds0 + slot * 4096);
        lds_set(ctl + C_DESC + slot, isp ? off + h * 4096 + 1 : off + h * 16 * 8192);
        ring_publish<LDEPTH_FIN>(ctl, gi + h);
        slot = slot + 1 == RING ? 0 : slot + 1;
      }
    }
    WAIT_VM0();
    lds_set(ctl + C_LANDED, ng);
  } else if (wv <= NST_FIN) {
    char* fdst = a.out[dl] + j * 32 * 8192;
    char* pdst = a.ws[dl] + (OFF_PS + (xl & 3) * 32 * 32768 + j * 8192);
    GAS u32* pflag = (GAS u32*)(a.ws[dl] + OFF_PFLAG + (xl & 3) * 32 * 64) + j;
    if (!PULL) {
      const GAS u32* gate = (const GAS u32*)(a.ws[0] + OFF_GATE + xl * 128) + j;
      while (__builtin_amdgcn_readfirstlane(__hip_atomic_load(gate, RLX, AGT)) < seq) {
        if (expired(t0)) break;
        __builtin_amdgcn_s_sleep(4);
      }
    }
    ring_storer(shm, ng, lane,
      [&](u32 gi, u32 desc, u32x4 d0, u32x4 d1, u32x4 d2, u32x4 d3) {
        if (lds_get(ctl + C_BAD)) d0.x = 0x7fc07fc0u;
        if (desc & 1) {
          char* db = pdst + (desc - 1);
          if (PULL) { st16_agent(p0, d0, db); st16_agent(p1, d1, db); st16_agent(p2, d2, db); st16_agent(p3, d3, db); }
          else { st16_sys(p0, d0, db); st16_sys(p1, d1, db); st16_sys(p2, d2, db); st16_sys(p3, d3, db); }
        } else {
          char* db = fdst + desc;
          if (PULL) { st16_agent(f0, d0, db); st16_agent(f1, d1, db); st16_agent(f2, d2, db); st16_agent(f3, d3, db); }
          else { st16_sys(f0, d0, db); st16_sys(f1, d1, db); st16_sys(f2, d2, db); st16_sys(f3, d3, db); }
        }
      },
      [&](u32 desc) {
        if ((desc & 1) && lds_add(ctl + C_CNT + (desc >> 15), lane) == 1 && lane == 0)
          __hip_atomic_store(pflag + (desc >> 15) * 16, seq, RLX, SYS);
      });
  } else if (wv == 7) {
    if (PULL) {
      const GAS u32* gate = (const GAS u32*)(a.ws[0] + OFF_INFLAG + (xl < 4 ? GATE_PIECE_S : GATE_PIECE_C) * 256 + xl * 16) + j;
      while (__builtin_amdgcn_readfirstlane(__hip_atomic_load(gate, RLX, SYS)) < seq) {
        if (expired(t0)) break;
        __builtin_amdgcn_s_sleep(4);
      }
    }
    const GAS u32* qf = (const GAS u32*)(a.ws[0] + OFF_QUEUE + sl * 1024) + lane;
    const GAS u32* qp = (const GAS u32*)(a.ws[0] + OFF_QUEUE + xl * 1024 + 512) + (lane & 31);
    u32 nf = 0, np = 0;
    for (;;) {
      u32 e0 = __hip_atomic_load(qf, RLX, SYS), e1 = __hip_atomic_load(qf + 64, RLX, SYS), e2 = __hip_atomic_load(qp, RLX, SYS);
      const bool late = expired(t0);
      if (late) { lds_set(ctl + C_BAD, 1); e0 = (seq << 8) | lane; e1 = (seq << 8) | (64 + lane); e2 = (seq << 8) | (lane & 31); }
      const u64 n0 = __ballot((e0 >> 8) != (seq & 0xffffff)), n1 = __ballot((e1 >> 8) != (seq & 0xffffff));
      const u32 n2 = (u32)__ballot((e2 >> 8) != (seq & 0xffffff));
      const u32 cf = n0 ? (u32)__builtin_ctzll(n0) : (n1 ? 64u + (u32)__builtin_ctzll(n1) : 128u);
      const u32 cp = np_tot ? (n2 ? (u32)__builtin_ctz(n2) : 32u) : 0u;
      if (cf != nf || cp != np) {
        if (lane >= nf) ctl[C_LIST + lane] = e0 & 127;
        if (64 + lane >= nf) ctl[C_LIST + 64 + lane] = e1 & 127;
        if (lane < 32 && lane >= np) ctl[C_LIST + 128 + lane] = e2 & 31;
        lds_set(ctl + C_NP, cp);
        lds_set(ctl + C_NF, cf);
        nf = cf; np = cp;
      }
      if (nf == 128 && np == np_tot) break;
      __builtin_amdgcn_s_sleep(8);
    }
  }
}

__device__ __forceinline__ u32 pack2(float lo, float hi) {
  bf16x2 v = {(__bf16)lo, (__bf16)hi};
  return __builtin_bit_cast(u32, v);
}
__device__ __forceinline__ float silu(float y) {
  const float c = __builtin_fminf(__builtin_fmaxf(y, -60.f), 60.f);
  return y * __builtin_amdgcn_rcpf(1.f + __builtin_amdgcn_exp2f(c * -1.44269504088896341f));
}

__global__ void __launch_bounds__(512, 2) fused_gemm_ar_silu(const Args a) {
  __shared__ __attribute__((aligned(1024))) char shm[LDS_BYTES];
  const u32 tid = threadIdx.x, lane = tid & 63;
  const u32 wv = __builtin_amdgcn_readfirstlane(tid >> 6);
  const u32 b = blockIdx.x, slot = b >> 3;
  const u32 xl = (a.xmap >> (4 * (b & 7))) & 7;
  LAS u32* lctr = (LAS u32*)(shm + LDS_CTL);
  const u32 lds0 = (u32)(size_t)(LAS char*)shm;
  const bool is_in = xl != 0 && slot < 4;
  const bool is_fin = xl != 0 && slot >= 4 && slot < 8;
  const u32 nkb = xl < 4 ? 5u : 3u;
  const u64 t0 = __builtin_amdgcn_s_memrealtime();
  const u32 rank = a.rank & 7;
  if (wv == 0) {
    lctr[lane] = 0; lctr[64 + lane] = 0;
    WG_BARRIER();
    if (is_in) inputs_loader(a.x, a.w, rank ^ xl, nkb, slot, lane, lctr, lds0);
  } else {
    asm volatile("s_barrier" ::: "memory");
    if (is_in && wv <= NST_IN) inputs_storer(a, xl, nkb, slot, lane, (LAS char*)shm);
  }
  char* const ws = a.ws[0];
  GAS u32* ctr = (GAS u32*)(ws + OFF_CTR) + b;
  const u32 seq = __builtin_amdgcn_readfirstlane(__hip_atomic_load(ctr, RLX, AGT)) + 1;
  const u32 te = a.tile[xl * 32 + slot];
  const u32 i4 = te & 3, j8 = (te >> 2) & 7, rpos = (te >> 5) & 31;
  if (wv == 7 && is_in) inputs_flagger(a, xl, nkb, slot, lane, seq, lctr);
  if (is_in) {
    WG_BARRIER();
    if (wv == 0) STAMP(1);
  }

  const u32 kind = xl < 4 ? 1u : 0u;
  const u32 blk = xl & 3;
  const u32 lr = kind * 8 + (blk >> 1) * 4 + i4, lc = (blk & 1) * 8 + j8;
  const u32 row0 = grow0(rank, lr), col0 = gcol0(rank, lc);
  const u32 tix = blk * 32 + i4 * 8 + j8;
  const u32 vl = lane * 16;

  if (wv >= 4) {
    const u32 w4 = wv - 4;
    const char* xo = a.x + (size_t)(row0 + w4 * 32) * 1024;
    const char* wo = a.w + ((size_t)w4 * 16 * 8192 + col0 * 2);
    const char* xs = ws + (OFF_XS + (size_t)lr * 32 * 16384 + w4 * 4096);
    const char* wsg = ws + (OFF_WS + (size_t)lc * 32 * 16384 + w4 * 4096);
    const u32 lq = lds0 + w4 * 4096;
    const GAS u32x4* inflag = (const GAS u32x4*)(ws + OFF_INFLAG + rpos * 256 + (lane & 7) * 16);
    const GAS u32x4* lastflag = (lane & 7) ? inflag : (const GAS u32x4*)(ws + OFF_PFLAG + tix * 64);
    { const u32 d = START_DELAY + rpos * START_SLOPE; while (__builtin_amdgcn_s_memrealtime() - t0 < d) __builtin_amdgcn_s_sleep(4); }
    if (wv == 7) {
      __builtin_amdgcn_fence(__ATOMIC_ACQUIRE, "");
      WAIT_VM0();
    }
    __builtin_amdgcn_s_setprio(LOADER_PRIO);
    {
      const u32 x0 = src_x(0, lane), x1 = src_x(1, lane), x2 = src_x(2, lane), x3 = src_x(3, lane);
      const u32 y0 = src_w(0, lane), y1 = src_w(1, lane), y2 = src_w(2, lane), y3 = src_w(3, lane);
#pragma unroll
      for (u32 kb = 0; kb < 5; ++kb) {
        if (kb >= 3) asm volatile("s_waitcnt vmcnt(16)\n\ts_barrier" ::: "memory");
        glds4v<0>(x0, x1, x2, x3, xo + kb * 128, lq + (kb & 3) * STAGE_BYTES);
        glds4v<0>(y0, y1, y2, y3, wo + (size_t)kb * 64 * 8192, lq + (kb & 3) * STAGE_BYTES + 16384);
      }
    }
    const u32 nst = 32 + kind;
    u32 avail = 5, issued = 5;
    auto refresh = [&]() {
      u32 mask;
      if (w4 == 0) {
        u32x4 f;
        const GAS u32x4* fp = issued >= 17 ? lastflag : inflag;
        asm volatile("global_load_dwordx4 %0, %1, off sc0 sc1\n\ts_waitcnt vmcnt(0)" : "=&v"(f) : "v"(fp) : "memory");
        mask = (u32)__ballot(f.x >= seq && f.y >= seq && f.z >= seq && f.w >= seq) & 0xffu;
        if (expired(t0)) { lds_set(lctr + C_BAD, 1); mask = 0xff; }
        lds_set(lctr + C_MASK, mask);
      } else {
        mask = lds_get(lctr + C_MASK);
      }
      const u32 m = (mask >> 4) | ((mask & 0xe) << 3) | ((mask & 1) << 7);
      const u32 n = __builtin_ctz(~m);
      const u32 av = n <= 4 ? 5 + 3 * n : (n <= 7 ? 5 * n - 3 : 33);
      avail = av < nst ? av : nst;
    };
#pragma unroll 1
    for (u32 s = 2; s <= 32; ++s) {
      while (issued <= s + 2) {
        if (issued >= avail) {
          if (issued >= nst) break;
          refresh();
          while (issued >= avail && issued == s) { __builtin_amdgcn_s_sleep(POLL_SLEEP); refresh(); }
          if (issued >= avail) break;
        }
        const u32 l = lq + (issued & 3) * STAGE_BYTES;
        const char* sx = xs + issued * 16384;
        const char* sw = wsg + issued * 16384;
        if (issued == 32) { sx = ws + (OFF_PS + (size_t)tix * 32768 + w4 * 4096); sw = sx + 16384; }
        glds4v<0>(vl, vl, vl, vl, sx, l);
        glds4v<0>(vl, vl, vl, vl, sw, l + 16384);
        ++issued;
      }
      const u32 ahead = issued - s;
      if (ahead >= 3) asm volatile("s_waitcnt vmcnt(16)\n\ts_barrier" ::: "memory");
      else if (ahead == 2) asm volatile("s_waitcnt vmcnt(8)\n\ts_barrier" ::: "memory");
      else asm volatile("s_waitcnt vmcnt(0) lgkmcnt(0)\n\ts_barrier" ::: "memory");
    }
    __builtin_amdgcn_s_setprio(0);
    u32 pos = 0;
    if (wv == 4 && lane == 0) pos = __hip_atomic_fetch_add((GAS u32*)(ws + OFF_TICK + (kind ? 0 : xl) * 128), 1u, RLX, AGT);
    pos = __builtin_amdgcn_readfirstlane(pos);
    WG_BARRIER();
    if (wv == 4) {
      const u32 e = (seq << 8) | tix;
      if (kind) {
        if (PULL) { if (lane >= 1 && lane < 8) __hip_atomic_store((GAS u32*)(a.ws[lane & 7] + OFF_QUEUE + lane * 1024) + (pos & 127), e, RLX, SYS); }
        else if (lane == 0) __hip_atomic_store((GAS u32*)(ws + OFF_QUEUE) + (pos & 127), e, RLX, AGT);
      } else if (lane == 0) {
        __hip_atomic_store((GAS u32*)(a.ws[PULL ? xl : 0] + OFF_QUEUE + xl * 1024 + 512) + (pos & 31), e, RLX, SYS);
      }
      if (lane == 0) __hip_atomic_store(ctr, seq, RLX, AGT);
      STAMP(7);
    }
  } else {
    const u32 wn = wv & 1, wm = wv >> 1;
    const u32 g = lane >> 4, q = (lane >> 2) & 3, pp = lane & 3;
    u32 ax[2];
#pragma unroll
    for (int ks = 0; ks < 2; ++ks) ax[ks] = (wm * 64 + (lane & 15)) * 128 + (((4 * ks + g) ^ ((lane & 15) >> 1)) * 16);
    u32 aw[4];
#pragma unroll
    for (int nt = 0; nt < 4; ++nt)
      aw[nt] = 16384 + (8 * g + q) * 256 + (((2 * (wn * 4 + nt) + (pp >> 1)) ^ ((q | ((g & 1) << 2)) << 1)) * 16) + (pp & 1) * 8;

    f32x4 acc[4][4];
#pragma unroll
    for (int i = 0; i < 4; ++i)
#pragma unroll
      for (int jj = 0; jj < 4; ++jj) acc[i][jj] = (f32x4){0.f, 0.f, 0.f, 0.f};
    bf16x8 xa[4], wa[4], xb[4], wb[4];
    auto rd = [&](bf16x8* fx, bf16x8* fw, u32 t, int ks) {
      const LAS char* st = (const LAS char*)shm + (t & (NSTAGE - 1)) * STAGE_BYTES;
#pragma unroll
      for (int i = 0; i < 4; ++i) {
        const bf16x4 lo = __builtin_amdgcn_ds_read_tr16_b64_v4bf16((LAS bf16x4*)(st + aw[i] + ks * 8192));
        const bf16x4 hi = __builtin_amdgcn_ds_read_tr16_b64_v4bf16((LAS bf16x4*)(st + aw[i] + ks * 8192 + 1024));
        fw[i] = __builtin_shufflevector(lo, hi, 0, 1, 2, 3, 4, 5, 6, 7);
        fx[i] = *(const LAS bf16x8*)(st + ax[ks] + i * 2048);
      }
    };
    auto mm = [&](const bf16x8* fx, const bf16x8* fw) {
#pragma unroll
      for (int mt = 0; mt < 4; ++mt)
#pragma unroll
        for (int nt = 0; nt < 4; ++nt)
          acc[nt][mt] = __builtin_amdgcn_mfma_f32_16x16x32_bf16(fw[nt], fx[mt], acc[nt][mt], 0, 0, 0);
    };
#define INTERLEAVE() do { _Pragma("unroll") for (int i_ = 0; i_ < 12; ++i_) { \
      __builtin_amdgcn_sched_group_barrier(0x8, 1, 0); __builtin_amdgcn_sched_group_barrier(0x100, 1, 0); } \
      __builtin_amdgcn_sched_group_barrier(0x8, 4, 0); } while (0)
    WG_BARRIER();
    rd(xa, wa, 0, 0);
    __builtin_amdgcn_sched_barrier(0);
    for (u32 t = 0; t < 31; ++t) {
      rd(xb, wb, t, 1);
      mm(xa, wa);
      INTERLEAVE();
      __builtin_amdgcn_sched_barrier(0);
      WG_BARRIER();
      __builtin_amdgcn_sched_barrier(0);
      rd(xa, wa, t + 1, 0);
      mm(xb, wb);
      INTERLEAVE();
      __builtin_amdgcn_sched_barrier(0);
    }
    rd(xb, wb, 31, 1);
    mm(xa, wa);
    INTERLEAVE();
    __builtin_amdgcn_sched_barrier(0);
    mm(xb, wb);
    if (wv == 0) STAMP(5);
    WG_BARRIER();
    const bool bad = lds_get(lctr + C_BAD) != 0;
    const float qnan = __builtin_nanf("");
    if (kind == 0) {
      char* db = ws + OFF_PO + (size_t)tix * 32768 + wv * 8192;
#pragma unroll
      for (int mt = 0; mt < 4; ++mt)
#pragma unroll
        for (int np = 0; np < 2; ++np) {
          f32x4 u = acc[2 * np][mt], v = acc[2 * np + 1][mt];
          if (bad) u[0] = qnan;
          const u32x4 o = {pack2(u[0], u[1]), pack2(u[2], u[3]), pack2(v[0], v[1]), pack2(v[2], v[3])};
          st16_sys(vl, o, db + (mt * 2 + np) * 1024);
        }
    } else {
      const LAS u32x4* pr = (const LAS u32x4*)(shm + (32 & (NSTAGE - 1)) * STAGE_BYTES + wv * 8192) + lane;
      char* ob = a.out[0] + (size_t)(row0 + wm * 64) * 8192 + (col0 + wn * 64) * 2;
      const u32 vo = (lane & 15) * 8192 + (g & 1) * 32 + (g >> 1) * 16;
#pragma unroll
      for (int mt = 0; mt < 4; ++mt)
#pragma unroll
        for (int np = 0; np < 2; ++np) {
          const u32x4 pv = pr[(mt * 2 + np) * 64];
          f32x4 u = acc[2 * np][mt], v = acc[2 * np + 1][mt];
          u[0] += __uint_as_float(pv.x << 16); u[1] += __uint_as_float(pv.x & 0xffff0000u);
          u[2] += __uint_as_float(pv.y << 16); u[3] += __uint_as_float(pv.y & 0xffff0000u);
          v[0] += __uint_as_float(pv.z << 16); v[1] += __uint_as_float(pv.z & 0xffff0000u);
          v[2] += __uint_as_float(pv.w << 16); v[3] += __uint_as_float(pv.w & 0xffff0000u);
          if (bad) u[0] = qnan;
          const u32 a0 = pack2(silu(u[0]), silu(u[1])), a1 = pack2(silu(u[2]), silu(u[3]));
          const u32 b0 = pack2(silu(v[0]), silu(v[1])), b1 = pack2(silu(v[2]), silu(v[3]));
          const auto s0 = __builtin_amdgcn_permlane16_swap(a0, b0, false, false);
          const auto s1 = __builtin_amdgcn_permlane16_swap(a1, b1, false, false);
          const u32x4 o = {s0[0], s1[0], s0[1], s1[1]};
          st16_sys(vo, o, ob + (size_t)mt * 16 * 8192 + np * 64);
        }
    }
    WAIT_VM0();
    WG_BARRIER();
  }

  if (is_fin) {
    const u32 j = slot - 4;
    move_tiles(a, rank, xl, j, wv, lane, seq, t0, (LAS char*)shm, lds0);
    if (!PULL && tid == 0) {
      __hip_atomic_store((GAS u32*)(a.ws[xl] + OFF_DONE + xl * 128) + j, seq, RLX, SYS);
      const GAS u32* df = (const GAS u32*)(ws + OFF_DONE + xl * 128) + j;
      while (__hip_atomic_load(df, RLX, SYS) < seq) {
        if (expired(t0)) {
          __hip_atomic_store((GAS u32*)(ws + OFF_TMO), seq, RLX, AGT);
          *(GAS u32*)(a.out[0] + (size_t)grow0(rank, 8) * 8192 + gcol0(rank, 0) * 2) = 0x7fc07fc0u;
          break;
        }
        __builtin_amdgcn_s_sleep(2);
      }
    }
    if (wv == 0) STAMP(9);
  }
}

static void build_tables(Args& a) {
  int pos[32];
  for (int i = 0; i < 32; ++i) pos[PIECES[i]] = i;
  for (int xl = 0; xl < 8; ++xl) {
    const int kind = xl < 4, rb = (xl >> 1) & 1, cb = xl & 1;
    int rdy[32], key[32], idx[32];
    for (int t = 0; t < 32; ++t) {
      const int pa = pos[kind * 8 + rb * 4 + (t & 3)], pb = pos[16 + cb * 8 + (t >> 2)];
      rdy[t] = pa > pb ? pa : pb;
      key[t] = rdy[t] * 64 + (pa < pb ? pa : pb);
      idx[t] = t;
    }
    for (int i = 1; i < 32; ++i)
      for (int j = i; j > 0 && key[idx[j]] < key[idx[j - 1]]; --j) { int s = idx[j]; idx[j] = idx[j - 1]; idx[j - 1] = s; }
    for (int slot = 0; slot < 32; ++slot) {
      const int t = idx[xl == 0 ? slot : (slot < 4 ? 28 + slot : slot - 4)];
      a.tile[xl * 32 + slot] = (t & 3) | ((t >> 2) << 2) | (rdy[t] << 5);
    }
  }
}

extern "C" void kernel_launch(int rank, int nranks, ncclComm_t comm, hipStream_t stream,
                              void* const* d_in, const int* in_sizes, int n_in,
                              void* d_out, int out_size, const int* in_dtypes, int out_dtype,
                              void* const* d_out_all, void* const* d_ws_all, size_t ws_bytes) {
  if (nranks != 8 || (unsigned)rank >= 8u || n_in < 2 || ws_bytes < WS_NEED || out_size != 4096 * 4096 ||
      in_sizes[0] != 4096 * 512 || in_sizes[1] != 512 * 4096) {
    fprintf(stderr, "kernel_launch: unexpected problem shape\n");
    return;
  }
  Args a;
  a.x = (const char*)d_in[0];
  a.w = (const char*)d_in[1];
  for (int l = 0; l < 8; ++l) { a.out[l] = (char*)d_out_all[rank ^ l]; a.ws[l] = (char*)d_ws_all[rank ^ l]; }
  a.rank = (u32)rank;
  static const unsigned char near_xcd[8][8] = {{0, 4, 2, 3, 6, 5, 1, 7}, {4, 0, 7, 5, 2, 3, 6, 1}, {2, 7, 0, 1, 4, 5, 3, 6}, {3, 5, 1, 0, 7, 4, 2, 6},
                                               {5, 2, 4, 7, 0, 1, 6, 3}, {6, 3, 5, 4, 1, 0, 7, 2}, {1, 6, 3, 2, 5, 7, 0, 4}, {7, 1, 6, 5, 3, 2, 4, 0}};
  a.xmap = 0;
  for (int p = 0; p < 8; ++p) a.xmap |= (u32)(rank ^ p) << (4 * near_xcd[rank][p]);
  build_tables(a);
  fused_gemm_ar_silu<<<256, 512, 0, stream>>>(a);
  hipError_t e = hipGetLastError();
  if (e != hipSuccess) fprintf(stderr, "launch failed: %s\n", hipGetErrorString(e));
}
```

```cpp
#include <hip/hip_runtime.h>
#include <rccl/rccl.h>
#include <cstdio>
#include <cstdint>

typedef unsigned int u32;
typedef unsigned long long u64;
typedef __attribute__((ext_vector_type(4))) float f32x4;
typedef __attribute__((ext_vector_type(4))) unsigned int u32x4;
typedef __attribute__((ext_vector_type(2))) unsigned int u32x2;
typedef __attribute__((ext_vector_type(4))) __bf16 bf16x4;
typedef __attribute__((ext_vector_type(8))) __bf16 bf16x8;
typedef __attribute__((ext_vector_type(2))) __bf16 bf16x2;
#define LAS __attribute__((address_space(3)))
#define GAS __attribute__((address_space(1)))
#define RLX __ATOMIC_RELAXED
#define SYS __HIP_MEMORY_SCOPE_SYSTEM
#define AGT __HIP_MEMORY_SCOPE_AGENT
#define WGP __HIP_MEMORY_SCOPE_WORKGROUP

constexpr size_t OFF_CTR = 0;
constexpr size_t OFF_TMO = 4096;
constexpr size_t OFF_TICK = 5120;
constexpr size_t OFF_INFLAG = 8192;
constexpr size_t OFF_PFLAG = 16384;
constexpr size_t OFF_QUEUE = 32768;
constexpr size_t OFF_SFLAG = 45056;
constexpr size_t OFF_DBG = 65536;
constexpr size_t OFF_XS = 1u << 20;
constexpr size_t OFF_WS = 9u << 20;
constexpr size_t OFF_PS = 17u << 20;
constexpr size_t OFF_PO = 21u << 20;
constexpr size_t OFF_SX = 25u << 20;
constexpr size_t OFF_SW = 29u << 20;
constexpr size_t WS_NEED = 33u << 20;

constexpr u64 DEADLINE = 100000000ull;
constexpr int NSTAGE = 4;
constexpr int STAGE_BYTES = 32768;
constexpr int LDS_CTL = NSTAGE * STAGE_BYTES;
constexpr int LDS_BYTES = LDS_CTL + 2048;

#define PIECE_ORDER 0, 4, 16, 24, 1, 5, 17, 25, 2, 6, 18, 26, 8, 3, 7, 19, 27, 12, 20, 28, 21, 29, 9, 13, 22, 30, 23, 31, 10, 14, 11, 15
constexpr u32 PIECES[32] = {PIECE_ORDER};
constexpr u64 pack_pieces(int first) {
  u64 v = 0;
  for (int i = 0; i < 8; ++i) v |= (u64)PIECES[first + i] << (8 * i);
  return v;
}
__device__ __forceinline__ u32 piece_at(u32 i) {
  constexpr u64 p0 = pack_pieces(0), p1 = pack_pieces(8), p2 = pack_pieces(16), p3 = pack_pieces(24);
  const u64 v = i < 16 ? (i < 8 ? p0 : p1) : (i < 24 ? p2 : p3);
  return (u32)(v >> (8 * (i & 7))) & 31u;
}

constexpr u64 pack_pos(int first) {
  u64 v = 0;
  for (int i = 0; i < 32; ++i) if ((int)PIECES[i] >= first && (int)PIECES[i] < first + 8) v |= (u64)i << (8 * (PIECES[i] - first));
  return v;
}
__device__ __forceinline__ u32 pos_of(u32 c) {
  constexpr u64 p0 = pack_pos(0), p1 = pack_pos(8), p2 = pack_pos(16), p3 = pack_pos(24);
  const u64 v = c < 16 ? (c < 8 ? p0 : p1) : (c < 24 ? p2 : p3);
  return (u32)(v >> (8 * (c & 7))) & 31u;
}
#define NP0_S 1
#define NP0_C 2

struct Args {
  const char* x;
  const char* w;
  u32 rank;
  u32 xmap;
  char* ws[8];
  char* out[8];
  u32 tile[256];
};

#define STAMP(i) do {} while (0)
#define TSTAMP(i) do {} while (0)

__device__ __forceinline__ u32 grow0(u32 r, u32 lr) {
  const u32 g = r >> 2, qi = (r >> 1) & 1, kind = lr >> 3, rb = ((lr >> 2) & 1) ^ qi, i4 = lr & 3;
  const u32 RB = g ? qi + 2 * rb : 2 * qi + rb;
  return RB * 1024 + (kind ^ g ^ 1) * 512 + i4 * 128;
}
__device__ __forceinline__ u32 gcol0(u32 r, u32 lc) {
  const u32 g = r >> 2, qj = r & 1, cb = (lc >> 3) ^ qj, j8 = lc & 7;
  const u32 CB = g ? qj + 2 * cb : 2 * qj + cb;
  return CB * 1024 + j8 * 128;
}

__device__ __forceinline__ u32 tile_off(u32 r, u32 id) {
  return grow0(r, 8 + ((id >> 6) & 1) * 4 + ((id >> 3) & 3)) * 8192 + gcol0(r, ((id >> 5) & 1) * 8 + (id & 7)) * 2;
}

__device__ __forceinline__ void st16_sys(u32 voff, u32x4 v, char* sbase) {
  asm volatile("global_store_dwordx4 %0, %1, %2 sc0 sc1\n\ts_nop 1" :: "v"(voff), "v"(v), "s"(sbase) : "memory");
}
__device__ __forceinline__ void st16_agent(u32 voff, u32x4 v, char* sbase) {
  asm volatile("global_store_dwordx4 %0, %1, %2 sc1\n\ts_nop 1" :: "v"(voff), "v"(v), "s"(sbase) : "memory");
}
__device__ __forceinline__ bool expired(u64 t0) { return __builtin_amdgcn_s_memrealtime() - t0 > DEADLINE; }
#define WAIT_VM0() asm volatile("s_waitcnt vmcnt(0)" ::: "memory")
#define WG_BARRIER() asm volatile("s_waitcnt lgkmcnt(0)\n\ts_barrier" ::: "memory")

#define WIN 8
#define WINC1 1
#define WINC2 2
#define WINC3 3
#define NST 4
constexpr u32 RING = 24, RING0 = 8192;
#define SCHED_AHEAD 3
#define LOADER_PRIO 2
#define POLL_SLEEP 2
#define START_DELAY 100
#define START_SLOPE 60
#define STAGE_DELAY 80
#define EARLY_MASK 0xffu
constexpr u32 C_CNT = 0;
constexpr u32 C_LISTED = 64, C_TICKET = 65, C_BAD = 66, C_MASK = 67;
constexpr u32 C_AVAIL = 68;
constexpr u32 C_LANDED = 72, C_TAKEN = 73, C_FREE = 80;
constexpr u32 C_LIST = 128;

__device__ __forceinline__ void glds4v(u32 v0, u32 v1, u32 v2, u32 v3, const char* sbase, u32 lds) {
  asm volatile(
      "s_mov_b32 m0, %5\n\ts_nop 0\n\t"
      "global_load_lds_dwordx4 %0, %4\n\t"
      "global_load_lds_dwordx4 %1, %4 offset:1024\n\t"
      "global_load_lds_dwordx4 %2, %4 offset:2048\n\t"
      "global_load_lds_dwordx4 %3, %4 offset:3072"
      :: "v"(v0), "v"(v1), "v"(v2), "v"(v3), "s"(sbase), "s"(lds) : "memory");
}
__device__ __forceinline__ void pull4l(u32 v0, u32 v1, u32 v2, u32 v3, const char* sbase, u32 lds) {
  asm volatile(
      "s_mov_b32 m0, %5\n\ts_waitcnt vmcnt(%6)\n\t"
      "global_load_lds_dwordx4 %0, %4 sc0 sc1\n\ts_waitcnt vmcnt(%7)\n\t"
      "global_load_lds_dwordx4 %1, %4 offset:1024 sc0 sc1\n\ts_waitcnt vmcnt(%8)\n\t"
      "global_load_lds_dwordx4 %2, %4 offset:2048 sc0 sc1\n\ts_waitcnt vmcnt(%9)\n\t"
      "global_load_lds_dwordx4 %3, %4 offset:3072 sc0 sc1"
      :: "v"(v0), "v"(v1), "v"(v2), "v"(v3), "s"(sbase), "s"(lds), "n"(WIN), "n"(WIN + WINC1), "n"(WIN + WINC2), "n"(WIN + WINC3) : "memory");
}
__device__ __forceinline__ u32 src_x(u32 s, u32 lane) {
  return (8 * s + (lane >> 3)) * 1024 + (((lane & 7) ^ (4 * (s & 1) + (lane >> 4))) * 16) - s * 1024;
}
__device__ __forceinline__ u32 src_w(u32 s, u32 lane) {
  return (4 * s + (lane >> 4)) * 8192 + (((lane & 15) ^ (((lane >> 4) | ((s >> 1) << 2)) << 1)) * 16) - s * 1024;
}
__device__ __forceinline__ u32 lds_get(LAS u32* p) {
  asm volatile("" ::: "memory");
  const u32 v = __builtin_amdgcn_readfirstlane(__hip_atomic_load(p, RLX, WGP));
  asm volatile("" ::: "memory");
  return v;
}
__device__ __forceinline__ void lds_set(LAS u32* p, u32 v) {
  asm volatile("" ::: "memory");
  __hip_atomic_store(p, v, RLX, WGP);
  asm volatile("" ::: "memory");
}
__device__ __forceinline__ u32 lds_add(LAS u32* p, u32 lane) {
  u32 t = 0;
  if (lane == 0) t = __hip_atomic_fetch_add(p, 1u, RLX, WGP);
  return __builtin_amdgcn_readfirstlane(t);
}

__device__ __forceinline__ void ld4(u32x4& d0, u32x4& d1, u32x4& d2, u32x4& d3, u32 v0, u32 v1, u32 v2, u32 v3, const char* sb) {
  asm volatile("global_load_dwordx4 %0, %4, %8\n\tglobal_load_dwordx4 %1, %5, %8 offset:1024\n\t"
               "global_load_dwordx4 %2, %6, %8 offset:2048\n\tglobal_load_dwordx4 %3, %7, %8 offset:3072\n\ts_waitcnt vmcnt(0)"
               : "=&v"(d0), "=&v"(d1), "=&v"(d2), "=&v"(d3) : "v"(v0), "v"(v1), "v"(v2), "v"(v3), "s"(sb) : "memory");
}
__device__ __forceinline__ u32 kb_first(u32 xl) { return xl < 4 ? 12 + 5 * xl : 3 * xl - 7; }

__device__ __forceinline__ void push_leading(const Args& a, u32 p, u32 xl, u32 nkb, u32 i, u32 j, u32 kb, u32 lane) {
  const u32 pc = piece_at(i), k0 = (nkb == 5 ? 0u : 320u) + kb * 64;
  u32x4 d0, d1, d2, d3;
  if (pc >> 4) ld4(d0, d1, d2, d3, src_w(0, lane), src_w(1, lane), src_w(2, lane), src_w(3, lane),
                   a.w + ((k0 + j * 16) * 8192 + gcol0(p, pc & 15) * 2));
  else ld4(d0, d1, d2, d3, src_x(0, lane), src_x(1, lane), src_x(2, lane), src_x(3, lane),
           a.x + ((grow0(p, pc) + j * 32) * 1024 + k0 * 2));
  char* db = a.ws[xl] + (OFF_XS + (pc * 32 + kb_first(xl) + kb) * 16384 + j * 4096);
  const u32 vd = lane * 16;
  st16_sys(vd, d0, db); st16_sys(vd, d1, db + 1024); st16_sys(vd, d2, db + 2048); st16_sys(vd, d3, db + 3072);
  WAIT_VM0();
}

#define STAGE_ORDER 0xfedcb76a54932810ull
__device__ __forceinline__ void stage_unit(const Args& a, u32 rank, u32 unit, u32 wv, u32 lane, u32& calls, char*& pws) {
  const u32 j = unit & 3, cls = (u32)(STAGE_ORDER >> (4 * (unit >> 4))) & 15;
  const bool isw = cls >= 8;
  const u32 ch = isw ? ((unit >> 2) & 3) * 8 + (cls & 7) : ((cls & 1) * 4 + ((unit >> 2) & 3)) * 4 + (cls >> 1);
  u32x4 d0, d1, d2, d3;
  if (isw) ld4(d0, d1, d2, d3, src_w(0, lane), src_w(1, lane), src_w(2, lane), src_w(3, lane), a.w + ((wv * 64 + j * 16) * 8192 + ch * 256));
  else ld4(d0, d1, d2, d3, src_x(0, lane), src_x(1, lane), src_x(2, lane), src_x(3, lane), a.x + ((ch * 128 + j * 32) * 1024 + wv * 128));
  char* db = a.ws[0] + ((isw ? OFF_SW : OFF_SX) + (ch * 8 + wv) * 16384 + j * 4096);
  const u32 vd = lane * 16;
  st16_sys(vd, d0, db); st16_sys(vd, d1, db + 1024); st16_sys(vd, d2, db + 2048); st16_sys(vd, d3, db + 3072);
  asm volatile("s_waitcnt vmcnt(0)\n\ts_barrier" : "+v"(calls), "+v"(pws) :: "memory");
  if (wv == 0 && lane >= 1 && lane < 8) {
    const u32 p = rank ^ lane, g = p >> 2, q = isw ? p & 1 : (p >> 1) & 1, B = ch >> 3;
    const u32 lb = (g ? B >> 1 : B & 1) ^ q;
    const u32 code = isw ? 16 + lb * 8 + (ch & 7) : ((((ch >> 2) & 1) ^ g ^ 1) * 8 + lb * 4 + (ch & 3));
    const u32 pos = pos_of(code);
    if ((g ? B & 1 : B >> 1) == q && pos >= (lane < 4 ? NP0_S : NP0_C))
      __hip_atomic_store((GAS u32*)(pws + OFF_SFLAG + (lane * 4 + j) * 128) + pos, calls + 1, RLX, SYS);
  }
}

__device__ __forceinline__ void link_engine(const Args& a, u32 rank, u32 xl, u32 j, u32 wv, u32 lane, u32 seq, u64 t0,
                                            LAS char* shm, u32 lds0) {
  LAS u32* ctl = (LAS u32*)(shm + LDS_CTL);
  const u32 nkb = xl < 4 ? 5u : 3u, np0 = xl < 4 ? NP0_S : NP0_C;
  const u32 ng = (32 - np0) * nkb + (xl < 4 ? 256u : 320u);
  const u32 f0 = (lane >> 4) * 8192 + (lane & 15) * 16, vl = lane * 16;
  char* const ws = a.ws[0];
  volatile LAS u32x2* list = (volatile LAS u32x2*)shm;
  if (wv == 5) {
    const u32 isrc = OFF_SX + (nkb == 5 ? 0 : 5 * 16384) + j * 4096;
    const u32 idst = OFF_XS + kb_first(xl) * 16384 + j * 4096, pq = (xl & 3) * 32 * 32768 + j * 8192;
    u32 gi = 0, ni = np0, nf = 0, left = xl < 4 ? 0u : ~0u;
#pragma unroll 1
    while (gi < ng) {
      u32 ap;
      for (;;) {
        asm volatile("" ::: "memory");
        const u32x4 av = *(volatile LAS u32x4*)(ctl + C_AVAIL);
        const u32 tk = lds_get(ctl + C_TICKET);
        ap = __builtin_amdgcn_readfirstlane(av.y) & left & (ni < 32 ? EARLY_MASK : ~0u);
        if (gi < tk + SCHED_AHEAD &&
            (ap || (ni < 32 ? ni < __builtin_amdgcn_readfirstlane(av.x) : nf < __builtin_amdgcn_readfirstlane(av.z)))) break;
        __builtin_amdgcn_s_sleep(1);
      }
      u32 src, dst, n, step;
      if (ap) {
        const u32 id = __builtin_ctz(ap);
        left &= ~(1u << id);
        src = OFF_PO + pq + id * 32768; dst = OFF_PS + pq + id * 32768 + (32 + id) * 2 + 1; n = 2; step = 4096;
      } else if (ni < 32) {
        const u32 pc = piece_at(ni);
        src = isrc + (pc >> 4 ? 4096 + gcol0(rank, pc & 15) : grow0(rank, pc)) * 1024;
        dst = idst + pc * 32 * 16384 + ni * 2 + 1; n = nkb; step = 16384;
        ++ni;
      } else {
        src = dst = lds_get(ctl + C_LIST + nf); n = 2; step = 16 * 8192;
        ++nf;
      }
      if (lane < n) list[gi + lane] = (u32x2){src + lane * step, dst + lane * step};
      gi += n;
      lds_set(ctl + C_LISTED, gi);
    }
  } else if (wv == 0) {
    constexpr u32 BACK = (WIN + WINC3) / 4;
    const char* const pws = a.ws[xl];
    const char* const fsrc = a.out[xl] + j * 32 * 8192;
    u32 slot = 0, pub = 0;
    __builtin_amdgcn_s_setprio(3);
#pragma unroll 1
    for (u32 gi = 0; gi < ng; ++gi) {
      u32 src, lin;
      for (u32 tries = 0;; ++tries) {
        asm volatile("" ::: "memory");
        const u32 li = __hip_atomic_load(ctl + C_LISTED, RLX, WGP), fr = __hip_atomic_load(ctl + C_FREE + slot, RLX, WGP);
        const u32x2 d = list[gi];
        src = __builtin_amdgcn_readfirstlane(d.x); lin = __builtin_amdgcn_readfirstlane(d.y) & 1;
        if (__builtin_amdgcn_readfirstlane(li) > gi && __builtin_amdgcn_readfirstlane(fr) + RING > gi) break;
        if (tries == 3) {
          WAIT_VM0(); lds_set(ctl + C_LANDED, gi); pub = gi;
        }
        __builtin_amdgcn_s_sleep(1);
      }
      if (lin) pull4l(vl, vl, vl, vl, pws + src, lds0 + RING0 + slot * 4096);
      else pull4l(f0, f0 + 31744, f0 + 63488, f0 + 95232, fsrc + src, lds0 + RING0 + slot * 4096);
      if (gi > pub + BACK) { pub = gi - BACK; lds_set(ctl + C_LANDED, pub); }
      lds_set(ctl + C_TICKET, gi + 1);
      slot = slot + 1 == RING ? 0 : slot + 1;
    }
    WAIT_VM0();
    lds_set(ctl + C_LANDED, ng);
  } else if (wv <= NST) {
    char* const fdst = a.out[0] + j * 32 * 8192;
    GAS u32* pflag = (GAS u32*)(ws + OFF_PFLAG + (xl & 3) * 32 * 64) + j;
    u32 landed = 0, prev = 0;
    auto done = [&]() {
      if (prev & 1) {
        const u32 c = (prev >> 1) & 63;
        if (lds_add(ctl + C_CNT + c, lane) == 1 && c >= 32 && lane == 0) __hip_atomic_store(pflag + (c - 32) * 16, seq, RLX, AGT);
      }
      prev = 0;
    };
#pragma unroll 1
    for (;;) {
      const u32 gi = lds_add(ctl + C_TAKEN, lane);
      if (gi >= ng) break;
      if (landed <= gi) {
        landed = lds_get(ctl + C_LANDED);
        if (landed <= gi) {
          WAIT_VM0();
          done();
          do { __builtin_amdgcn_s_sleep(1); landed = lds_get(ctl + C_LANDED); } while (landed <= gi);
        }
      }
      const u32 slot = gi % RING;
      const u32 desc = __builtin_amdgcn_readfirstlane(list[gi].y);
      const LAS u32x4* rs = (const LAS u32x4*)(shm + RING0 + slot * 4096) + lane;
      u32x4 d0 = rs[0];
      const u32x4 d1 = rs[64], d2 = rs[128], d3 = rs[192];
      lds_set(ctl + C_FREE + slot, gi + 1);
      if (lds_get(ctl + C_BAD)) d0.x = 0x7fc07fc0u;
      WAIT_VM0();
      if (desc & 1) {
        char* db = ws + (desc & ~4095u);
        st16_agent(vl, d0, db); st16_agent(vl, d1, db + 1024); st16_agent(vl, d2, db + 2048); st16_agent(vl, d3, db + 3072);
      } else {
        char* db = fdst + desc;
        st16_agent(f0, d0, db); st16_agent(f0, d1, db + 32768); st16_agent(f0, d2, db + 65536); st16_agent(f0, d3, db + 98304);
      }
      done();
      prev = desc;
    }
    WAIT_VM0();
    done();
  } else if (wv == 6) {
    const GAS u32* qf = (const GAS u32*)(ws + OFF_QUEUE + xl * 1024) + lane;
    const GAS u32* qs = lane < 32 ? (const GAS u32*)(ws + OFF_SFLAG + (xl * 4 + j) * 128) + lane : qf + 96;
    u32 nf = 0;
    for (;;) {
      u32 e0 = __hip_atomic_load(qf, RLX, SYS), e1 = __hip_atomic_load(qf + 64, RLX, SYS), e2 = __hip_atomic_load(qs, RLX, SYS);
      if (expired(t0)) { lds_set(ctl + C_BAD, 1); e0 = (seq << 8) | lane; e1 = (seq << 8) | (64 + lane); e2 = seq; }
      const u64 n0 = __ballot((e0 >> 8) != (seq & 0xffffff)), n1 = __ballot((e1 >> 8) != (seq & 0xffffff));
      const u64 m2 = __ballot(e2 >= seq || lane < np0);
      const u32 cf = n0 ? (u32)__builtin_ctzll(n0) : (n1 ? 64u + (u32)__builtin_ctzll(n1) : 128u);
      const u32 staged = (u32)m2 == ~0u ? 32u : (u32)__builtin_ctz(~(u32)m2), cp = (u32)(m2 >> 32);
      if (cf != nf) {
        if (lane >= nf) ctl[C_LIST + lane] = tile_off(rank ^ xl, e0);
        if (64 + lane >= nf) ctl[C_LIST + 64 + lane] = tile_off(rank ^ xl, e1);
        nf = cf;
      }
      asm volatile("" ::: "memory");
      *(volatile LAS u32x4*)(ctl + C_AVAIL) = (u32x4){staged, cp, cf, 0u};
      if (cf == 128 && (cp == ~0u || xl < 4)) break;
      if (staged == 32) __builtin_amdgcn_s_sleep(8);
    }
  } else if (wv == 7) {
    GAS u32* const flag = (GAS u32*)(ws + OFF_INFLAG + xl * 16 + j * 4);
    const GAS u32* lf = (const GAS u32*)(ws + OFF_SFLAG + (xl * 4 + j) * 128) + (lane & 31);
    while (__ballot(__hip_atomic_load(lf, RLX, SYS) < seq && lane < np0)) {
      if (expired(t0)) { lds_set(ctl + C_BAD, 1); break; }
      __builtin_amdgcn_s_sleep(1);
    }
    if (lane < np0) __hip_atomic_store(flag + lane * 64, seq, RLX, AGT);
#pragma unroll 1
    for (u32 pi = np0; pi < 32; ++pi) {
      while (lds_get(ctl + C_CNT + pi) < nkb) __builtin_amdgcn_s_sleep(1);
      if (lane == 0) __hip_atomic_store(flag + pi * 64, seq, RLX, AGT);
    }
  }
}

__device__ __forceinline__ u32 pack2(float lo, float hi) {
  bf16x2 v = {(__bf16)lo, (__bf16)hi};
  return __builtin_bit_cast(u32, v);
}
__device__ __forceinline__ float silu(float y) {
  const float c = __builtin_fminf(__builtin_fmaxf(y, -60.f), 60.f);
  return y * __builtin_amdgcn_rcpf(1.f + __builtin_amdgcn_exp2f(c * -1.44269504088896341f));
}

__global__ void __launch_bounds__(512, 2) fused_gemm_ar_silu(const Args a) {
  __shared__ __attribute__((aligned(1024))) char shm[LDS_BYTES];
  const u32 tid = threadIdx.x, lane = tid & 63;
  const u32 wv = __builtin_amdgcn_readfirstlane(tid >> 6);
  const u32 b = blockIdx.x, slot = b >> 3;
  const u32 xl = (a.xmap >> (4 * (b & 7))) & 7;
  LAS u32* lctr = (LAS u32*)(shm + LDS_CTL);
  const u32 lds0 = (u32)(size_t)(LAS char*)shm;
  const bool engine = xl != 0 && slot < 4;
  const u32 nkb = xl < 4 ? 5u : 3u;
  const u64 t0 = __builtin_amdgcn_s_memrealtime();
  const u32 rank = a.rank & 7;
  const u32 np0 = xl < 4 ? NP0_S : NP0_C;
  char* const ws = a.ws[0];
  GAS u32* ctr = (GAS u32*)(ws + OFF_CTR) + b;
  u32 calls = __hip_atomic_load(ctr, RLX, AGT);
  char* pws = a.ws[lane & 7];
  if (wv == 7) { lctr[lane] = 0; lctr[64 + lane] = 0; }
  const bool is_lead = xl != 0 && slot >= 20 && slot < 20 + 4 * np0;
  if (is_lead && wv < nkb) push_leading(a, rank ^ xl, xl, nkb, (slot - 20) >> 2, slot & 3, wv, lane);
  if (slot >= 4 && slot < 20) {
    const u32 u = b - 32;
    if (u >= 64) while (__builtin_amdgcn_s_memrealtime() - t0 < STAGE_DELAY) __builtin_amdgcn_s_sleep(2);
    stage_unit(a, rank, u, wv, lane, calls, pws);
    if (wv == 0) STAMP(11);
    stage_unit(a, rank, 128 + u, wv, lane, calls, pws);
    if (wv == 0) STAMP(10);
  }
  asm volatile("" : "+v"(calls));
  const u32 seq = __builtin_amdgcn_readfirstlane(calls) + 1;
  WG_BARRIER();
  if (is_lead && tid == 448)
    __hip_atomic_store((GAS u32*)(a.ws[xl] + OFF_SFLAG + (xl * 4 + (slot & 3)) * 128) + ((slot - 20) >> 2), seq, RLX, SYS);
  if (is_lead && wv == 7) STAMP(11);
  if (tid == 0) __hip_atomic_store(ctr, seq, RLX, AGT);
  if (engine) {
    link_engine(a, rank, xl, slot, wv, lane, seq, t0, (LAS char*)shm, lds0);
    return;
  }

#pragma unroll 1
  for (u32 ts = slot;; ts -= 4) {
  const u32 te = a.tile[xl * 32 + ts];
  const u32 i4 = te & 3, j8 = (te >> 2) & 7, rpos = (te >> 5) & 31;
  const u32 kind = xl < 4 ? 1u : 0u;
  const u32 blk = xl & 3;
  const u32 lr = kind * 8 + (blk >> 1) * 4 + i4, lc = (blk & 1) * 8 + j8;
  const u32 row0 = grow0(rank, lr), col0 = gcol0(rank, lc);
  const u32 tix = blk * 32 + i4 * 8 + j8;
  const u32 vl = lane * 16;

  if (wv >= 4) {
    const u32 w4 = wv - 4;
    const char* xo = a.x + (size_t)(row0 + w4 * 32) * 1024;
    const char* wo = a.w + ((size_t)w4 * 16 * 8192 + col0 * 2);
    const char* xs = ws + (OFF_XS + (size_t)lr * 32 * 16384 + w4 * 4096);
    const char* wsg = ws + (OFF_WS + (size_t)lc * 32 * 16384 + w4 * 4096);
    const u32 lq = lds0 + w4 * 4096;
    const GAS u32x4* inflag = (const GAS u32x4*)(ws + OFF_INFLAG + rpos * 256 + (lane & 7) * 16);
    const GAS u32x4* lastflag = (lane & 7) ? inflag : (const GAS u32x4*)(ws + OFF_PFLAG + tix * 64);
    { const u32 d = START_DELAY + rpos * START_SLOPE; while (__builtin_amdgcn_s_memrealtime() - t0 < d) __builtin_amdgcn_s_sleep(4); }
    if (wv == 7 && ts == slot) {
      __builtin_amdgcn_fence(__ATOMIC_ACQUIRE, "");
      WAIT_VM0();
    }
    __builtin_amdgcn_s_setprio(LOADER_PRIO);
    {
      const u32 x0 = src_x(0, lane), x1 = src_x(1, lane), x2 = src_x(2, lane), x3 = src_x(3, lane);
      const u32 y0 = src_w(0, lane), y1 = src_w(1, lane), y2 = src_w(2, lane), y3 = src_w(3, lane);
#pragma unroll
      for (u32 kb = 0; kb < 5; ++kb) {
        if (kb >= 3) asm volatile("s_waitcnt vmcnt(16)\n\ts_barrier" ::: "memory");
        glds4v(x0, x1, x2, x3, xo + kb * 128, lq + (kb & 3) * STAGE_BYTES);
        glds4v(y0, y1, y2, y3, wo + (size_t)kb * 64 * 8192, lq + (kb & 3) * STAGE_BYTES + 16384);
      }
    }
    const u32 nst = 32 + kind;
    u32 avail = 5, issued = 5;
    auto refresh = [&]() {
      u32 mask;
      if (w4 == 0) {
        u32x4 f;
        const GAS u32x4* fp = issued >= 17 ? lastflag : inflag;
        asm volatile("global_load_dwordx4 %0, %1, off sc0 sc1\n\ts_waitcnt vmcnt(0)" : "=&v"(f) : "v"(fp) : "memory");
        mask = (u32)__ballot(f.x >= seq && f.y >= seq && f.z >= seq && f.w >= seq) & 0xffu;
        if (expired(t0)) { lds_set(lctr + C_BAD, 1); mask = 0xff; }
        lds_set(lctr + C_MASK, mask);
      } else {
        mask = lds_get(lctr + C_MASK);
      }
      const u32 m = (mask >> 4) | ((mask & 0xe) << 3) | ((mask & 1) << 7);
      const u32 n = __builtin_ctz(~m);
      const u32 av = n <= 4 ? 5 + 3 * n : (n <= 7 ? 5 * n - 3 : 33);
      avail = av < nst ? av : nst;
    };
#pragma unroll 1
    for (u32 s = 2; s <= 32; ++s) {
      while (issued <= s + 2) {
        if (issued >= avail) {
          if (issued >= nst) break;
          refresh();
          while (issued >= avail && issued == s) { __builtin_amdgcn_s_sleep(POLL_SLEEP); refresh(); }
          if (issued >= avail) break;
        }
        const u32 l = lq + (issued & 3) * STAGE_BYTES;
        const char* sx = xs + issued * 16384;
        const char* sw = wsg + issued * 16384;
        if (issued == 32) { sx = ws + (OFF_PS + (size_t)tix * 32768 + w4 * 4096); sw = sx + 16384; }
        glds4v(vl, vl, vl, vl, sx, l);
        glds4v(vl, vl, vl, vl, sw, l + 16384);
        ++issued;
      }
      const u32 ahead = issued - s;
      if (ahead >= 3) asm volatile("s_waitcnt vmcnt(16)\n\ts_barrier" ::: "memory");
      else if (ahead == 2) asm volatile("s_waitcnt vmcnt(8)\n\ts_barrier" ::: "memory");
      else asm volatile("s_waitcnt vmcnt(0) lgkmcnt(0)\n\ts_barrier" ::: "memory");
    }
    __builtin_amdgcn_s_setprio(0);
    u32 pos = 0;
    if (kind && wv == 4 && lane == 0) pos = __hip_atomic_fetch_add((GAS u32*)(ws + OFF_TICK), 1u, RLX, AGT);
    pos = __builtin_amdgcn_readfirstlane(pos);
    if (wv == 4) lds_set(lctr + C_MASK, 0);
    WG_BARRIER();
    if (wv == 4) {
      const u32 e = (seq << 8) | tix;
      if (kind) {
        if (lane >= 1 && lane < 8) __hip_atomic_store((GAS u32*)(pws + OFF_QUEUE + lane * 1024) + (pos & 127), e, RLX, SYS);
      } else if (lane == 0) {
        __hip_atomic_store((GAS u32*)(a.ws[xl] + OFF_QUEUE + xl * 1024 + 512) + (tix & 31), seq, RLX, SYS);
      }
      TSTAMP(7);
    }
  } else {
    const u32 wn = wv & 1, wm = wv >> 1;
    const u32 g = lane >> 4, q = (lane >> 2) & 3, pp = lane & 3;
    u32 ax[2];
#pragma unroll
    for (int ks = 0; ks < 2; ++ks) ax[ks] = (wm * 64 + (lane & 15)) * 128 + (((4 * ks + g) ^ ((lane & 15) >> 1)) * 16);
    u32 aw[4];
#pragma unroll
    for (int nt = 0; nt < 4; ++nt)
      aw[nt] = 16384 + (8 * g + q) * 256 + (((2 * (wn * 4 + nt) + (pp >> 1)) ^ ((q | ((g & 1) << 2)) << 1)) * 16) + (pp & 1) * 8;

    f32x4 acc[4][4];
#pragma unroll
    for (int i = 0; i < 4; ++i)
#pragma unroll
      for (int jj = 0; jj < 4; ++jj) acc[i][jj] = (f32x4){0.f, 0.f, 0.f, 0.f};
    bf16x8 xa[4], wa[4], xb[4], wb[4];
    auto rd = [&](bf16x8* fx, bf16x8* fw, u32 t, int ks) {
      const LAS char* st = (const LAS char*)shm + (t & (NSTAGE - 1)) * STAGE_BYTES;
#pragma unroll
      for (int i = 0; i < 4; ++i) {
        const bf16x4 lo = __builtin_amdgcn_ds_read_tr16_b64_v4bf16((LAS bf16x4*)(st + aw[i] + ks * 8192));
        const bf16x4 hi = __builtin_amdgcn_ds_read_tr16_b64_v4bf16((LAS bf16x4*)(st + aw[i] + ks * 8192 + 1024));
        fw[i] = __builtin_shufflevector(lo, hi, 0, 1, 2, 3, 4, 5, 6, 7);
        fx[i] = *(const LAS bf16x8*)(st + ax[ks] + i * 2048);
      }
    };
    auto mm = [&](const bf16x8* fx, const bf16x8* fw) {
#pragma unroll
      for (int mt = 0; mt < 4; ++mt)
#pragma unroll
        for (int nt = 0; nt < 4; ++nt)
          acc[nt][mt] = __builtin_amdgcn_mfma_f32_16x16x32_bf16(fw[nt], fx[mt], acc[nt][mt], 0, 0, 0);
    };
#define INTERLEAVE() do { _Pragma("unroll") for (int i_ = 0; i_ < 12; ++i_) { \
      __builtin_amdgcn_sched_group_barrier(0x8, 1, 0); __builtin_amdgcn_sched_group_barrier(0x100, 1, 0); } \
      __builtin_amdgcn_sched_group_barrier(0x8, 4, 0); } while (0)
    WG_BARRIER();
    rd(xa, wa, 0, 0);
    __builtin_amdgcn_sched_barrier(0);
    for (u32 t = 0; t < 31; ++t) {
      rd(xb, wb, t, 1);
      mm(xa, wa);
      INTERLEAVE();
      __builtin_amdgcn_sched_barrier(0);
      WG_BARRIER();
      __builtin_amdgcn_sched_barrier(0);
      rd(xa, wa, t + 1, 0);
      mm(xb, wb);
      INTERLEAVE();
      __builtin_amdgcn_sched_barrier(0);
    }
    rd(xb, wb, 31, 1);
    mm(xa, wa);
    INTERLEAVE();
    __builtin_amdgcn_sched_barrier(0);
    mm(xb, wb);
    if (wv == 0) TSTAMP(5);
    WG_BARRIER();
    const bool bad = lds_get(lctr + C_BAD) != 0;
    const float qnan = __builtin_nanf("");
    if (kind == 0) {
      char* db = ws + OFF_PO + (size_t)tix * 32768 + wv * 8192;
#pragma unroll
      for (int mt = 0; mt < 4; ++mt)
#pragma unroll
        for (int np = 0; np < 2; ++np) {
          f32x4 u = acc[2 * np][mt], v = acc[2 * np + 1][mt];
          if (bad) u[0] = qnan;
          const u32x4 o = {pack2(u[0], u[1]), pack2(u[2], u[3]), pack2(v[0], v[1]), pack2(v[2], v[3])};
          st16_sys(vl, o, db + (mt * 2 + np) * 1024);
        }
    } else {
      const LAS u32x4* pr = (const LAS u32x4*)(shm + (32 & (NSTAGE - 1)) * STAGE_BYTES + wv * 8192) + lane;
      char* ob = a.out[0] + (size_t)(row0 + wm * 64) * 8192 + (col0 + wn * 64) * 2;
      const u32 vo = (lane & 15) * 8192 + (g & 1) * 32 + (g >> 1) * 16;
#pragma unroll
      for (int mt = 0; mt < 4; ++mt)
#pragma unroll
        for (int np = 0; np < 2; ++np) {
          const u32x4 pv = pr[(mt * 2 + np) * 64];
          f32x4 u = acc[2 * np][mt], v = acc[2 * np + 1][mt];
          u[0] += __uint_as_float(pv.x << 16); u[1] += __uint_as_float(pv.x & 0xffff0000u);
          u[2] += __uint_as_float(pv.y << 16); u[3] += __uint_as_float(pv.y & 0xffff0000u);
          v[0] += __uint_as_float(pv.z << 16); v[1] += __uint_as_float(pv.z & 0xffff0000u);
          v[2] += __uint_as_float(pv.w << 16); v[3] += __uint_as_float(pv.w & 0xffff0000u);
          if (bad) u[0] = qnan;
          const u32 a0 = pack2(silu(u[0]), silu(u[1])), a1 = pack2(silu(u[2]), silu(u[3]));
          const u32 b0 = pack2(silu(v[0]), silu(v[1])), b1 = pack2(silu(v[2]), silu(v[3]));
          const auto s0 = __builtin_amdgcn_permlane16_swap(a0, b0, false, false);
          const auto s1 = __builtin_amdgcn_permlane16_swap(a1, b1, false, false);
          const u32x4 o = {s0[0], s1[0], s0[1], s1[1]};
          st16_sys(vo, o, ob + (size_t)mt * 16 * 8192 + np * 64);
        }
    }
    WAIT_VM0();
    WG_BARRIER();
  }

  if (xl == 0 || ts < 4 || ts >= 8) break;
  }
}

static void build_tables(Args& a) {
  int pos[32];
  for (int i = 0; i < 32; ++i) pos[PIECES[i]] = i;
  for (int xl = 0; xl < 8; ++xl) {
    const int kind = xl < 4, rb = (xl >> 1) & 1, cb = xl & 1;
    int rdy[32], key[32], idx[32];
    for (int t = 0; t < 32; ++t) {
      const int pa = pos[kind * 8 + rb * 4 + (t & 3)], pb = pos[16 + cb * 8 + (t >> 2)];
      rdy[t] = pa > pb ? pa : pb;
      key[t] = rdy[t] * 64 + (pa < pb ? pa : pb);
      idx[t] = t;
    }
    for (int i = 1; i < 32; ++i)
      for (int j = i; j > 0 && key[idx[j]] < key[idx[j - 1]]; --j) { int s = idx[j]; idx[j] = idx[j - 1]; idx[j - 1] = s; }
    for (int slot = 0; slot < 32; ++slot) {
      const int t = idx[xl == 0 ? slot : (slot < 4 ? 28 + slot : slot - 4)];
      a.tile[xl * 32 + slot] = (t & 3) | ((t >> 2) << 2) | (rdy[t] << 5);
    }
  }
}

extern "C" void kernel_launch(int rank, int nranks, ncclComm_t comm, hipStream_t stream,
                              void* const* d_in, const int* in_sizes, int n_in,
                              void* d_out, int out_size, const int* in_dtypes, int out_dtype,
                              void* const* d_out_all, void* const* d_ws_all, size_t ws_bytes) {
  if (nranks != 8 || (unsigned)rank >= 8u || n_in < 2 || ws_bytes < WS_NEED || out_size != 4096 * 4096 ||
      in_sizes[0] != 4096 * 512 || in_sizes[1] != 512 * 4096) {
    fprintf(stderr, "kernel_launch: unexpected problem shape\n");
    return;
  }
  Args a;
  a.x = (const char*)d_in[0];
  a.w = (const char*)d_in[1];
  for (int l = 0; l < 8; ++l) { a.out[l] = (char*)d_out_all[rank ^ l]; a.ws[l] = (char*)d_ws_all[rank ^ l]; }
  a.rank = (u32)rank;
  static const unsigned char near_xcd[8][8] = {{0, 4, 2, 3, 6, 5, 1, 7}, {4, 0, 7, 5, 2, 3, 6, 1}, {2, 7, 0, 1, 4, 5, 3, 6}, {3, 5, 1, 0, 7, 4, 2, 6},
                                               {5, 2, 4, 7, 0, 1, 6, 3}, {6, 3, 5, 4, 1, 0, 7, 2}, {1, 6, 3, 2, 5, 7, 0, 4}, {7, 1, 6, 5, 3, 2, 4, 0}};
  a.xmap = 0;
  for (int p = 0; p < 8; ++p) a.xmap |= (u32)(rank ^ p) << (4 * near_xcd[rank][p]);
  build_tables(a);
  fused_gemm_ar_silu<<<256, 512, 0, stream>>>(a);
  hipError_t e = hipGetLastError();
  if (e != hipSuccess) fprintf(stderr, "launch failed: %s\n", hipGetErrorString(e));
}
```

```cpp
#include <hip/hip_runtime.h>
#include <rccl/rccl.h>
#include <cstdio>
#include <cstdint>

typedef unsigned int u32;
typedef unsigned long long u64;
typedef __attribute__((ext_vector_type(4))) float f32x4;
typedef __attribute__((ext_vector_type(4))) unsigned int u32x4;
typedef __attribute__((ext_vector_type(2))) unsigned int u32x2;
typedef __attribute__((ext_vector_type(4))) __bf16 bf16x4;
typedef __attribute__((ext_vector_type(8))) __bf16 bf16x8;
typedef __attribute__((ext_vector_type(2))) __bf16 bf16x2;
#define LAS __attribute__((address_space(3)))
#define GAS __attribute__((address_space(1)))
#define RLX __ATOMIC_RELAXED
#define SYS __HIP_MEMORY_SCOPE_SYSTEM
#define AGT __HIP_MEMORY_SCOPE_AGENT
#define WGP __HIP_MEMORY_SCOPE_WORKGROUP

constexpr size_t OFF_CTR = 0;
constexpr size_t OFF_TMO = 4096;
constexpr size_t OFF_TICK = 5120;
constexpr size_t OFF_POOL = 5376;
constexpr size_t OFF_HOLD = 6144;
constexpr size_t OFF_INFLAG = 8192;
constexpr size_t OFF_PFLAG = 16384;
constexpr size_t OFF_QUEUE = 32768;
constexpr size_t OFF_SFLAG = 45056;
constexpr size_t OFF_DBG = 65536;
constexpr size_t OFF_XS = 1u << 20;
constexpr size_t OFF_WS = 9u << 20;
constexpr size_t OFF_PS = 17u << 20;
constexpr size_t OFF_PO = 21u << 20;
constexpr size_t OFF_SX = 25u << 20;
constexpr size_t OFF_SW = 29u << 20;
constexpr size_t WS_NEED = 33u << 20;

constexpr u64 DEADLINE = 100000000ull;
constexpr int NSTAGE = 4;
constexpr int STAGE_BYTES = 32768;
constexpr int LDS_CTL = NSTAGE * STAGE_BYTES;
constexpr int LDS_BYTES = LDS_CTL + 2048;

#define PIECE_ORDER 0, 16, 1, 17, 2, 18, 3, 19, 8, 20, 21, 9, 22, 23, 10, 24, 25, 11, 26, 27, 4, 28, 29, 30, 31, 5, 12, 6, 13, 7, 14, 15
constexpr u32 PIECES[32] = {PIECE_ORDER};
constexpr u64 pack_pieces(int first) {
  u64 v = 0;
  for (int i = 0; i < 8; ++i) v |= (u64)PIECES[first + i] << (8 * i);
  return v;
}
__device__ __forceinline__ u32 piece_at(u32 i) {
  constexpr u64 p0 = pack_pieces(0), p1 = pack_pieces(8), p2 = pack_pieces(16), p3 = pack_pieces(24);
  const u64 v = i < 16 ? (i < 8 ? p0 : p1) : (i < 24 ? p2 : p3);
  return (u32)(v >> (8 * (i & 7))) & 31u;
}

constexpr u64 pack_pos(int first) {
  u64 v = 0;
  for (int i = 0; i < 32; ++i) if ((int)PIECES[i] >= first && (int)PIECES[i] < first + 8) v |= (u64)i << (8 * (PIECES[i] - first));
  return v;
}
__device__ __forceinline__ u32 pos_of(u32 c) {
  constexpr u64 p0 = pack_pos(0), p1 = pack_pos(8), p2 = pack_pos(16), p3 = pack_pos(24);
  const u64 v = c < 16 ? (c < 8 ? p0 : p1) : (c < 24 ? p2 : p3);
  return (u32)(v >> (8 * (c & 7))) & 31u;
}
#define NP0 2
#define RP 108
constexpr u32 NPOOL = 2 * RP;
struct StageOrder { u64 v[4]; };
constexpr StageOrder make_stage_order() {
  StageOrder o = {{0, 0, 0, 0}};
  bool seen[24] = {};
  int n = 0;
  for (int k = NP0; k < 32 + NP0; ++k) {
    const int c = (int)PIECES[k & 31], cl = c < 16 ? (c & 7) : c - 8;
    if (seen[cl]) continue;
    seen[cl] = true;
    for (int e = 0; e < (c < 16 ? 2 : 1); ++e, ++n) o.v[n >> 3] |= (u64)(c < 16 ? cl * 2 + e : c) << (8 * (n & 7));
  }
  return o;
}
__device__ __forceinline__ u32 stage_class(u32 i) {
  constexpr StageOrder o = make_stage_order();
  constexpr u64 p0 = o.v[0], p1 = o.v[1], p2 = o.v[2], p3 = o.v[3];
  const u64 v = i < 16 ? (i < 8 ? p0 : p1) : (i < 24 ? p2 : p3);
  return (u32)(v >> (8 * (i & 7))) & 31u;
}

struct Args {
  const char* x;
  const char* w;
  u32 rank;
  u32 xmap;
  char* ws[8];
  char* out[8];
  u32 tile[256];
};

#define STAMP(i) do {} while (0)
#define TSTAMP(i) do {} while (0)

__device__ __forceinline__ u32 grow0(u32 r, u32 lr) {
  return ((r >> 1) & 1) * 2048 + ((lr >> 3) ^ (r >> 2) ^ 1) * 1024 + (lr & 7) * 128;
}
__device__ __forceinline__ u32 gcol0(u32 r, u32 lc) { return (r & 1) * 2048 + lc * 128; }
__device__ __forceinline__ u32 tile_off(u32 r, u32 id) {
  return grow0(r, 8 + ((id >> 4) & 7)) * 8192 + gcol0(r, id & 15) * 2;
}

__device__ __forceinline__ void st16_sys(u32 voff, u32x4 v, char* sbase) {
  asm volatile("global_store_dwordx4 %0, %1, %2 sc0 sc1\n\ts_nop 1" :: "v"(voff), "v"(v), "s"(sbase) : "memory");
}
__device__ __forceinline__ void st16_agent(u32 voff, u32x4 v, char* sbase) {
  asm volatile("global_store_dwordx4 %0, %1, %2 sc1\n\ts_nop 1" :: "v"(voff), "v"(v), "s"(sbase) : "memory");
}
__device__ __forceinline__ bool expired(u64 t0) { return __builtin_amdgcn_s_memrealtime() - t0 > DEADLINE; }
#define WAIT_VM0() asm volatile("s_waitcnt vmcnt(0)" ::: "memory")
#define WG_BARRIER() asm volatile("s_waitcnt lgkmcnt(0)\n\ts_barrier" ::: "memory")

#define WIN 8
#define WINC1 1
#define WINC2 2
#define WINC3 3
#define NST 4
constexpr u32 RING = 24, RING0 = 16384;
#define SCHED_AHEAD 3
#define LOADER_PRIO 2
#define POLL_SLEEP 2
#define START_DELAY 100
#define START_SLOPE 100
#define STAGE_DELAY 80
#define POOL_AHEAD 6
constexpr u32 C_CNT = 0;
constexpr u32 C_AVAIL = 128;
constexpr u32 C_LISTED = 136, C_TOTAL = 137, C_TICKET = 138, C_BAD = 139, C_MASK = 140, C_LANDED = 141, C_TAKEN = 142, C_HELD = 143;
constexpr u32 C_FREE = 160;
constexpr u32 C_LIST = 192, C_LIST2 = 320;

__device__ __forceinline__ void glds4v(u32 v0, u32 v1, u32 v2, u32 v3, const char* sbase, u32 lds) {
  asm volatile(
      "s_mov_b32 m0, %5\n\ts_nop 0\n\t"
      "global_load_lds_dwordx4 %0, %4\n\t"
      "global_load_lds_dwordx4 %1, %4 offset:1024\n\t"
      "global_load_lds_dwordx4 %2, %4 offset:2048\n\t"
      "global_load_lds_dwordx4 %3, %4 offset:3072"
      :: "v"(v0), "v"(v1), "v"(v2), "v"(v3), "s"(sbase), "s"(lds) : "memory");
}
__device__ __forceinline__ void pull4l(u32 v0, u32 v1, u32 v2, u32 v3, const char* sbase, u32 lds) {
  asm volatile(
      "s_mov_b32 m0, %5\n\ts_waitcnt vmcnt(%6)\n\t"
      "global_load_lds_dwordx4 %0, %4 sc0 sc1\n\ts_waitcnt vmcnt(%7)\n\t"
      "global_load_lds_dwordx4 %1, %4 offset:1024 sc0 sc1\n\ts_waitcnt vmcnt(%8)\n\t"
      "global_load_lds_dwordx4 %2, %4 offset:2048 sc0 sc1\n\ts_waitcnt vmcnt(%9)\n\t"
      "global_load_lds_dwordx4 %3, %4 offset:3072 sc0 sc1"
      :: "v"(v0), "v"(v1), "v"(v2), "v"(v3), "s"(sbase), "s"(lds), "n"(WIN), "n"(WIN + WINC1), "n"(WIN + WINC2), "n"(WIN + WINC3) : "memory");
}
__device__ __forceinline__ u32 src_x(u32 s, u32 lane) {
  return (8 * s + (lane >> 3)) * 1024 + (((lane & 7) ^ (4 * (s & 1) + (lane >> 4))) * 16) - s * 1024;
}
__device__ __forceinline__ u32 src_w(u32 s, u32 lane) {
  return (4 * s + (lane >> 4)) * 8192 + (((lane & 15) ^ (((lane >> 4) | ((s >> 1) << 2)) << 1)) * 16) - s * 1024;
}
__device__ __forceinline__ u32 lds_get(LAS u32* p) {
  asm volatile("" ::: "memory");
  const u32 v = __builtin_amdgcn_readfirstlane(__hip_atomic_load(p, RLX, WGP));
  asm volatile("" ::: "memory");
  return v;
}
__device__ __forceinline__ void lds_set(LAS u32* p, u32 v) {
  asm volatile("" ::: "memory");
  __hip_atomic_store(p, v, RLX, WGP);
  asm volatile("" ::: "memory");
}
__device__ __forceinline__ u32 lds_add(LAS u32* p, u32 lane) {
  u32 t = 0;
  if (lane == 0) t = __hip_atomic_fetch_add(p, 1u, RLX, WGP);
  return __builtin_amdgcn_readfirstlane(t);
}

__device__ __forceinline__ void ld4(u32x4& d0, u32x4& d1, u32x4& d2, u32x4& d3, u32 v0, u32 v1, u32 v2, u32 v3, const char* sb) {
  asm volatile("global_load_dwordx4 %0, %4, %8\n\tglobal_load_dwordx4 %1, %5, %8 offset:1024\n\t"
               "global_load_dwordx4 %2, %6, %8 offset:2048\n\tglobal_load_dwordx4 %3, %7, %8 offset:3072\n\ts_waitcnt vmcnt(0)"
               : "=&v"(d0), "=&v"(d1), "=&v"(d2), "=&v"(d3) : "v"(v0), "v"(v1), "v"(v2), "v"(v3), "s"(sb) : "memory");
}
__device__ __forceinline__ u32 kb_first(u32 xl) { return xl < 4 ? 4 + 4 * xl : 4 * xl; }

__device__ __forceinline__ void push_leading(const Args& a, u32 p, u32 xl, u32 i, u32 j, u32 kb, u32 lane) {
  const u32 pc = piece_at(i), k0 = (p >> 2) * 256 + kb * 64;
  u32x4 d0, d1, d2, d3;
  if (pc >> 4) ld4(d0, d1, d2, d3, src_w(0, lane), src_w(1, lane), src_w(2, lane), src_w(3, lane),
                   a.w + ((k0 + j * 16) * 8192 + gcol0(p, pc & 15) * 2));
  else ld4(d0, d1, d2, d3, src_x(0, lane), src_x(1, lane), src_x(2, lane), src_x(3, lane),
           a.x + ((grow0(p, pc) + j * 32) * 1024 + k0 * 2));
  char* db = a.ws[xl] + (OFF_XS + (pc * 32 + kb_first(xl) + kb) * 16384 + j * 4096);
  const u32 vd = lane * 16;
  st16_sys(vd, d0, db); st16_sys(vd, d1, db + 1024); st16_sys(vd, d2, db + 2048); st16_sys(vd, d3, db + 3072);
  WAIT_VM0();
}

__device__ __forceinline__ void stage_unit(const Args& a, u32 rank, u32 unit, u32 wv, u32 lane, u32& calls, char*& pws) {
  const u32 j = unit & 3, sub = (unit >> 2) & 1, cls = stage_class(unit >> 3);
  const bool isw = cls >= 16;
  const u32 ch = isw ? sub * 16 + (cls & 15) : (cls & 1) * 16 + sub * 8 + (cls >> 1);
  u32x4 d0, d1, d2, d3;
  if (isw) ld4(d0, d1, d2, d3, src_w(0, lane), src_w(1, lane), src_w(2, lane), src_w(3, lane), a.w + ((wv * 64 + j * 16) * 8192 + ch * 256));
  else ld4(d0, d1, d2, d3, src_x(0, lane), src_x(1, lane), src_x(2, lane), src_x(3, lane), a.x + ((ch * 128 + j * 32) * 1024 + wv * 128));
  char* db = a.ws[0] + ((isw ? OFF_SW : OFF_SX) + (ch * 8 + wv) * 16384 + j * 4096);
  const u32 vd = lane * 16;
  st16_sys(vd, d0, db); st16_sys(vd, d1, db + 1024); st16_sys(vd, d2, db + 2048); st16_sys(vd, d3, db + 3072);
  asm volatile("s_waitcnt vmcnt(0)\n\ts_barrier" : "+v"(calls), "+v"(pws) :: "memory");
  if (wv == 0 && lane < 8 && (lane & 3)) {
    const u32 p = rank ^ lane;
    const u32 pos = pos_of(isw ? cls : (sub ^ (p >> 2) ^ 1) * 8 + (cls >> 1));
    if ((isw ? p & 1 : (p >> 1) & 1) == ch >> 4 && pos >= NP0)
      __hip_atomic_store((GAS u32*)(pws + OFF_SFLAG + (lane * 4 + j) * 128) + pos, calls + 1, RLX, SYS);
  }
}

__device__ __forceinline__ void link_engine(const Args& a, u32 rank, u32 xl, u32 j, u32 wv, u32 lane, u32 seq, u64 t0,
                                            LAS char* shm, u32 lds0) {
  LAS u32* ctl = (LAS u32*)(shm + LDS_CTL);
  const bool pair = xl == 4;
  const u32 f0 = (lane >> 4) * 8192 + (lane & 15) * 16, vl = lane * 16;
  char* const ws = a.ws[0];
  volatile LAS u32x2* list = (volatile LAS u32x2*)shm;
  if (wv == 5) {
    const u32 isrc = OFF_SX + (rank >> 2) * 4 * 16384 + j * 4096;
    const u32 idst = OFF_XS + kb_first(xl) * 16384 + j * 4096;
    u32 gi = 0, ni = pair ? 32u : NP0, nf = pair ? RP : 0u;
    u64 left0 = pair ? ~0ull : 0ull, left1 = left0;
    bool pool = true;
#pragma unroll 1
    while (pool || nf < 128 || (left0 | left1)) {
      asm volatile("" ::: "memory");
      const u32x4 av = *(volatile LAS u32x4*)(ctl + C_AVAIL), pm = *(volatile LAS u32x4*)(ctl + C_AVAIL + 4);
      const u32 room = lds_get(ctl + C_TICKET) + POOL_AHEAD - gi;
      const u64 a0 = left0 & (((u64)__builtin_amdgcn_readfirstlane(pm.y) << 32) | (u32)__builtin_amdgcn_readfirstlane(pm.x));
      const u64 a1 = left1 & (((u64)__builtin_amdgcn_readfirstlane(pm.w) << 32) | (u32)__builtin_amdgcn_readfirstlane(pm.z));
      const bool near = room - 1 < POOL_AHEAD && room > POOL_AHEAD - SCHED_AHEAD;
      u32 src = 0, dst = 0, n = 0, step = 16 * 8192;
      if (a0 | a1) {
        if (near) {
          const u32 id = a0 ? __builtin_ctzll(a0) : 64 + __builtin_ctzll(a1);
          if (a0) left0 &= ~(1ull << id); else left1 &= ~(1ull << (id - 64));
          src = OFF_PO + id * 32768 + j * 8192; dst = OFF_PS + id * 32768 + j * 8192 + id * 2 + 1; n = 2; step = 4096;
        }
      } else if (ni < 32) {
        if (near && ni < __builtin_amdgcn_readfirstlane(av.x)) {
          const u32 pc = piece_at(ni);
          src = isrc + (pc >> 4 ? 4096 + gcol0(rank, pc & 15) : grow0(rank, pc)) * 1024;
          dst = idst + pc * 32 * 16384 + ni * 2 + 1; n = 4; step = 16384;
          ++ni;
        }
      } else if (nf < __builtin_amdgcn_readfirstlane(av.y)) {
        if (near) {
          src = lds_get(ctl + C_LIST + nf) + j * 32 * 8192; dst = src + (nf < RP ? 2 : 0); n = 2;
          ++nf;
        }
      } else if (pool && __builtin_amdgcn_readfirstlane(av.z) && room - 1 < POOL_AHEAD) {
        u32 t = 0;
        if (lane == 0) t = __hip_atomic_fetch_add((GAS u32*)(ws + OFF_POOL), 1u, RLX, AGT);
        t = __builtin_amdgcn_readfirstlane(t) - (seq - 1) * (NPOOL + 28);
        if (t < NPOOL) { src = dst = lds_get(ctl + (pair ? C_LIST : C_LIST2) + (t >> 1)) + (t & 1) * 64 * 8192; n = 4; }
        else pool = false;
      }
      if (n) {
        if (lane < n) list[gi + lane] = (u32x2){src + lane * step, dst + lane * step};
        gi += n;
        lds_set(ctl + C_LISTED, gi);
      } else {
        __builtin_amdgcn_s_sleep(1);
      }
    }
    lds_set(ctl + C_TOTAL, gi + 1);
  } else if (wv == 0) {
    constexpr u32 BACK = (WIN + WINC3) / 4;
    const char* const pws = a.ws[xl];
    const char* const fsrc = a.out[xl];
    u32 slot = 0, pub = 0, gi = 0;
    __builtin_amdgcn_s_setprio(3);
#pragma unroll 1
    for (;; ++gi) {
      u32 src, lin;
      bool end = false;
      for (u32 tries = 0;; ++tries) {
        asm volatile("" ::: "memory");
        const u32x2 lt = *(volatile LAS u32x2*)(ctl + C_LISTED);
        const u32 fr = __hip_atomic_load(ctl + C_FREE + slot, RLX, WGP);
        asm volatile("" ::: "memory");
        const u32x2 d = list[gi];
        src = __builtin_amdgcn_readfirstlane(d.x); lin = __builtin_amdgcn_readfirstlane(d.y) & 1;
        if (__builtin_amdgcn_readfirstlane(lt.x) > gi && __builtin_amdgcn_readfirstlane(fr) + RING > gi) break;
        if (__builtin_amdgcn_readfirstlane(lt.y) == gi + 1) { end = true; break; }
        if (tries == 3) {
          WAIT_VM0(); lds_set(ctl + C_LANDED, gi); pub = gi;
        }
        __builtin_amdgcn_s_sleep(1);
      }
      if (end) break;
      if (lin) pull4l(vl, vl, vl, vl, pws + src, lds0 + RING0 + slot * 4096);
      else pull4l(f0, f0 + 31744, f0 + 63488, f0 + 95232, fsrc + src, lds0 + RING0 + slot * 4096);
      if (gi > pub + BACK) { pub = gi - BACK; lds_set(ctl + C_LANDED, pub); }
      lds_set(ctl + C_TICKET, gi + 1);
      slot = slot + 1 == RING ? 0 : slot + 1;
    }
    WAIT_VM0();
    lds_set(ctl + C_LANDED, gi);
  } else if (wv <= NST) {
    char* const fdst = a.out[0];
    u32 landed = 0, prev = 0;
    auto done = [&]() {
      if (prev & 1) {
        const u32 c = (prev >> 1) & 127;
        if (lds_add(ctl + C_CNT + c, lane) == 1 && pair && lane == 0)
          __hip_atomic_store((GAS u32*)(ws + OFF_PFLAG + c * 64) + j, seq, RLX, AGT);
      } else if (prev & 2) {
        if (lds_add(ctl + C_HELD, lane) == 2 * RP - 1 && lane == 0)
          __hip_atomic_store((GAS u32*)(a.ws[xl ^ 4] + OFF_HOLD + (xl ^ 4) * 128) + j, seq, RLX, SYS);
      }
      prev = 0;
    };
#pragma unroll 1
    for (;;) {
      const u32 gi = lds_add(ctl + C_TAKEN, lane);
      if (landed <= gi) {
        landed = lds_get(ctl + C_LANDED);
        if (landed <= gi) {
          WAIT_VM0();
          done();
          bool end = false;
          do {
            end = lds_get(ctl + C_TOTAL) - 1 <= gi;
            if (end) break;
            __builtin_amdgcn_s_sleep(1);
            landed = lds_get(ctl + C_LANDED);
          } while (landed <= gi);
          if (end) break;
        }
      }
      const u32 slot = gi % RING;
      const u32 desc = __builtin_amdgcn_readfirstlane(list[gi].y);
      const LAS u32x4* rs = (const LAS u32x4*)(shm + RING0 + slot * 4096) + lane;
      u32x4 d0 = rs[0];
      const u32x4 d1 = rs[64], d2 = rs[128], d3 = rs[192];
      lds_set(ctl + C_FREE + slot, gi + 1);
      if (lds_get(ctl + C_BAD)) d0.x = 0x7fc07fc0u;
      WAIT_VM0();
      if (desc & 1) {
        char* db = ws + (desc & ~4095u);
        st16_agent(vl, d0, db); st16_agent(vl, d1, db + 1024); st16_agent(vl, d2, db + 2048); st16_agent(vl, d3, db + 3072);
      } else {
        char* db = fdst + (desc & ~255u);
        st16_sys(f0, d0, db); st16_sys(f0, d1, db + 32768); st16_sys(f0, d2, db + 65536); st16_sys(f0, d3, db + 98304);
      }
      done();
      prev = desc;
    }
  } else if (wv == 6) {
    const u32 tag = seq & 0xffffff;
    const GAS u32* qf = (const GAS u32*)(ws + OFF_QUEUE + xl * 1024) + lane;
    const GAS u32* qs = pair ? qf + 128 : lane < 32 ? (const GAS u32*)(ws + OFF_SFLAG + (xl * 4 + j) * 128) + lane
                                                    : (const GAS u32*)(ws + OFF_HOLD + xl * 128) + (lane & 3);
    u32 nf = 0, open = 0;
    for (;;) {
      u32 e0 = __hip_atomic_load(qf, RLX, SYS), e1 = __hip_atomic_load(qf + 64, RLX, SYS), e2 = __hip_atomic_load(qs, RLX, SYS);
      u32 e3 = seq;
      if (pair) e3 = __hip_atomic_load(qs + 64, RLX, SYS);
      const bool late = expired(t0);
      if (late) { lds_set(ctl + C_BAD, 1); e0 = (seq << 8) | lane; e1 = (seq << 8) | (64 + lane); e2 = e3 = seq; }
      const u64 n0 = __ballot((e0 >> 8) != tag), n1 = __ballot((e1 >> 8) != tag);
      const u64 m2 = __ballot(e2 >= seq || (!pair && lane < NP0)), m3 = __ballot(e3 >= seq);
      const u32 cf = n0 ? (u32)__builtin_ctzll(n0) : (n1 ? 64u + (u32)__builtin_ctzll(n1) : 128u);
      const u32 staged = (u32)m2 == ~0u ? 32u : (u32)__builtin_ctz(~(u32)m2);
      if (cf != nf) {
        if (lane >= nf) ctl[C_LIST + lane] = tile_off(rank ^ xl, e0);
        if (64 + lane >= nf) ctl[C_LIST + 64 + lane] = tile_off(rank ^ xl, e1);
        nf = cf;
      }
      if (pair) {
        open = cf >= RP;
        *(volatile LAS u32x4*)(ctl + C_AVAIL + 4) = (u32x4){(u32)m2, (u32)(m2 >> 32), (u32)m3, (u32)(m3 >> 32)};
      } else if (!open && (u32)(m2 >> 32) == ~0u) {
        const GAS u32* qp = (const GAS u32*)(ws + OFF_QUEUE + 4 * 1024) + lane;
        const u32 g0 = __hip_atomic_load(qp, RLX, SYS), g1 = __hip_atomic_load(qp + 64, RLX, SYS);
        ctl[C_LIST2 + lane] = tile_off(rank ^ 4, g0);
        ctl[C_LIST2 + 64 + lane] = tile_off(rank ^ 4, g1);
        open = late || !__ballot(((g0 >> 8) != tag && lane < RP) || ((g1 >> 8) != tag && 64 + lane < RP));
      }
      asm volatile("" ::: "memory");
      *(volatile LAS u32x4*)(ctl + C_AVAIL) = (u32x4){staged, cf, open, 0u};
      if (cf == 128 && open && (!pair || (m2 & m3) == ~0ull)) break;
      if (!pair && staged == 32) __builtin_amdgcn_s_sleep(8);
    }
  } else if (wv == 7 && !pair) {
    GAS u32* const flag = (GAS u32*)(ws + OFF_INFLAG + xl * 16 + j * 4);
    const GAS u32* lf = (const GAS u32*)(ws + OFF_SFLAG + (xl * 4 + j) * 128) + (lane & 31);
    while (__ballot(__hip_atomic_load(lf, RLX, SYS) < seq && lane < NP0)) {
      if (expired(t0)) { lds_set(ctl + C_BAD, 1); break; }
      __builtin_amdgcn_s_sleep(1);
    }
    if (lane < NP0) __hip_atomic_store(flag + lane * 64, seq, RLX, AGT);
#pragma unroll 1
    for (u32 pi = NP0; pi < 32; ++pi) {
      while (lds_get(ctl + C_CNT + pi) < 4) __builtin_amdgcn_s_sleep(1);
      if (lane == 0) __hip_atomic_store(flag + pi * 64, seq, RLX, AGT);
    }
  }
}

__device__ __forceinline__ u32 pack2(float lo, float hi) {
  bf16x2 v = {(__bf16)lo, (__bf16)hi};
  return __builtin_bit_cast(u32, v);
}
__device__ __forceinline__ float silu(float y) {
  const float c = __builtin_fminf(__builtin_fmaxf(y, -60.f), 60.f);
  return y * __builtin_amdgcn_rcpf(1.f + __builtin_amdgcn_exp2f(c * -1.44269504088896341f));
}

__global__ void __launch_bounds__(512, 2) fused_gemm_ar_silu(const Args a) {
  __shared__ __attribute__((aligned(1024))) char shm[LDS_BYTES];
  const u32 tid = threadIdx.x, lane = tid & 63;
  const u32 wv = __builtin_amdgcn_readfirstlane(tid >> 6);
  const u32 b = blockIdx.x, slot = b >> 3;
  const u32 xl = (a.xmap >> (4 * (b & 7))) & 7;
  LAS u32* lctr = (LAS u32*)(shm + LDS_CTL);
  const u32 lds0 = (u32)(size_t)(LAS char*)shm;
  const bool engine = xl != 0 && slot < 4;
  const u64 t0 = __builtin_amdgcn_s_memrealtime();
  const u32 rank = a.rank & 7;
  char* const ws = a.ws[0];
  GAS u32* ctr = (GAS u32*)(ws + OFF_CTR) + b;
  u32 calls = __hip_atomic_load(ctr, RLX, AGT);
  char* pws = a.ws[lane & 7];
  if (wv == 7) { lctr[lane] = 0; lctr[64 + lane] = 0; lctr[128 + lane] = 0; }
  const bool is_lead = (xl & 3) && slot >= 4 && slot < 4 + 4 * NP0;
  if (is_lead && wv < 4) push_leading(a, rank ^ xl, xl, (slot - 4) >> 2, slot & 3, wv, lane);
  if (slot >= 12 && slot < 28) {
    const u32 u = b - 96;
    if (u >= 64) while (__builtin_amdgcn_s_memrealtime() - t0 < STAGE_DELAY) __builtin_amdgcn_s_sleep(2);
    stage_unit(a, rank, u, wv, lane, calls, pws);
    if (wv == 0) STAMP(11);
    stage_unit(a, rank, 128 + u, wv, lane, calls, pws);
    if (wv == 0) STAMP(10);
  }
  asm volatile("" : "+v"(calls));
  const u32 seq = __builtin_amdgcn_readfirstlane(calls) + 1;
  WG_BARRIER();
  if (is_lead && tid == 448)
    __hip_atomic_store((GAS u32*)(a.ws[xl] + OFF_SFLAG + (xl * 4 + (slot & 3)) * 128) + ((slot - 4) >> 2), seq, RLX, SYS);
  if (is_lead && wv == 7) STAMP(11);
  if (tid == 0) __hip_atomic_store(ctr, seq, RLX, AGT);
  if (engine) {
    link_engine(a, rank, xl, slot, wv, lane, seq, t0, (LAS char*)shm, lds0);
    return;
  }

#pragma unroll 1
  for (u32 ts = slot;; ts -= 4) {
  const u32 te = a.tile[xl * 32 + ts];
  const u32 lr = te & 15, lc = (te >> 4) & 15, rpos = (te >> 8) & 31, tix = (te >> 13) & 127;
  const u32 kind = lr >> 3;
  const u32 row0 = grow0(rank, lr), col0 = gcol0(rank, lc);
  const u32 vl = lane * 16;

  if (wv >= 4) {
    const u32 w4 = wv - 4;
    const char* xo = a.x + (size_t)(row0 + w4 * 32) * 1024;
    const char* wo = a.w + ((size_t)w4 * 16 * 8192 + col0 * 2);
    const char* xs = ws + (OFF_XS + (size_t)lr * 32 * 16384 + w4 * 4096);
    const char* wsg = ws + (OFF_WS + (size_t)lc * 32 * 16384 + w4 * 4096);
    const u32 lq = lds0 + w4 * 4096;
    const GAS u32x4* inflag = (const GAS u32x4*)(ws + OFF_INFLAG + rpos * 256 + (lane & 7) * 16);
    const GAS u32x4* lastflag = (lane & 7) ? inflag : (const GAS u32x4*)(ws + OFF_PFLAG + tix * 64);
    { const u32 d = START_DELAY + rpos * START_SLOPE; while (__builtin_amdgcn_s_memrealtime() - t0 < d) __builtin_amdgcn_s_sleep(4); }
    if (wv == 7 && ts == slot) {
      __builtin_amdgcn_fence(__ATOMIC_ACQUIRE, "");
      WAIT_VM0();
    }
    __builtin_amdgcn_s_setprio(LOADER_PRIO);
    {
      const u32 x0 = src_x(0, lane), x1 = src_x(1, lane), x2 = src_x(2, lane), x3 = src_x(3, lane);
      const u32 y0 = src_w(0, lane), y1 = src_w(1, lane), y2 = src_w(2, lane), y3 = src_w(3, lane);
#pragma unroll
      for (u32 kb = 0; kb < 8; ++kb) {
        if (kb >= 3) asm volatile("s_waitcnt vmcnt(16)\n\ts_barrier" ::: "memory");
        glds4v(x0, x1, x2, x3, xo + kb * 128, lq + (kb & 3) * STAGE_BYTES);
        glds4v(y0, y1, y2, y3, wo + (size_t)kb * 64 * 8192, lq + (kb & 3) * STAGE_BYTES + 16384);
      }
    }
    const u32 nst = 32 + kind;
    u32 avail = 8, issued = 8, ord = 0, seen = 0;
    auto refresh = [&]() {
      if (w4 == 0) {
        u32x4 f;
        const GAS u32x4* fp = issued >= 20 ? lastflag : inflag;
        asm volatile("global_load_dwordx4 %0, %1, off sc0 sc1\n\ts_waitcnt vmcnt(0)" : "=&v"(f) : "v"(fp) : "memory");
        u32 mask = (u32)__ballot(f.x >= seq && f.y >= seq && f.z >= seq && f.w >= seq);
        if (expired(t0)) { lds_set(lctr + C_BAD, 1); mask = 0xff; }
        ord |= (mask & 1) << 21;
        for (u32 m = mask & 0xee & ~seen; m; m &= m - 1) {
          const u32 l = __builtin_ctz(m);
          ord = (ord | ((l < 4 ? l + 1 : l) << (3 * ((ord >> 18) & 7)))) + (1u << 18);
        }
        seen |= mask;
        lds_set(lctr + C_MASK, ord);
      } else {
        ord = lds_get(lctr + C_MASK);
      }
      const u32 n = (ord >> 18) & 7;
      avail = 8 + 4 * n + (n == 6 ? (ord >> 21) & kind : 0);
    };
#pragma unroll 1
    for (u32 s = 5; s <= 32; ++s) {
      while (issued <= s + 2) {
        if (issued >= avail) {
          if (issued >= nst) break;
          refresh();
          while (issued >= avail && issued == s) { __builtin_amdgcn_s_sleep(POLL_SLEEP); refresh(); }
          if (issued >= avail) break;
        }
        const u32 l = lq + (issued & 3) * STAGE_BYTES;
        const u32 kk = ((ord >> (3 * ((issued - 8) >> 2))) & 7) * 4 + (issued & 3);
        const char* sx = xs + kk * 16384;
        const char* sw = wsg + kk * 16384;
        if (issued == 32) { sx = ws + (OFF_PS + (size_t)tix * 32768 + w4 * 4096); sw = sx + 16384; }
        glds4v(vl, vl, vl, vl, sx, l);
        glds4v(vl, vl, vl, vl, sw, l + 16384);
        ++issued;
      }
      const u32 ahead = issued - s;
      if (ahead >= 3) asm volatile("s_waitcnt vmcnt(16)\n\ts_barrier" ::: "memory");
      else if (ahead == 2) asm volatile("s_waitcnt vmcnt(8)\n\ts_barrier" ::: "memory");
      else asm volatile("s_waitcnt vmcnt(0) lgkmcnt(0)\n\ts_barrier" ::: "memory");
    }
    __builtin_amdgcn_s_setprio(0);
    u32 pos = 0;
    if (kind && wv == 4 && lane == 0) pos = __hip_atomic_fetch_add((GAS u32*)(ws + OFF_TICK), 1u, RLX, AGT);
    pos = __builtin_amdgcn_readfirstlane(pos);
    if (wv == 4) lds_set(lctr + C_MASK, 0);
    WG_BARRIER();
    if (wv == 4) {
      const u32 e = (seq << 8) | ((lr & 7) * 16 + lc);
      if (kind) {
        if (lane >= 1 && lane < 8) __hip_atomic_store((GAS u32*)(pws + OFF_QUEUE + lane * 1024) + (pos & 127), e, RLX, SYS);
      } else if (lane == 0) {
        __hip_atomic_store((GAS u32*)(a.ws[4] + OFF_QUEUE + 4 * 1024 + 512) + tix, seq, RLX, SYS);
      }
      TSTAMP(7);
    }
  } else {
    const u32 wn = wv & 1, wm = wv >> 1;
    const u32 g = lane >> 4, q = (lane >> 2) & 3, pp = lane & 3;
    u32 ax[2];
#pragma unroll
    for (int ks = 0; ks < 2; ++ks) ax[ks] = (wm * 64 + (lane & 15)) * 128 + (((4 * ks + g) ^ ((lane & 15) >> 1)) * 16);
    u32 aw[4];
#pragma unroll
    for (int nt = 0; nt < 4; ++nt)
      aw[nt] = 16384 + (8 * g + q) * 256 + (((2 * (wn * 4 + nt) + (pp >> 1)) ^ ((q | ((g & 1) << 2)) << 1)) * 16) + (pp & 1) * 8;

    f32x4 acc[4][4];
#pragma unroll
    for (int i = 0; i < 4; ++i)
#pragma unroll
      for (int jj = 0; jj < 4; ++jj) acc[i][jj] = (f32x4){0.f, 0.f, 0.f, 0.f};
    bf16x8 xa[4], wa[4], xb[4], wb[4];
    auto rd = [&](bf16x8* fx, bf16x8* fw, u32 t, int ks) {
      const LAS char* st = (const LAS char*)shm + (t & (NSTAGE - 1)) * STAGE_BYTES;
#pragma unroll
      for (int i = 0; i < 4; ++i) {
        const bf16x4 lo = __builtin_amdgcn_ds_read_tr16_b64_v4bf16((LAS bf16x4*)(st + aw[i] + ks * 8192));
        const bf16x4 hi = __builtin_amdgcn_ds_read_tr16_b64_v4bf16((LAS bf16x4*)(st + aw[i] + ks * 8192 + 1024));
        fw[i] = __builtin_shufflevector(lo, hi, 0, 1, 2, 3, 4, 5, 6, 7);
        fx[i] = *(const LAS bf16x8*)(st + ax[ks] + i * 2048);
      }
    };
    auto mm = [&](const bf16x8* fx, const bf16x8* fw) {
#pragma unroll
      for (int mt = 0; mt < 4; ++mt)
#pragma unroll
        for (int nt = 0; nt < 4; ++nt)
          acc[nt][mt] = __builtin_amdgcn_mfma_f32_16x16x32_bf16(fw[nt], fx[mt], acc[nt][mt], 0, 0, 0);
    };
#define INTERLEAVE() do { _Pragma("unroll") for (int i_ = 0; i_ < 12; ++i_) { \
      __builtin_amdgcn_sched_group_barrier(0x8, 1, 0); __builtin_amdgcn_sched_group_barrier(0x100, 1, 0); } \
      __builtin_amdgcn_sched_group_barrier(0x8, 4, 0); } while (0)
    WG_BARRIER();
    rd(xa, wa, 0, 0);
    __builtin_amdgcn_sched_barrier(0);
    for (u32 t = 0; t < 31; ++t) {
      rd(xb, wb, t, 1);
      mm(xa, wa);
      INTERLEAVE();
      __builtin_amdgcn_sched_barrier(0);
      WG_BARRIER();
      __builtin_amdgcn_sched_barrier(0);
      rd(xa, wa, t + 1, 0);
      mm(xb, wb);
      INTERLEAVE();
      __builtin_amdgcn_sched_barrier(0);
    }
    rd(xb, wb, 31, 1);
    mm(xa, wa);
    INTERLEAVE();
    __builtin_amdgcn_sched_barrier(0);
    mm(xb, wb);
    if (wv == 0) TSTAMP(5);
    WG_BARRIER();
    const bool bad = lds_get(lctr + C_BAD) != 0;
    const float qnan = __builtin_nanf("");
    if (kind == 0) {
      char* db = ws + OFF_PO + (size_t)tix * 32768 + wv * 8192;
#pragma unroll
      for (int mt = 0; mt < 4; ++mt)
#pragma unroll
        for (int np = 0; np < 2; ++np) {
          f32x4 u = acc[2 * np][mt], v = acc[2 * np + 1][mt];
          if (bad) u[0] = qnan;
          const u32x4 o = {pack2(u[0], u[1]), pack2(u[2], u[3]), pack2(v[0], v[1]), pack2(v[2], v[3])};
          st16_sys(vl, o, db + (mt * 2 + np) * 1024);
        }
    } else {
      const LAS u32x4* pr = (const LAS u32x4*)(shm + (32 & (NSTAGE - 1)) * STAGE_BYTES + wv * 8192) + lane;
      char* ob = a.out[0] + (size_t)(row0 + wm * 64) * 8192 + (col0 + wn * 64) * 2;
      const u32 vo = (lane & 15) * 8192 + (g & 1) * 32 + (g >> 1) * 16;
#pragma unroll
      for (int mt = 0; mt < 4; ++mt)
#pragma unroll
        for (int np = 0; np < 2; ++np) {
          const u32x4 pv = pr[(mt * 2 + np) * 64];
          f32x4 u = acc[2 * np][mt], v = acc[2 * np + 1][mt];
          u[0] += __uint_as_float(pv.x << 16); u[1] += __uint_as_float(pv.x & 0xffff0000u);
          u[2] += __uint_as_float(pv.y << 16); u[3] += __uint_as_float(pv.y & 0xffff0000u);
          v[0] += __uint_as_float(pv.z << 16); v[1] += __uint_as_float(pv.z & 0xffff0000u);
          v[2] += __uint_as_float(pv.w << 16); v[3] += __uint_as_float(pv.w & 0xffff0000u);
          if (bad) u[0] = qnan;
          const u32 a0 = pack2(silu(u[0]), silu(u[1])), a1 = pack2(silu(u[2]), silu(u[3]));
          const u32 b0 = pack2(silu(v[0]), silu(v[1])), b1 = pack2(silu(v[2]), silu(v[3]));
          const auto s0 = __builtin_amdgcn_permlane16_swap(a0, b0, false, false);
          const auto s1 = __builtin_amdgcn_permlane16_swap(a1, b1, false, false);
          const u32x4 o = {s0[0], s1[0], s0[1], s1[1]};
          st16_sys(vo, o, ob + (size_t)mt * 16 * 8192 + np * 64);
        }
    }
    WAIT_VM0();
    WG_BARRIER();
  }

  if (xl == 0 || ts < 4 || ts >= 8) break;
  }
}

static void build_tables(Args& a) {
  int pos[32];
  for (int i = 0; i < 32; ++i) pos[PIECES[i]] = i;
  auto rdy = [&](int lr, int lc) { return pos[lr] > pos[16 + lc] ? pos[lr] : pos[16 + lc]; };
  int prank[128];
  for (int t = 0; t < 128; ++t) {
    const int kt = (rdy(8 + (t >> 4), t & 15) * 32 + rdy(t >> 4, t & 15)) * 128 + t;
    prank[t] = 0;
    for (int u = 0; u < 128; ++u) prank[t] += (rdy(8 + (u >> 4), u & 15) * 32 + rdy(u >> 4, u & 15)) * 128 + u < kt;
  }
  for (int xl = 0; xl < 8; ++xl) {
    int key[32], idx[32], ent[32];
    for (int t = 0; t < 32; ++t) {
      const int lr = (t >> 2) * 2 + (xl >> 2), lc = (t & 3) * 4 + (xl & 3), pr = prank[(lr & 7) * 16 + lc];
      key[t] = lr < 8 ? rdy(lr, lc) * 128 + pr : 10000 + pr;
      ent[t] = lr | (lc << 4) | (rdy(lr, lc) << 8) | (pr << 13);
      idx[t] = t;
    }
    for (int i = 1; i < 32; ++i)
      for (int j = i; j > 0 && key[idx[j]] < key[idx[j - 1]]; --j) { int s = idx[j]; idx[j] = idx[j - 1]; idx[j - 1] = s; }
    for (int slot = 0; slot < 32; ++slot) a.tile[xl * 32 + slot] = ent[idx[slot < 4 ? 28 + slot : slot - 4]];
  }
}

extern "C" void kernel_launch(int rank, int nranks, ncclComm_t comm, hipStream_t stream,
                              void* const* d_in, const int* in_sizes, int n_in,
                              void* d_out, int out_size, const int* in_dtypes, int out_dtype,
                              void* const* d_out_all, void* const* d_ws_all, size_t ws_bytes) {
  if (nranks != 8 || (unsigned)rank >= 8u || n_in < 2 || ws_bytes < WS_NEED || out_size != 4096 * 4096 ||
      in_sizes[0] != 4096 * 512 || in_sizes[1] != 512 * 4096) {
    fprintf(stderr, "kernel_launch: unexpected problem shape\n");
    return;
  }
  Args a;
  a.x = (const char*)d_in[0];
  a.w = (const char*)d_in[1];
  for (int l = 0; l < 8; ++l) { a.out[l] = (char*)d_out_all[rank ^ l]; a.ws[l] = (char*)d_ws_all[rank ^ l]; }
  a.rank = (u32)rank;
  static const unsigned char near_xcd[8][8] = {{0, 4, 2, 3, 6, 5, 1, 7}, {4, 0, 7, 5, 2, 3, 6, 1}, {2, 7, 0, 1, 4, 5, 3, 6}, {3, 5, 1, 0, 7, 4, 2, 6},
                                               {5, 2, 4, 7, 0, 1, 6, 3}, {6, 3, 5, 4, 1, 0, 7, 2}, {1, 6, 3, 2, 5, 7, 0, 4}, {7, 1, 6, 5, 3, 2, 4, 0}};
  a.xmap = 0;
  for (int p = 0; p < 8; ++p) a.xmap |= (u32)(rank ^ p) << (4 * near_xcd[rank][p]);
  build_tables(a);
  fused_gemm_ar_silu<<<256, 512, 0, stream>>>(a);
  hipError_t e = hipGetLastError();
  if (e != hipSuccess) fprintf(stderr, "launch failed: %s\n", hipGetErrorString(e));
}
```

```cpp
#include <hip/hip_runtime.h>
#include <rccl/rccl.h>
#include <cstdio>
#include <cstdint>

typedef unsigned int u32;
typedef unsigned long long u64;
typedef __attribute__((ext_vector_type(4))) float f32x4;
typedef __attribute__((ext_vector_type(4))) unsigned int u32x4;
typedef __attribute__((ext_vector_type(2))) unsigned int u32x2;
typedef __attribute__((ext_vector_type(4))) __bf16 bf16x4;
typedef __attribute__((ext_vector_type(8))) __bf16 bf16x8;
typedef __attribute__((ext_vector_type(2))) __bf16 bf16x2;
#define LAS __attribute__((address_space(3)))
#define GAS __attribute__((address_space(1)))
#define RLX __ATOMIC_RELAXED
#define SYS __HIP_MEMORY_SCOPE_SYSTEM
#define AGT __HIP_MEMORY_SCOPE_AGENT
#define WGP __HIP_MEMORY_SCOPE_WORKGROUP

constexpr size_t OFF_CTR = 0;
constexpr size_t OFF_TMO = 4096;
constexpr size_t OFF_TICK = 5120;
constexpr size_t OFF_POOL = 5376;
constexpr size_t OFF_HOLD = 6144;
constexpr size_t OFF_INFLAG = 8192;
constexpr size_t OFF_PFLAG = 16384;
constexpr size_t OFF_QUEUE = 32768;
constexpr size_t OFF_SFLAG = 45056;
constexpr size_t OFF_DBG = 65536;
constexpr size_t OFF_XS = 1u << 20;
constexpr size_t OFF_WS = 9u << 20;
constexpr size_t OFF_PS = 17u << 20;
constexpr size_t OFF_PO = 21u << 20;
constexpr size_t OFF_SX = 25u << 20;
constexpr size_t OFF_SW = 29u << 20;
constexpr size_t WS_NEED = 33u << 20;

constexpr u64 DEADLINE = 100000000ull;
constexpr int NSTAGE = 4;
constexpr int STAGE_BYTES = 32768;
constexpr int LDS_CTL = NSTAGE * STAGE_BYTES;
constexpr int LDS_BYTES = LDS_CTL + 2048;

#define PIECE_ORDER 0, 16, 1, 17, 2, 18, 3, 19, 8, 20, 21, 9, 22, 23, 10, 24, 25, 11, 26, 27, 4, 28, 29, 30, 31, 5, 12, 6, 13, 7, 14, 15
constexpr u32 PIECES[32] = {PIECE_ORDER};
constexpr u64 pack_pieces(int first) {
  u64 v = 0;
  for (int i = 0; i < 8; ++i) v |= (u64)PIECES[first + i] << (8 * i);
  return v;
}
__device__ __forceinline__ u32 piece_at(u32 i) {
  constexpr u64 p0 = pack_pieces(0), p1 = pack_pieces(8), p2 = pack_pieces(16), p3 = pack_pieces(24);
  const u64 v = i < 16 ? (i < 8 ? p0 : p1) : (i < 24 ? p2 : p3);
  return (u32)(v >> (8 * (i & 7))) & 31u;
}

constexpr u64 pack_pos(int first) {
  u64 v = 0;
  for (int i = 0; i < 32; ++i) if ((int)PIECES[i] >= first && (int)PIECES[i] < first + 8) v |= (u64)i << (8 * (PIECES[i] - first));
  return v;
}
__device__ __forceinline__ u32 pos_of(u32 c) {
  constexpr u64 p0 = pack_pos(0), p1 = pack_pos(8), p2 = pack_pos(16), p3 = pack_pos(24);
  const u64 v = c < 16 ? (c < 8 ? p0 : p1) : (c < 24 ? p2 : p3);
  return (u32)(v >> (8 * (c & 7))) & 31u;
}
#define NP0 3
#define RP 108
constexpr u32 NPOOL = 2 * RP;
struct StageOrder { u64 v[4]; };
constexpr StageOrder make_stage_order() {
  StageOrder o = {{0, 0, 0, 0}};
  bool seen[24] = {};
  int n = 0;
  for (int k = NP0; k < 32 + NP0; ++k) {
    const int c = (int)PIECES[k & 31], cl = c < 16 ? (c & 7) : c - 8;
    if (seen[cl]) continue;
    seen[cl] = true;
    for (int e = 0; e < (c < 16 ? 2 : 1); ++e, ++n) o.v[n >> 3] |= (u64)(c < 16 ? cl * 2 + e : c) << (8 * (n & 7));
  }
  return o;
}
__device__ __forceinline__ u32 stage_class(u32 i) {
  constexpr StageOrder o = make_stage_order();
  constexpr u64 p0 = o.v[0], p1 = o.v[1], p2 = o.v[2], p3 = o.v[3];
  const u64 v = i < 16 ? (i < 8 ? p0 : p1) : (i < 24 ? p2 : p3);
  return (u32)(v >> (8 * (i & 7))) & 31u;
}

struct Args {
  const char* x;
  const char* w;
  u32 rank;
  u32 xmap;
  char* ws[8];
  char* out[8];
  u32 tile[256];
};

#define STAMP(i) do {} while (0)
#define TSTAMP(i) do {} while (0)

__device__ __forceinline__ u32 grow0(u32 r, u32 lr) {
  return ((r >> 1) & 1) * 2048 + ((lr >> 3) ^ (r >> 2) ^ 1) * 1024 + (lr & 7) * 128;
}
__device__ __forceinline__ u32 gcol0(u32 r, u32 lc) { return (r & 1) * 2048 + lc * 128; }
__device__ __forceinline__ u32 tile_off(u32 r, u32 id) {
  return grow0(r, 8 + ((id >> 4) & 7)) * 8192 + gcol0(r, id & 15) * 2;
}

__device__ __forceinline__ void st16_sys(u32 voff, u32x4 v, char* sbase) {
  asm volatile("global_store_dwordx4 %0, %1, %2 sc0 sc1\n\ts_nop 1" :: "v"(voff), "v"(v), "s"(sbase) : "memory");
}
__device__ __forceinline__ void st16_agent(u32 voff, u32x4 v, char* sbase) {
  asm volatile("global_store_dwordx4 %0, %1, %2 sc1\n\ts_nop 1" :: "v"(voff), "v"(v), "s"(sbase) : "memory");
}
__device__ __forceinline__ bool expired(u64 t0) { return __builtin_amdgcn_s_memrealtime() - t0 > DEADLINE; }
#define WAIT_VM0() asm volatile("s_waitcnt vmcnt(0)" ::: "memory")
#define WG_BARRIER() asm volatile("s_waitcnt lgkmcnt(0)\n\ts_barrier" ::: "memory")

#define WIN 9
#define WINC1 1
#define WINC2 2
#define WINC3 3
#define NST 4
constexpr u32 RING = 24, RING0 = 16384;
#define SCHED_AHEAD 3
#define LOADER_PRIO 2
#define POLL_SLEEP 2
#define START_DELAY 100
#define START_SLOPE 100
#define STAGE0 (4 + 4 * NP0)
#define STAGE_SPLIT 24
#define PPOLL_SLEEP 8
#define LEAD_STEP 0
#define LEAD_T 0
#define STAGE_DELAY 80
#define POOL_AHEAD 6
constexpr u32 C_CNT = 0;
constexpr u32 C_AVAIL = 128;
constexpr u32 C_LISTED = 136, C_TOTAL = 137, C_TICKET = 138, C_BAD = 139, C_MASK = 140, C_LANDED = 141, C_TAKEN = 142, C_HELD = 143;
constexpr u32 C_FREE = 160;
constexpr u32 C_LIST = 192, C_LIST2 = 320;

__device__ __forceinline__ void glds4v(u32 v0, u32 v1, u32 v2, u32 v3, const char* sbase, u32 lds) {
  asm volatile(
      "s_mov_b32 m0, %5\n\ts_nop 0\n\t"
      "global_load_lds_dwordx4 %0, %4\n\t"
      "global_load_lds_dwordx4 %1, %4 offset:1024\n\t"
      "global_load_lds_dwordx4 %2, %4 offset:2048\n\t"
      "global_load_lds_dwordx4 %3, %4 offset:3072"
      :: "v"(v0), "v"(v1), "v"(v2), "v"(v3), "s"(sbase), "s"(lds) : "memory");
}
__device__ __forceinline__ void pull4l(u32 v0, u32 v1, u32 v2, u32 v3, const char* sbase, u32 lds) {
  asm volatile(
      "s_mov_b32 m0, %5\n\ts_waitcnt vmcnt(%6)\n\t"
      "global_load_lds_dwordx4 %0, %4 sc0 sc1\n\ts_waitcnt vmcnt(%7)\n\t"
      "global_load_lds_dwordx4 %1, %4 offset:1024 sc0 sc1\n\ts_waitcnt vmcnt(%8)\n\t"
      "global_load_lds_dwordx4 %2, %4 offset:2048 sc0 sc1\n\ts_waitcnt vmcnt(%9)\n\t"
      "global_load_lds_dwordx4 %3, %4 offset:3072 sc0 sc1"
      :: "v"(v0), "v"(v1), "v"(v2), "v"(v3), "s"(sbase), "s"(lds), "n"(WIN), "n"(WIN + WINC1), "n"(WIN + WINC2), "n"(WIN + WINC3) : "memory");
}
__device__ __forceinline__ u32 src_x(u32 s, u32 lane) {
  return (8 * s + (lane >> 3)) * 1024 + (((lane & 7) ^ (4 * (s & 1) + (lane >> 4))) * 16) - s * 1024;
}
__device__ __forceinline__ u32 src_w(u32 s, u32 lane) {
  return (4 * s + (lane >> 4)) * 8192 + (((lane & 15) ^ (((lane >> 4) | ((s >> 1) << 2)) << 1)) * 16) - s * 1024;
}
__device__ __forceinline__ u32 lds_get(LAS u32* p) {
  asm volatile("" ::: "memory");
  const u32 v = __builtin_amdgcn_readfirstlane(__hip_atomic_load(p, RLX, WGP));
  asm volatile("" ::: "memory");
  return v;
}
__device__ __forceinline__ void lds_set(LAS u32* p, u32 v) {
  asm volatile("" ::: "memory");
  __hip_atomic_store(p, v, RLX, WGP);
  asm volatile("" ::: "memory");
}
__device__ __forceinline__ u32 lds_add(LAS u32* p, u32 lane) {
  u32 t = 0;
  if (lane == 0) t = __hip_atomic_fetch_add(p, 1u, RLX, WGP);
  return __builtin_amdgcn_readfirstlane(t);
}

__device__ __forceinline__ void ld4(u32x4& d0, u32x4& d1, u32x4& d2, u32x4& d3, u32 v0, u32 v1, u32 v2, u32 v3, const char* sb) {
  asm volatile("global_load_dwordx4 %0, %4, %8\n\tglobal_load_dwordx4 %1, %5, %8 offset:1024\n\t"
               "global_load_dwordx4 %2, %6, %8 offset:2048\n\tglobal_load_dwordx4 %3, %7, %8 offset:3072\n\ts_waitcnt vmcnt(0)"
               : "=&v"(d0), "=&v"(d1), "=&v"(d2), "=&v"(d3) : "v"(v0), "v"(v1), "v"(v2), "v"(v3), "s"(sb) : "memory");
}
__device__ __forceinline__ u32 kb_first(u32 xl) { return xl < 4 ? 4 + 4 * xl : 4 * xl; }

__device__ __forceinline__ void push_leading(const Args& a, u32 p, u32 xl, u32 i, u32 j, u32 kb, u32 lane, u64 t0) {
  const u32 pc = piece_at(i), k0 = (p >> 2) * 256 + kb * 64;
  u32x4 d0, d1, d2, d3;
  if (pc >> 4) ld4(d0, d1, d2, d3, src_w(0, lane), src_w(1, lane), src_w(2, lane), src_w(3, lane),
                   a.w + ((k0 + j * 16) * 8192 + gcol0(p, pc & 15) * 2));
  else ld4(d0, d1, d2, d3, src_x(0, lane), src_x(1, lane), src_x(2, lane), src_x(3, lane),
           a.x + ((grow0(p, pc) + j * 32) * 1024 + k0 * 2));
  char* db = a.ws[xl] + (OFF_XS + (pc * 32 + kb_first(xl) + kb) * 16384 + j * 4096);
  const u32 vd = lane * 16;
  st16_sys(vd, d0, db); st16_sys(vd, d1, db + 1024); st16_sys(vd, d2, db + 2048); st16_sys(vd, d3, db + 3072);
  WAIT_VM0();
}

__device__ __forceinline__ void stage_unit(const Args& a, u32 rank, u32 unit, u32 wv, u32 lane, u32& calls, char*& pws) {
  const u32 j = unit & 3, sub = (unit >> 2) & 1, cls = stage_class(unit >> 3);
  const bool isw = cls >= 16;
  const u32 ch = isw ? sub * 16 + (cls & 15) : (cls & 1) * 16 + sub * 8 + (cls >> 1);
  u32x4 d0, d1, d2, d3;
  if (isw) ld4(d0, d1, d2, d3, src_w(0, lane), src_w(1, lane), src_w(2, lane), src_w(3, lane), a.w + ((wv * 64 + j * 16) * 8192 + ch * 256));
  else ld4(d0, d1, d2, d3, src_x(0, lane), src_x(1, lane), src_x(2, lane), src_x(3, lane), a.x + ((ch * 128 + j * 32) * 1024 + wv * 128));
  char* db = a.ws[0] + ((isw ? OFF_SW : OFF_SX) + (ch * 8 + wv) * 16384 + j * 4096);
  const u32 vd = lane * 16;
  st16_sys(vd, d0, db); st16_sys(vd, d1, db + 1024); st16_sys(vd, d2, db + 2048); st16_sys(vd, d3, db + 3072);
  asm volatile("s_waitcnt vmcnt(0)\n\ts_barrier" : "+v"(calls), "+v"(pws) :: "memory");
  if (wv == 0 && lane < 8 && (lane & 3)) {
    const u32 p = rank ^ lane;
    const u32 pos = pos_of(isw ? cls : (sub ^ (p >> 2) ^ 1) * 8 + (cls >> 1));
    if ((isw ? p & 1 : (p >> 1) & 1) == ch >> 4 && pos >= NP0)
      __hip_atomic_store((GAS u32*)(pws + OFF_SFLAG + (lane * 4 + j) * 128) + pos, calls + 1, RLX, SYS);
  }
}

__device__ __forceinline__ void link_engine(const Args& a, u32 rank, u32 xl, u32 j, u32 wv, u32 lane, u32 seq, u64 t0,
                                            LAS char* shm, u32 lds0) {
  LAS u32* ctl = (LAS u32*)(shm + LDS_CTL);
  const bool pair = xl == 4;
  const u32 f0 = (lane >> 4) * 8192 + (lane & 15) * 16, vl = lane * 16;
  char* const ws = a.ws[0];
  volatile LAS u32x2* list = (volatile LAS u32x2*)shm;
  if (wv == 5) {
    const u32 isrc = OFF_SX + (rank >> 2) * 4 * 16384 + j * 4096;
    const u32 idst = OFF_XS + kb_first(xl) * 16384 + j * 4096;
    u32 gi = 0, ni = pair ? 32u : NP0, nf = pair ? RP : 0u;
    u64 left0 = pair ? ~0ull : 0ull, left1 = left0;
    bool pool = true;
#pragma unroll 1
    while (pool || nf < 128 || (left0 | left1)) {
      asm volatile("" ::: "memory");
      const u32x4 av = *(volatile LAS u32x4*)(ctl + C_AVAIL), pm = *(volatile LAS u32x4*)(ctl + C_AVAIL + 4);
      const u32 room = lds_get(ctl + C_TICKET) + POOL_AHEAD - gi;
      const u64 a0 = left0 & (((u64)__builtin_amdgcn_readfirstlane(pm.y) << 32) | (u32)__builtin_amdgcn_readfirstlane(pm.x));
      const u64 a1 = left1 & (((u64)__builtin_amdgcn_readfirstlane(pm.w) << 32) | (u32)__builtin_amdgcn_readfirstlane(pm.z));
      const bool near = room - 1 < POOL_AHEAD && room > POOL_AHEAD - SCHED_AHEAD;
      u32 src = 0, dst = 0, n = 0, step = 16 * 8192;
      if (a0 | a1) {
        if (near) {
          const u32 id = a0 ? __builtin_ctzll(a0) : 64 + __builtin_ctzll(a1);
          if (a0) left0 &= ~(1ull << id); else left1 &= ~(1ull << (id - 64));
          src = OFF_PO + id * 32768 + j * 8192; dst = OFF_PS + id * 32768 + j * 8192 + id * 2 + 1; n = 2; step = 4096;
        }
      } else if (ni < 32) {
        if (near && ni < __builtin_amdgcn_readfirstlane(av.x)) {
          const u32 pc = piece_at(ni);
          src = isrc + (pc >> 4 ? 4096 + gcol0(rank, pc & 15) : grow0(rank, pc)) * 1024;
          dst = idst + pc * 32 * 16384 + ni * 2 + 1; n = 4; step = 16384;
          ++ni;
        }
      } else if (nf < __builtin_amdgcn_readfirstlane(av.y)) {
        if (near) {
          src = lds_get(ctl + C_LIST + nf) + j * 32 * 8192; dst = src + (nf < RP ? 2 : 0); n = 2;
          ++nf;
        }
      } else if (pool && __builtin_amdgcn_readfirstlane(av.z) && room - 1 < POOL_AHEAD) {
        u32 t = 0;
        if (lane == 0) t = __hip_atomic_fetch_add((GAS u32*)(ws + OFF_POOL), 1u, RLX, AGT);
        t = __builtin_amdgcn_readfirstlane(t) - (seq - 1) * (NPOOL + 28);
        if (t < NPOOL) { src = dst = lds_get(ctl + (pair ? C_LIST : C_LIST2) + (t >> 1)) + (t & 1) * 64 * 8192; n = 4; }
        else pool = false;
      }
      if (n) {
        if (lane < n) list[gi + lane] = (u32x2){src + lane * step, dst + lane * step};
        gi += n;
        lds_set(ctl + C_LISTED, gi);
      } else {
        __builtin_amdgcn_s_sleep(1);
      }
    }
    lds_set(ctl + C_TOTAL, gi + 1);
  } else if (wv == 0) {
    constexpr u32 BACK = (WIN + WINC3) / 4;
    const char* const pws = a.ws[xl];
    const char* const fsrc = a.out[xl];
    u32 slot = 0, pub = 0, gi = 0;
    __builtin_amdgcn_s_setprio(3);
#pragma unroll 1
    for (;; ++gi) {
      u32 src, lin;
      bool end = false;
      for (u32 tries = 0;; ++tries) {
        asm volatile("" ::: "memory");
        const u32x2 lt = *(volatile LAS u32x2*)(ctl + C_LISTED);
        const u32 fr = __hip_atomic_load(ctl + C_FREE + slot, RLX, WGP);
        asm volatile("" ::: "memory");
        const u32x2 d = list[gi];
        src = __builtin_amdgcn_readfirstlane(d.x); lin = __builtin_amdgcn_readfirstlane(d.y) & 1;
        if (__builtin_amdgcn_readfirstlane(lt.x) > gi && __builtin_amdgcn_readfirstlane(fr) + RING > gi) break;
        if (__builtin_amdgcn_readfirstlane(lt.y) == gi + 1) { end = true; break; }
        if (tries == 3) {
          WAIT_VM0(); lds_set(ctl + C_LANDED, gi); pub = gi;
        }
        __builtin_amdgcn_s_sleep(1);
      }
      if (end) break;
      if (lin) pull4l(vl, vl, vl, vl, pws + src, lds0 + RING0 + slot * 4096);
      else pull4l(f0, f0 + 31744, f0 + 63488, f0 + 95232, fsrc + src, lds0 + RING0 + slot * 4096);
      if (gi > pub + BACK) { pub = gi - BACK; lds_set(ctl + C_LANDED, pub); }
      lds_set(ctl + C_TICKET, gi + 1);
      slot = slot + 1 == RING ? 0 : slot + 1;
    }
    WAIT_VM0();
    lds_set(ctl + C_LANDED, gi);
  } else if (wv <= NST) {
    char* const fdst = a.out[0];
    u32 landed = 0, prev = 0;
    auto done = [&]() {
      if (prev & 1) {
        const u32 c = (prev >> 1) & 127;
        if (lds_add(ctl + C_CNT + c, lane) == 1 && pair && lane == 0)
          __hip_atomic_store((GAS u32*)(ws + OFF_PFLAG + c * 64) + j, seq, RLX, AGT);
      } else if (prev & 2) {
        if (lds_add(ctl + C_HELD, lane) == 2 * RP - 1 && lane == 0)
          __hip_atomic_store((GAS u32*)(a.ws[xl ^ 4] + OFF_HOLD + (xl ^ 4) * 128) + j, seq, RLX, SYS);
      }
      prev = 0;
    };
#pragma unroll 1
    for (;;) {
      const u32 gi = lds_add(ctl + C_TAKEN, lane);
      if (landed <= gi) {
        landed = lds_get(ctl + C_LANDED);
        if (landed <= gi) {
          WAIT_VM0();
          done();
          bool end = false;
          do {
            end = lds_get(ctl + C_TOTAL) - 1 <= gi;
            if (end) break;
            __builtin_amdgcn_s_sleep(1);
            landed = lds_get(ctl + C_LANDED);
          } while (landed <= gi);
          if (end) break;
        }
      }
      const u32 slot = gi % RING;
      const u32 desc = __builtin_amdgcn_readfirstlane(list[gi].y);
      const LAS u32x4* rs = (const LAS u32x4*)(shm + RING0 + slot * 4096) + lane;
      u32x4 d0 = rs[0];
      const u32x4 d1 = rs[64], d2 = rs[128], d3 = rs[192];
      lds_set(ctl + C_FREE + slot, gi + 1);
      if (lds_get(ctl + C_BAD)) d0.x = 0x7fc07fc0u;
      WAIT_VM0();
      if (desc & 1) {
        char* db = ws + (desc & ~4095u);
        st16_agent(vl, d0, db); st16_agent(vl, d1, db + 1024); st16_agent(vl, d2, db + 2048); st16_agent(vl, d3, db + 3072);
      } else {
        char* db = fdst + (desc & ~255u);
        st16_sys(f0, d0, db); st16_sys(f0, d1, db + 32768); st16_sys(f0, d2, db + 65536); st16_sys(f0, d3, db + 98304);
      }
      done();
      prev = desc;
    }
  } else if (wv == 6) {
    const u32 tag = seq & 0xffffff;
    const GAS u32* qf = (const GAS u32*)(ws + OFF_QUEUE + xl * 1024) + lane;
    const GAS u32* qs = pair ? qf + 128 : lane < 32 ? (const GAS u32*)(ws + OFF_SFLAG + (xl * 4 + j) * 128) + lane
                                                    : (const GAS u32*)(ws + OFF_HOLD + xl * 128) + (lane & 3);
    u32 nf = 0, open = 0;
    for (;;) {
      u32 e0 = __hip_atomic_load(qf, RLX, SYS), e1 = __hip_atomic_load(qf + 64, RLX, SYS), e2 = __hip_atomic_load(qs, RLX, SYS);
      u32 e3 = seq;
      if (pair) e3 = __hip_atomic_load(qs + 64, RLX, SYS);
      const bool late = expired(t0);
      if (late) { lds_set(ctl + C_BAD, 1); e0 = (seq << 8) | lane; e1 = (seq << 8) | (64 + lane); e2 = e3 = seq; }
      const u64 n0 = __ballot((e0 >> 8) != tag), n1 = __ballot((e1 >> 8) != tag);
      const u64 m2 = __ballot(e2 >= seq || (!pair && lane < NP0)), m3 = __ballot(e3 >= seq);
      const u32 cf = n0 ? (u32)__builtin_ctzll(n0) : (n1 ? 64u + (u32)__builtin_ctzll(n1) : 128u);
      const u32 staged = (u32)m2 == ~0u ? 32u : (u32)__builtin_ctz(~(u32)m2);
      if (cf != nf) {
        if (lane >= nf) ctl[C_LIST + lane] = tile_off(rank ^ xl, e0);
        if (64 + lane >= nf) ctl[C_LIST + 64 + lane] = tile_off(rank ^ xl, e1);
        nf = cf;
      }
      if (pair) {
        open = cf >= RP;
        *(volatile LAS u32x4*)(ctl + C_AVAIL + 4) = (u32x4){(u32)m2, (u32)(m2 >> 32), (u32)m3, (u32)(m3 >> 32)};
      } else if (!open && (u32)(m2 >> 32) == ~0u) {
        const GAS u32* qp = (const GAS u32*)(ws + OFF_QUEUE + 4 * 1024) + lane;
        const u32 g0 = __hip_atomic_load(qp, RLX, SYS), g1 = __hip_atomic_load(qp + 64, RLX, SYS);
        ctl[C_LIST2 + lane] = tile_off(rank ^ 4, g0);
        ctl[C_LIST2 + 64 + lane] = tile_off(rank ^ 4, g1);
        open = late || !__ballot(((g0 >> 8) != tag && lane < RP) || ((g1 >> 8) != tag && 64 + lane < RP));
      }
      asm volatile("" ::: "memory");
      *(volatile LAS u32x4*)(ctl + C_AVAIL) = (u32x4){staged, cf, open, 0u};
      if (cf == 128 && open && (!pair || (m2 & m3) == ~0ull)) break;
      if (!pair && staged == 32) __builtin_amdgcn_s_sleep(8);
    }
  } else if (wv == 7 && !pair) {
    GAS u32* const flag = (GAS u32*)(ws + OFF_INFLAG + xl * 16 + j * 4);
    const GAS u32* lf = (const GAS u32*)(ws + OFF_SFLAG + (xl * 4 + j) * 128) + (lane & 31);
    while (__ballot(__hip_atomic_load(lf, RLX, SYS) < seq && lane < NP0)) {
      if (expired(t0)) { lds_set(ctl + C_BAD, 1); break; }
      __builtin_amdgcn_s_sleep(1);
    }
    if (lane < NP0) __hip_atomic_store(flag + lane * 64, seq, RLX, AGT);
#pragma unroll 1
    for (u32 pi = NP0; pi < 32; ++pi) {
      while (lds_get(ctl + C_CNT + pi) < 4) __builtin_amdgcn_s_sleep(1);
      if (lane == 0) __hip_atomic_store(flag + pi * 64, seq, RLX, AGT);
    }
  }
}

__device__ __forceinline__ u32 pack2(float lo, float hi) {
  bf16x2 v = {(__bf16)lo, (__bf16)hi};
  return __builtin_bit_cast(u32, v);
}
__device__ __forceinline__ float silu(float y) {
  const float c = __builtin_fminf(__builtin_fmaxf(y, -60.f), 60.f);
  return y * __builtin_amdgcn_rcpf(1.f + __builtin_amdgcn_exp2f(c * -1.44269504088896341f));
}

__global__ void __launch_bounds__(512, 2) fused_gemm_ar_silu(const Args a) {
  __shared__ __attribute__((aligned(1024))) char shm[LDS_BYTES];
  const u32 tid = threadIdx.x, lane = tid & 63;
  const u32 wv = __builtin_amdgcn_readfirstlane(tid >> 6);
  const u32 b = blockIdx.x, slot = b >> 3;
  const u32 xl = (a.xmap >> (4 * (b & 7))) & 7;
  LAS u32* lctr = (LAS u32*)(shm + LDS_CTL);
  const u32 lds0 = (u32)(size_t)(LAS char*)shm;
  const bool engine = xl != 0 && slot < 4;
  const u64 t0 = __builtin_amdgcn_s_memrealtime();
  const u32 rank = a.rank & 7;
  char* const ws = a.ws[0];
  GAS u32* ctr = (GAS u32*)(ws + OFF_CTR) + b;
  u32 calls = __hip_atomic_load(ctr, RLX, AGT);
  char* pws = a.ws[lane & 7];
  if (wv == 7) { lctr[lane] = 0; lctr[64 + lane] = 0; lctr[128 + lane] = 0; }
  const u32 lead = ((slot - 4) >> 2) + (wv >> 2) * 32;
  const bool is_lead = (xl & 3) && slot >= 4 && lead < NP0;
  if (is_lead) push_leading(a, rank ^ xl, xl, lead, slot & 3, wv & 3, lane, t0);
  if (!(xl & 3)) {
    if (slot >= 32 - STAGE_SPLIT) {
      stage_unit(a, rank, (slot - (32 - STAGE_SPLIT)) * 2 + (xl >> 2), wv, lane, calls, pws);
      if (wv == 0) STAMP(11);
    }
  } else if (slot >= STAGE0) {
    while (__builtin_amdgcn_s_memrealtime() - t0 < STAGE_DELAY) __builtin_amdgcn_s_sleep(2);
    for (u32 u = 2 * STAGE_SPLIT + (slot - STAGE0) * 6 + (xl < 4 ? xl - 1 : xl - 2); u < 256; u += (32 - STAGE0) * 6) {
      stage_unit(a, rank, u, wv, lane, calls, pws);
      if (wv == 0) STAMP(10);
    }
  }
  asm volatile("" : "+v"(calls));
  const u32 seq = __builtin_amdgcn_readfirstlane(calls) + 1;
  WG_BARRIER();
  if (is_lead && (tid & 255) == 0)
    __hip_atomic_store((GAS u32*)(a.ws[xl] + OFF_SFLAG + (xl * 4 + (slot & 3)) * 128) + lead, seq, RLX, SYS);
  if (is_lead && wv == 0) STAMP(11);
  if (tid == 0) __hip_atomic_store(ctr, seq, RLX, AGT);
  if (engine) {
    link_engine(a, rank, xl, slot, wv, lane, seq, t0, (LAS char*)shm, lds0);
    return;
  }

#pragma unroll 1
  for (u32 ts = slot;; ts -= 4) {
  const u32 te = a.tile[xl * 32 + ts];
  const u32 lr = te & 15, lc = (te >> 4) & 15, rpos = (te >> 8) & 31, tix = (te >> 13) & 127;
  const u32 kind = lr >> 3;
  const u32 row0 = grow0(rank, lr), col0 = gcol0(rank, lc);
  const u32 vl = lane * 16;

  if (wv >= 4) {
    const u32 w4 = wv - 4;
    const char* xo = a.x + (size_t)(row0 + w4 * 32) * 1024;
    const char* wo = a.w + ((size_t)w4 * 16 * 8192 + col0 * 2);
    const char* xs = ws + (OFF_XS + (size_t)lr * 32 * 16384 + w4 * 4096);
    const char* wsg = ws + (OFF_WS + (size_t)lc * 32 * 16384 + w4 * 4096);
    const u32 lq = lds0 + w4 * 4096;
    const GAS u32x4* inflag = (const GAS u32x4*)(ws + OFF_INFLAG + rpos * 256 + (lane & 7) * 16);
    const GAS u32x4* lastflag = (lane & 7) ? inflag : (const GAS u32x4*)(ws + OFF_PFLAG + tix * 64);
    { const u32 d = START_DELAY + rpos * START_SLOPE; while (__builtin_amdgcn_s_memrealtime() - t0 < d) __builtin_amdgcn_s_sleep(4); }
    if (wv == 7 && ts == slot) {
      __builtin_amdgcn_fence(__ATOMIC_ACQUIRE, "");
      WAIT_VM0();
    }
    __builtin_amdgcn_s_setprio(LOADER_PRIO);
    {
      const u32 x0 = src_x(0, lane), x1 = src_x(1, lane), x2 = src_x(2, lane), x3 = src_x(3, lane);
      const u32 y0 = src_w(0, lane), y1 = src_w(1, lane), y2 = src_w(2, lane), y3 = src_w(3, lane);
#pragma unroll
      for (u32 kb = 0; kb < 8; ++kb) {
        if (kb >= 3) asm volatile("s_waitcnt vmcnt(16)\n\ts_barrier" ::: "memory");
        glds4v(x0, x1, x2, x3, xo + kb * 128, lq + (kb & 3) * STAGE_BYTES);
        glds4v(y0, y1, y2, y3, wo + (size_t)kb * 64 * 8192, lq + (kb & 3) * STAGE_BYTES + 16384);
      }
    }
    const u32 nst = 32 + kind;
    u32 avail = 8, issued = 8, ord = 0, seen = 0;
    auto refresh = [&]() {
      if (w4 == 0) {
        u32x4 f;
        const GAS u32x4* fp = issued >= 20 ? lastflag : inflag;
        asm volatile("global_load_dwordx4 %0, %1, off sc0 sc1\n\ts_waitcnt vmcnt(0)" : "=&v"(f) : "v"(fp) : "memory");
        u32 mask = (u32)__ballot(f.x >= seq && f.y >= seq && f.z >= seq && f.w >= seq);
        if (expired(t0)) { lds_set(lctr + C_BAD, 1); mask = 0xff; }
        ord |= (mask & 1) << 21;
        for (u32 m = mask & 0xee & ~seen; m; m &= m - 1) {
          const u32 l = __builtin_ctz(m);
          ord = (ord | ((l < 4 ? l + 1 : l) << (3 * ((ord >> 18) & 7)))) + (1u << 18);
        }
        seen |= mask;
        lds_set(lctr + C_MASK, ord);
      } else {
        ord = lds_get(lctr + C_MASK);
      }
      const u32 n = (ord >> 18) & 7;
      avail = 8 + 4 * n + (n == 6 ? (ord >> 21) & kind : 0);
    };
#pragma unroll 1
    for (u32 s = 5; s <= 32; ++s) {
      while (issued <= s + 2) {
        if (issued >= avail) {
          if (issued >= nst) break;
          refresh();
          while (issued >= avail && issued == s) {
            if (issued == 32) __builtin_amdgcn_s_sleep(PPOLL_SLEEP); else __builtin_amdgcn_s_sleep(POLL_SLEEP);
            refresh();
          }
          if (issued >= avail) break;
        }
        const u32 l = lq + (issued & 3) * STAGE_BYTES;
        const u32 kk = ((ord >> (3 * ((issued - 8) >> 2))) & 7) * 4 + (issued & 3);
        const char* sx = xs + kk * 16384;
        const char* sw = wsg + kk * 16384;
        if (issued == 32) { sx = ws + (OFF_PS + (size_t)tix * 32768 + w4 * 4096); sw = sx + 16384; }
        glds4v(vl, vl, vl, vl, sx, l);
        glds4v(vl, vl, vl, vl, sw, l + 16384);
        ++issued;
      }
      const u32 ahead = issued - s;
      if (ahead >= 3) asm volatile("s_waitcnt vmcnt(16)\n\ts_barrier" ::: "memory");
      else if (ahead == 2) asm volatile("s_waitcnt vmcnt(8)\n\ts_barrier" ::: "memory");
      else asm volatile("s_waitcnt vmcnt(0) lgkmcnt(0)\n\ts_barrier" ::: "memory");
    }
    __builtin_amdgcn_s_setprio(0);
    u32 pos = 0;
    if (kind && wv == 4 && lane == 0) pos = __hip_atomic_fetch_add((GAS u32*)(ws + OFF_TICK), 1u, RLX, AGT);
    pos = __builtin_amdgcn_readfirstlane(pos);
    if (wv == 4) lds_set(lctr + C_MASK, 0);
    WG_BARRIER();
    if (wv == 4) {
      const u32 e = (seq << 8) | ((lr & 7) * 16 + lc);
      if (kind) {
        if (lane >= 1 && lane < 8) __hip_atomic_store((GAS u32*)(pws + OFF_QUEUE + lane * 1024) + (pos & 127), e, RLX, SYS);
      } else if (lane == 0) {
        __hip_atomic_store((GAS u32*)(a.ws[4] + OFF_QUEUE + 4 * 1024 + 512) + tix, seq, RLX, SYS);
      }
      TSTAMP(7);
    }
  } else {
    const u32 wn = wv & 1, wm = wv >> 1;
    const u32 g = lane >> 4, q = (lane >> 2) & 3, pp = lane & 3;
    u32 ax[2];
#pragma unroll
    for (int ks = 0; ks < 2; ++ks) ax[ks] = (wm * 64 + (lane & 15)) * 128 + (((4 * ks + g) ^ ((lane & 15) >> 1)) * 16);
    u32 aw[4];
#pragma unroll
    for (int nt = 0; nt < 4; ++nt)
      aw[nt] = 16384 + (8 * g + q) * 256 + (((2 * (wn * 4 + nt) + (pp >> 1)) ^ ((q | ((g & 1) << 2)) << 1)) * 16) + (pp & 1) * 8;

    f32x4 acc[4][4];
#pragma unroll
    for (int i = 0; i < 4; ++i)
#pragma unroll
      for (int jj = 0; jj < 4; ++jj) acc[i][jj] = (f32x4){0.f, 0.f, 0.f, 0.f};
    bf16x8 xa[4], wa[4], xb[4], wb[4];
    auto rd = [&](bf16x8* fx, bf16x8* fw, u32 t, int ks) {
      const LAS char* st = (const LAS char*)shm + (t & (NSTAGE - 1)) * STAGE_BYTES;
#pragma unroll
      for (int i = 0; i < 4; ++i) {
        const bf16x4 lo = __builtin_amdgcn_ds_read_tr16_b64_v4bf16((LAS bf16x4*)(st + aw[i] + ks * 8192));
        const bf16x4 hi = __builtin_amdgcn_ds_read_tr16_b64_v4bf16((LAS bf16x4*)(st + aw[i] + ks * 8192 + 1024));
        fw[i] = __builtin_shufflevector(lo, hi, 0, 1, 2, 3, 4, 5, 6, 7);
        fx[i] = *(const LAS bf16x8*)(st + ax[ks] + i * 2048);
      }
    };
    auto mm = [&](const bf16x8* fx, const bf16x8* fw) {
#pragma unroll
      for (int mt = 0; mt < 4; ++mt)
#pragma unroll
        for (int nt = 0; nt < 4; ++nt)
          acc[nt][mt] = __builtin_amdgcn_mfma_f32_16x16x32_bf16(fw[nt], fx[mt], acc[nt][mt], 0, 0, 0);
    };
#define INTERLEAVE() do { _Pragma("unroll") for (int i_ = 0; i_ < 12; ++i_) { \
      __builtin_amdgcn_sched_group_barrier(0x8, 1, 0); __builtin_amdgcn_sched_group_barrier(0x100, 1, 0); } \
      __builtin_amdgcn_sched_group_barrier(0x8, 4, 0); } while (0)
    WG_BARRIER();
    rd(xa, wa, 0, 0);
    __builtin_amdgcn_sched_barrier(0);
    for (u32 t = 0; t < 31; ++t) {
      rd(xb, wb, t, 1);
      mm(xa, wa);
      INTERLEAVE();
      __builtin_amdgcn_sched_barrier(0);
      WG_BARRIER();
      __builtin_amdgcn_sched_barrier(0);
      rd(xa, wa, t + 1, 0);
      mm(xb, wb);
      INTERLEAVE();
      __builtin_amdgcn_sched_barrier(0);
    }
    rd(xb, wb, 31, 1);
    mm(xa, wa);
    INTERLEAVE();
    __builtin_amdgcn_sched_barrier(0);
    mm(xb, wb);
    if (wv == 0) TSTAMP(5);
    WG_BARRIER();
    const bool bad = lds_get(lctr + C_BAD) != 0;
    const float qnan = __builtin_nanf("");
    if (kind == 0) {
      char* db = ws + OFF_PO + (size_t)tix * 32768 + wv * 8192;
#pragma unroll
      for (int mt = 0; mt < 4; ++mt)
#pragma unroll
        for (int np = 0; np < 2; ++np) {
          f32x4 u = acc[2 * np][mt], v = acc[2 * np + 1][mt];
          if (bad) u[0] = qnan;
          const u32x4 o = {pack2(u[0], u[1]), pack2(u[2], u[3]), pack2(v[0], v[1]), pack2(v[2], v[3])};
          st16_sys(vl, o, db + (mt * 2 + np) * 1024);
        }
    } else {
      const LAS u32x4* pr = (const LAS u32x4*)(shm + (32 & (NSTAGE - 1)) * STAGE_BYTES + wv * 8192) + lane;
      char* ob = a.out[0] + (size_t)(row0 + wm * 64) * 8192 + (col0 + wn * 64) * 2;
      const u32 vo = (lane & 15) * 8192 + (g & 1) * 32 + (g >> 1) * 16;
#pragma unroll
      for (int mt = 0; mt < 4; ++mt)
#pragma unroll
        for (int np = 0; np < 2; ++np) {
          const u32x4 pv = pr[(mt * 2 + np) * 64];
          f32x4 u = acc[2 * np][mt], v = acc[2 * np + 1][mt];
          u[0] += __uint_as_float(pv.x << 16); u[1] += __uint_as_float(pv.x & 0xffff0000u);
          u[2] += __uint_as_float(pv.y << 16); u[3] += __uint_as_float(pv.y & 0xffff0000u);
          v[0] += __uint_as_float(pv.z << 16); v[1] += __uint_as_float(pv.z & 0xffff0000u);
          v[2] += __uint_as_float(pv.w << 16); v[3] += __uint_as_float(pv.w & 0xffff0000u);
          if (bad) u[0] = qnan;
          const u32 a0 = pack2(silu(u[0]), silu(u[1])), a1 = pack2(silu(u[2]), silu(u[3]));
          const u32 b0 = pack2(silu(v[0]), silu(v[1])), b1 = pack2(silu(v[2]), silu(v[3]));
          const auto s0 = __builtin_amdgcn_permlane16_swap(a0, b0, false, false);
          const auto s1 = __builtin_amdgcn_permlane16_swap(a1, b1, false, false);
          const u32x4 o = {s0[0], s1[0], s0[1], s1[1]};
          st16_sys(vo, o, ob + (size_t)mt * 16 * 8192 + np * 64);
        }
    }
    WAIT_VM0();
    WG_BARRIER();
  }

  if (xl == 0 || ts < 4 || ts >= 8) break;
  }
}

static void build_tables(Args& a) {
  int pos[32];
  for (int i = 0; i < 32; ++i) pos[PIECES[i]] = i;
  auto rdy = [&](int lr, int lc) { return pos[lr] > pos[16 + lc] ? pos[lr] : pos[16 + lc]; };
  int prank[128];
  for (int t = 0; t < 128; ++t) {
    const int kt = (rdy(8 + (t >> 4), t & 15) * 32 + rdy(t >> 4, t & 15)) * 128 + t;
    prank[t] = 0;
    for (int u = 0; u < 128; ++u) prank[t] += (rdy(8 + (u >> 4), u & 15) * 32 + rdy(u >> 4, u & 15)) * 128 + u < kt;
  }
  for (int xl = 0; xl < 8; ++xl) {
    int key[32], idx[32], ent[32];
    for (int t = 0; t < 32; ++t) {
      const int lr = (t >> 2) * 2 + (xl >> 2), lc = (t & 3) * 4 + (xl & 3), pr = prank[(lr & 7) * 16 + lc];
      key[t] = lr < 8 ? rdy(lr, lc) * 128 + pr : 10000 + pr;
      ent[t] = lr | (lc << 4) | (rdy(lr, lc) << 8) | (pr << 13);
      idx[t] = t;
    }
    for (int i = 1; i < 32; ++i)
      for (int j = i; j > 0 && key[idx[j]] < key[idx[j - 1]]; --j) { int s = idx[j]; idx[j] = idx[j - 1]; idx[j - 1] = s; }
    for (int slot = 0; slot < 32; ++slot) a.tile[xl * 32 + slot] = ent[idx[slot < 4 ? 28 + slot : slot - 4]];
  }
}

extern "C" void kernel_launch(int rank, int nranks, ncclComm_t comm, hipStream_t stream,
                              void* const* d_in, const int* in_sizes, int n_in,
                              void* d_out, int out_size, const int* in_dtypes, int out_dtype,
                              void* const* d_out_all, void* const* d_ws_all, size_t ws_bytes) {
  if (nranks != 8 || (unsigned)rank >= 8u || n_in < 2 || ws_bytes < WS_NEED || out_size != 4096 * 4096 ||
      in_sizes[0] != 4096 * 512 || in_sizes[1] != 512 * 4096) {
    fprintf(stderr, "kernel_launch: unexpected problem shape\n");
    return;
  }
  Args a;
  a.x = (const char*)d_in[0];
  a.w = (const char*)d_in[1];
  for (int l = 0; l < 8; ++l) { a.out[l] = (char*)d_out_all[rank ^ l]; a.ws[l] = (char*)d_ws_all[rank ^ l]; }
  a.rank = (u32)rank;
  static const unsigned char near_xcd[8][8] = {{0, 4, 2, 3, 6, 5, 1, 7}, {4, 0, 7, 5, 2, 3, 6, 1}, {2, 7, 0, 1, 4, 5, 3, 6}, {3, 5, 1, 0, 7, 4, 2, 6},
                                               {5, 2, 4, 7, 0, 1, 6, 3}, {6, 3, 5, 4, 1, 0, 7, 2}, {1, 6, 3, 2, 5, 7, 0, 4}, {7, 1, 6, 5, 3, 2, 4, 0}};
  a.xmap = 0;
  for (int p = 0; p < 8; ++p) a.xmap |= (u32)(rank ^ p) << (4 * near_xcd[rank][p]);
  build_tables(a);
  fused_gemm_ar_silu<<<256, 512, 0, stream>>>(a);
  hipError_t e = hipGetLastError();
  if (e != hipSuccess) fprintf(stderr, "launch failed: %s\n", hipGetErrorString(e));
}
```

```cpp
#include <hip/hip_runtime.h>
#include <rccl/rccl.h>
#include <cstdio>
#include <cstdint>

typedef unsigned int u32;
typedef unsigned long long u64;
typedef __attribute__((ext_vector_type(4))) float f32x4;
typedef __attribute__((ext_vector_type(4))) unsigned int u32x4;
typedef __attribute__((ext_vector_type(2))) unsigned int u32x2;
typedef __attribute__((ext_vector_type(4))) __bf16 bf16x4;
typedef __attribute__((ext_vector_type(8))) __bf16 bf16x8;
typedef __attribute__((ext_vector_type(2))) __bf16 bf16x2;
#define LAS __attribute__((address_space(3)))
#define GAS __attribute__((address_space(1)))
#define RLX __ATOMIC_RELAXED
#define SYS __HIP_MEMORY_SCOPE_SYSTEM
#define AGT __HIP_MEMORY_SCOPE_AGENT
#define WGP __HIP_MEMORY_SCOPE_WORKGROUP

constexpr size_t OFF_CTR = 0;
constexpr size_t OFF_TMO = 4096;
constexpr size_t OFF_TICK = 5120;
constexpr size_t OFF_POOL = 5376;
constexpr size_t OFF_HOLD = 6144;
constexpr size_t OFF_INFLAG = 8192;
constexpr size_t OFF_PFLAG = 16384;
constexpr size_t OFF_QUEUE = 32768;
constexpr size_t OFF_SFLAG = 45056;
constexpr size_t OFF_DBG = 65536;
constexpr size_t OFF_XS = 1u << 20;
constexpr size_t OFF_WS = 9u << 20;
constexpr size_t OFF_PS = 17u << 20;
constexpr size_t OFF_PO = 21u << 20;
constexpr size_t OFF_SX = 25u << 20;
constexpr size_t OFF_SW = 29u << 20;
constexpr size_t WS_NEED = 33u << 20;

constexpr u64 DEADLINE = 100000000ull;
constexpr int NSTAGE = 4;
constexpr int STAGE_BYTES = 32768;
constexpr int LDS_CTL = NSTAGE * STAGE_BYTES;
constexpr int LDS_BYTES = LDS_CTL + 2048;

#define PIECE_ORDER 0, 16, 1, 17, 2, 18, 3, 19, 8, 20, 21, 9, 22, 23, 10, 24, 25, 11, 26, 27, 4, 28, 29, 30, 31, 5, 12, 6, 13, 7, 14, 15
constexpr u32 PIECES[32] = {PIECE_ORDER};
constexpr u64 pack_pieces(int first) {
  u64 v = 0;
  for (int i = 0; i < 8; ++i) v |= (u64)PIECES[first + i] << (8 * i);
  return v;
}
__device__ __forceinline__ u32 piece_at(u32 i) {
  constexpr u64 p0 = pack_pieces(0), p1 = pack_pieces(8), p2 = pack_pieces(16), p3 = pack_pieces(24);
  const u64 v = i < 16 ? (i < 8 ? p0 : p1) : (i < 24 ? p2 : p3);
  return (u32)(v >> (8 * (i & 7))) & 31u;
}

constexpr u64 pack_pos(int first) {
  u64 v = 0;
  for (int i = 0; i < 32; ++i) if ((int)PIECES[i] >= first && (int)PIECES[i] < first + 8) v |= (u64)i << (8 * (PIECES[i] - first));
  return v;
}
__device__ __forceinline__ u32 pos_of(u32 c) {
  constexpr u64 p0 = pack_pos(0), p1 = pack_pos(8), p2 = pack_pos(16), p3 = pack_pos(24);
  const u64 v = c < 16 ? (c < 8 ? p0 : p1) : (c < 24 ? p2 : p3);
  return (u32)(v >> (8 * (c & 7))) & 31u;
}
#define NP0 3
#define RP 100
#define ANN 1
#define ANN_SLEEP 32
#define PAIR_OPEN 64
#define FINE 28
constexpr u32 NBIG = RP - FINE, NPOOL = NBIG + 2 * FINE;
struct StageOrder { u64 v[4]; };
constexpr StageOrder make_stage_order() {
  StageOrder o = {{0, 0, 0, 0}};
  bool seen[24] = {};
  int n = 0;
  for (int k = NP0; k < 32 + NP0; ++k) {
    const int c = (int)PIECES[k & 31], cl = c < 16 ? (c & 7) : c - 8;
    if (seen[cl]) continue;
    seen[cl] = true;
    for (int e = 0; e < (c < 16 ? 2 : 1); ++e, ++n) o.v[n >> 3] |= (u64)(c < 16 ? cl * 2 + e : c) << (8 * (n & 7));
  }
  return o;
}
__device__ __forceinline__ u32 stage_class(u32 i) {
  constexpr StageOrder o = make_stage_order();
  constexpr u64 p0 = o.v[0], p1 = o.v[1], p2 = o.v[2], p3 = o.v[3];
  const u64 v = i < 16 ? (i < 8 ? p0 : p1) : (i < 24 ? p2 : p3);
  return (u32)(v >> (8 * (i & 7))) & 31u;
}

struct Args {
  const char* x;
  const char* w;
  u32 rank;
  u32 xmap;
  char* ws[8];
  char* out[8];
  u32 tile[256];
};

#define STAMP(i) do {} while (0)
#define TSTAMP(i) do {} while (0)

__device__ __forceinline__ u32 grow0(u32 r, u32 lr) {
  return ((r >> 1) & 1) * 2048 + ((lr >> 3) ^ (r >> 2) ^ 1) * 1024 + (lr & 7) * 128;
}
__device__ __forceinline__ u32 gcol0(u32 r, u32 lc) { return (r & 1) * 2048 + lc * 128; }
__device__ __forceinline__ u32 tile_off(u32 r, u32 id) {
  return grow0(r, 8 + ((id >> 4) & 7)) * 8192 + gcol0(r, id & 15) * 2;
}

__device__ __forceinline__ void st16_sys(u32 voff, u32x4 v, char* sbase) {
  asm volatile("global_store_dwordx4 %0, %1, %2 sc0 sc1\n\ts_nop 1" :: "v"(voff), "v"(v), "s"(sbase) : "memory");
}
__device__ __forceinline__ void st16_agent(u32 voff, u32x4 v, char* sbase) {
  asm volatile("global_store_dwordx4 %0, %1, %2 sc1\n\ts_nop 1" :: "v"(voff), "v"(v), "s"(sbase) : "memory");
}
__device__ __forceinline__ bool expired(u64 t0) { return __builtin_amdgcn_s_memrealtime() - t0 > DEADLINE; }
#define WAIT_VM0() asm volatile("s_waitcnt vmcnt(0)" ::: "memory")
#define WG_BARRIER() asm volatile("s_waitcnt lgkmcnt(0)\n\ts_barrier" ::: "memory")

#define WIN 8
#define WINC1 1
#define WINC2 2
#define WINC3 3
#define NST 4
constexpr u32 RING = 24, RING0 = 16384;
#define SCHED_AHEAD 7
#define LOADER_PRIO 2
#define POLL_SLEEP 2
#define START_DELAY 100
#define START_SLOPE 100
#define LEAD0 4
#define STAGE0 (LEAD0 + 4 * NP0)
#define STAGE_SPLIT 24
#define STAGE_QUIET (2 * STAGE_SPLIT)
#define PPOLL_SLEEP 8
#define LEAD_STEP 0
#define LEAD_T 0
#define STAGE_DELAY 80
#define POOL_AHEAD 7
constexpr u32 C_CNT = 0;
constexpr u32 C_AVAIL = 128;
constexpr u32 C_LISTED = 136, C_TOTAL = 137, C_TICKET = 138, C_BAD = 139, C_MASK = 140, C_LANDED = 141, C_TAKEN = 142, C_HELD = 143;
constexpr u32 C_FREE = 160;
constexpr u32 C_LIST = 192, C_LIST2 = 320;

__device__ __forceinline__ void glds4v(u32 v0, u32 v1, u32 v2, u32 v3, const char* sbase, u32 lds) {
  asm volatile(
      "s_mov_b32 m0, %5\n\ts_nop 0\n\t"
      "global_load_lds_dwordx4 %0, %4\n\t"
      "global_load_lds_dwordx4 %1, %4 offset:1024\n\t"
      "global_load_lds_dwordx4 %2, %4 offset:2048\n\t"
      "global_load_lds_dwordx4 %3, %4 offset:3072"
      :: "v"(v0), "v"(v1), "v"(v2), "v"(v3), "s"(sbase), "s"(lds) : "memory");
}
__device__ __forceinline__ void pull4l(u32 v0, u32 v1, u32 v2, u32 v3, const char* sbase, u32 lds) {
  asm volatile(
      "s_mov_b32 m0, %5\n\ts_waitcnt vmcnt(%6)\n\t"
      "global_load_lds_dwordx4 %0, %4 sc0 sc1\n\ts_waitcnt vmcnt(%7)\n\t"
      "global_load_lds_dwordx4 %1, %4 offset:1024 sc0 sc1\n\ts_waitcnt vmcnt(%8)\n\t"
      "global_load_lds_dwordx4 %2, %4 offset:2048 sc0 sc1\n\ts_waitcnt vmcnt(%9)\n\t"
      "global_load_lds_dwordx4 %3, %4 offset:3072 sc0 sc1"
      :: "v"(v0), "v"(v1), "v"(v2), "v"(v3), "s"(sbase), "s"(lds), "n"(WIN), "n"(WIN + WINC1), "n"(WIN + WINC2), "n"(WIN + WINC3) : "memory");
}
__device__ __forceinline__ u32 src_x(u32 s, u32 lane) {
  return (8 * s + (lane >> 3)) * 1024 + (((lane & 7) ^ (4 * (s & 1) + (lane >> 4))) * 16) - s * 1024;
}
__device__ __forceinline__ u32 src_w(u32 s, u32 lane) {
  return (4 * s + (lane >> 4)) * 8192 + (((lane & 15) ^ (((lane >> 4) | ((s >> 1) << 2)) << 1)) * 16) - s * 1024;
}
__device__ __forceinline__ u32 lds_get(LAS u32* p) {
  asm volatile("" ::: "memory");
  const u32 v = __builtin_amdgcn_readfirstlane(__hip_atomic_load(p, RLX, WGP));
  asm volatile("" ::: "memory");
  return v;
}
__device__ __forceinline__ void lds_set(LAS u32* p, u32 v) {
  asm volatile("" ::: "memory");
  __hip_atomic_store(p, v, RLX, WGP);
  asm volatile("" ::: "memory");
}
__device__ __forceinline__ u32 lds_add(LAS u32* p, u32 lane) {
  u32 t = 0;
  if (lane == 0) t = __hip_atomic_fetch_add(p, 1u, RLX, WGP);
  return __builtin_amdgcn_readfirstlane(t);
}

__device__ __forceinline__ void ld4(u32x4& d0, u32x4& d1, u32x4& d2, u32x4& d3, u32 v0, u32 v1, u32 v2, u32 v3, const char* sb) {
  asm volatile("global_load_dwordx4 %0, %4, %8\n\tglobal_load_dwordx4 %1, %5, %8 offset:1024\n\t"
               "global_load_dwordx4 %2, %6, %8 offset:2048\n\tglobal_load_dwordx4 %3, %7, %8 offset:3072\n\ts_waitcnt vmcnt(0)"
               : "=&v"(d0), "=&v"(d1), "=&v"(d2), "=&v"(d3) : "v"(v0), "v"(v1), "v"(v2), "v"(v3), "s"(sb) : "memory");
}
__device__ __forceinline__ u32 kb_first(u32 xl) { return xl < 4 ? 4 + 4 * xl : 4 * xl; }

__device__ __forceinline__ void push_leading(const Args& a, u32 p, u32 xl, u32 i, u32 j, u32 kb, u32 lane, u64 t0) {
  const u32 pc = piece_at(i), k0 = (p >> 2) * 256 + kb * 64;
  u32x4 d0, d1, d2, d3;
  if (pc >> 4) ld4(d0, d1, d2, d3, src_w(0, lane), src_w(1, lane), src_w(2, lane), src_w(3, lane),
                   a.w + ((k0 + j * 16) * 8192 + gcol0(p, pc & 15) * 2));
  else ld4(d0, d1, d2, d3, src_x(0, lane), src_x(1, lane), src_x(2, lane), src_x(3, lane),
           a.x + ((grow0(p, pc) + j * 32) * 1024 + k0 * 2));
  char* db = a.ws[xl] + (OFF_XS + (pc * 32 + kb_first(xl) + kb) * 16384 + j * 4096);
  const u32 vd = lane * 16;
  st16_sys(vd, d0, db); st16_sys(vd, d1, db + 1024); st16_sys(vd, d2, db + 2048); st16_sys(vd, d3, db + 3072);
  WAIT_VM0();
}

__device__ __forceinline__ void stage_unit(const Args& a, u32 rank, u32 unit, u32 wv, u32 lane, u32& calls, char*& pws) {
  const u32 j = unit & 3, sub = (unit >> 2) & 1, cls = stage_class(unit >> 3);
  const bool isw = cls >= 16;
  const u32 ch = isw ? sub * 16 + (cls & 15) : (cls & 1) * 16 + sub * 8 + (cls >> 1);
  u32x4 d0, d1, d2, d3;
  if (isw) ld4(d0, d1, d2, d3, src_w(0, lane), src_w(1, lane), src_w(2, lane), src_w(3, lane), a.w + ((wv * 64 + j * 16) * 8192 + ch * 256));
  else ld4(d0, d1, d2, d3, src_x(0, lane), src_x(1, lane), src_x(2, lane), src_x(3, lane), a.x + ((ch * 128 + j * 32) * 1024 + wv * 128));
  char* db = a.ws[0] + ((isw ? OFF_SW : OFF_SX) + (ch * 8 + wv) * 16384 + j * 4096);
  const u32 vd = lane * 16;
  st16_sys(vd, d0, db); st16_sys(vd, d1, db + 1024); st16_sys(vd, d2, db + 2048); st16_sys(vd, d3, db + 3072);
  asm volatile("s_waitcnt vmcnt(0)\n\ts_barrier" : "+v"(calls), "+v"(pws) :: "memory");
  if (wv == 0 && lane < 8 && (lane & 3)) {
    const u32 p = rank ^ lane;
    const u32 pos = pos_of(isw ? cls : (sub ^ (p >> 2) ^ 1) * 8 + (cls >> 1));
    if ((isw ? p & 1 : (p >> 1) & 1) == ch >> 4 && pos >= NP0)
      __hip_atomic_store((GAS u32*)(pws + OFF_SFLAG + (lane * 4 + j) * 128) + pos, calls + 1, RLX, SYS);
  }
}

__device__ __forceinline__ void link_engine(const Args& a, u32 rank, u32 xl, u32 j, u32 wv, u32 lane, u32 seq, u64 t0,
                                            LAS char* shm, u32 lds0) {
  LAS u32* ctl = (LAS u32*)(shm + LDS_CTL);
  const bool pair = xl == 4;
  const u32 f0 = (lane >> 4) * 8192 + (lane & 15) * 16, vl = lane * 16;
  char* const ws = a.ws[0];
  volatile LAS u32x2* list = (volatile LAS u32x2*)shm;
  if (wv == 5) {
    const u32 isrc = OFF_SX + (rank >> 2) * 4 * 16384 + j * 4096;
    const u32 idst = OFF_XS + kb_first(xl) * 16384 + j * 4096;
    u32 gi = 0, ni = pair ? 32u : NP0, nf = pair ? RP : 0u;
    u64 left0 = pair ? ~0ull : 0ull, left1 = left0;
    bool pool = true;
#pragma unroll 1
    while (pool || nf < 128 || (left0 | left1)) {
      asm volatile("" ::: "memory");
      const u32x4 av = *(volatile LAS u32x4*)(ctl + C_AVAIL), pm = *(volatile LAS u32x4*)(ctl + C_AVAIL + 4);
      const u32 room = lds_get(ctl + C_TICKET) + POOL_AHEAD - gi;
      const u64 a0 = left0 & (((u64)__builtin_amdgcn_readfirstlane(pm.y) << 32) | (u32)__builtin_amdgcn_readfirstlane(pm.x));
      const u64 a1 = left1 & (((u64)__builtin_amdgcn_readfirstlane(pm.w) << 32) | (u32)__builtin_amdgcn_readfirstlane(pm.z));
      const bool near = room - 1 < POOL_AHEAD && room > POOL_AHEAD - SCHED_AHEAD;
      u32 src = 0, dst = 0, n = 0, step = 16 * 8192;
      if (a0 | a1) {
        if (near) {
          const u32 id = a0 ? __builtin_ctzll(a0) : 64 + __builtin_ctzll(a1);
          if (a0) left0 &= ~(1ull << id); else left1 &= ~(1ull << (id - 64));
          src = OFF_PO + id * 32768 + j * 8192; dst = OFF_PS + id * 32768 + j * 8192 + id * 2 + 1; n = 2; step = 4096;
        }
      } else if (ni < 32) {
        if (near && ni < __builtin_amdgcn_readfirstlane(av.x)) {
          const u32 pc = piece_at(ni);
          src = isrc + (pc >> 4 ? 4096 + gcol0(rank, pc & 15) : grow0(rank, pc)) * 1024;
          dst = idst + pc * 32 * 16384 + ni * 2 + 1; n = 4; step = 16384;
          ++ni;
        }
      } else if (nf < __builtin_amdgcn_readfirstlane(av.y)) {
        if (near) {
          src = lds_get(ctl + C_LIST + nf) + j * 32 * 8192; dst = src + (nf < RP ? 2 : 0); n = 2;
          ++nf;
        }
      } else if (pool && __builtin_amdgcn_readfirstlane(av.z) && room - 1 < POOL_AHEAD) {
        u32 t = 0;
        if (lane == 0) t = __hip_atomic_fetch_add((GAS u32*)(ws + OFF_POOL), 1u, RLX, AGT);
        t = __builtin_amdgcn_readfirstlane(t) - (seq - 1) * (NPOOL + 28);
        if (t < NPOOL) {
          const u32 ht = t < NBIG ? 2 * t : t + NBIG;
          if (pair) while (lds_get(ctl + C_AVAIL + 1) <= (ht >> 1)) __builtin_amdgcn_s_sleep(1);
          src = dst = lds_get(ctl + (pair ? C_LIST : C_LIST2) + (ht >> 1)) + (ht & 1) * 64 * 8192;
          n = t < NBIG ? 8 : 4;
        } else {
          pool = false;
        }
      }
      if (n) {
        if (lane < n) list[gi + lane] = (u32x2){src + lane * step, dst + lane * step};
        gi += n;
        lds_set(ctl + C_LISTED, gi);
      } else {
        __builtin_amdgcn_s_sleep(1);
      }
    }
    lds_set(ctl + C_TOTAL, gi + 1);
  } else if (wv == 0) {
    constexpr u32 BACK = (WIN + WINC3) / 4;
    const char* const pws = a.ws[xl];
    const char* const fsrc = a.out[xl];
    u32 slot = 0, pub = 0, gi = 0;
    __builtin_amdgcn_s_setprio(3);
#pragma unroll 1
    for (;; ++gi) {
      u32 src, lin;
      bool end = false;
      for (u32 tries = 0;; ++tries) {
        asm volatile("" ::: "memory");
        const u32x2 lt = *(volatile LAS u32x2*)(ctl + C_LISTED);
        const u32 fr = __hip_atomic_load(ctl + C_FREE + slot, RLX, WGP);
        asm volatile("" ::: "memory");
        const u32x2 d = list[gi];
        src = __builtin_amdgcn_readfirstlane(d.x); lin = __builtin_amdgcn_readfirstlane(d.y) & 1;
        if (__builtin_amdgcn_readfirstlane(lt.x) > gi && __builtin_amdgcn_readfirstlane(fr) + RING > gi) break;
        if (__builtin_amdgcn_readfirstlane(lt.y) == gi + 1) { end = true; break; }
        if (tries == 3) {
          WAIT_VM0(); lds_set(ctl + C_LANDED, gi); pub = gi;
        }
        __builtin_amdgcn_s_sleep(1);
      }
      if (end) break;
      if (lin) pull4l(vl, vl, vl, vl, pws + src, lds0 + RING0 + slot * 4096);
      else pull4l(f0, f0 + 31744, f0 + 63488, f0 + 95232, fsrc + src, lds0 + RING0 + slot * 4096);
      if (gi > pub + BACK) { pub = gi - BACK; lds_set(ctl + C_LANDED, pub); }
      lds_set(ctl + C_TICKET, gi + 1);
      slot = slot + 1 == RING ? 0 : slot + 1;
    }
    WAIT_VM0();
    lds_set(ctl + C_LANDED, gi);
  } else if (wv <= NST) {
    char* const fdst = a.out[0];
    u32 landed = 0, prev = 0;
    auto done = [&]() {
      if (prev & 1) {
        const u32 c = (prev >> 1) & 127;
        if (lds_add(ctl + C_CNT + c, lane) == 1 && pair && lane == 0)
          __hip_atomic_store((GAS u32*)(ws + OFF_PFLAG + c * 64) + j, seq, RLX, AGT);
      } else if (prev & 2) {
        if (lds_add(ctl + C_HELD, lane) == 2 * RP - 1 && lane == 0)
          __hip_atomic_store((GAS u32*)(a.ws[xl ^ 4] + OFF_HOLD + (xl ^ 4) * 128) + j, seq, RLX, SYS);
      }
      prev = 0;
    };
#pragma unroll 1
    for (;;) {
      const u32 gi = lds_add(ctl + C_TAKEN, lane);
      if (landed <= gi) {
        landed = lds_get(ctl + C_LANDED);
        if (landed <= gi) {
          WAIT_VM0();
          done();
          bool end = false;
          do {
            end = lds_get(ctl + C_TOTAL) - 1 <= gi;
            if (end) break;
            __builtin_amdgcn_s_sleep(1);
            landed = lds_get(ctl + C_LANDED);
          } while (landed <= gi);
          if (end) break;
        }
      }
      const u32 slot = gi % RING;
      const u32 desc = __builtin_amdgcn_readfirstlane(list[gi].y);
      const LAS u32x4* rs = (const LAS u32x4*)(shm + RING0 + slot * 4096) + lane;
      u32x4 d0 = rs[0];
      const u32x4 d1 = rs[64], d2 = rs[128], d3 = rs[192];
      lds_set(ctl + C_FREE + slot, gi + 1);
      if (lds_get(ctl + C_BAD)) d0.x = 0x7fc07fc0u;
      WAIT_VM0();
      if (desc & 1) {
        char* db = ws + (desc & ~4095u);
        st16_agent(vl, d0, db); st16_agent(vl, d1, db + 1024); st16_agent(vl, d2, db + 2048); st16_agent(vl, d3, db + 3072);
      } else {
        char* db = fdst + (desc & ~255u);
        st16_sys(f0, d0, db); st16_sys(f0, d1, db + 32768); st16_sys(f0, d2, db + 65536); st16_sys(f0, d3, db + 98304);
      }
      done();
      prev = desc;
    }
  } else if (wv == 6) {
    const u32 tag = seq & 0xffffff;
    const GAS u32* qf = (const GAS u32*)(ws + OFF_QUEUE + xl * 1024) + lane;
    const GAS u32* qs = pair ? qf + 128 : lane < 32 ? (const GAS u32*)(ws + OFF_SFLAG + (xl * 4 + j) * 128) + lane
                                                    : (const GAS u32*)(ws + OFF_HOLD + xl * 128) + (lane & 3);
    u32 nf = 0, open = 0;
    for (;;) {
      u32 e0 = __hip_atomic_load(qf, RLX, SYS), e1 = __hip_atomic_load(qf + 64, RLX, SYS), e2 = __hip_atomic_load(qs, RLX, SYS);
      u32 e3 = seq;
      if (pair) e3 = __hip_atomic_load(qs + 64, RLX, SYS);
      const bool late = expired(t0);
      if (late) { lds_set(ctl + C_BAD, 1); e0 = (seq << 8) | lane; e1 = (seq << 8) | (64 + lane); e2 = e3 = seq; }
      const u64 n0 = __ballot((e0 >> 8) != tag), n1 = __ballot((e1 >> 8) != tag);
      const u64 m2 = __ballot(e2 >= seq || (!pair && lane < NP0)), m3 = __ballot(e3 >= seq);
      const u32 cf = n0 ? (u32)__builtin_ctzll(n0) : (n1 ? 64u + (u32)__builtin_ctzll(n1) : 128u);
      const u32 staged = (u32)m2 == ~0u ? 32u : (u32)__builtin_ctz(~(u32)m2);
      if (cf != nf) {
        if (lane >= nf) ctl[C_LIST + lane] = tile_off(rank ^ xl, e0);
        if (64 + lane >= nf) ctl[C_LIST + 64 + lane] = tile_off(rank ^ xl, e1);
        nf = cf;
      }
      if (pair) {
        open = cf >= PAIR_OPEN;
        *(volatile LAS u32x4*)(ctl + C_AVAIL + 4) = (u32x4){(u32)m2, (u32)(m2 >> 32), (u32)m3, (u32)(m3 >> 32)};
      } else if (!open && (u32)(m2 >> 32) == ~0u) {
        const GAS u32* qp = (const GAS u32*)(ws + OFF_QUEUE + 4 * 1024) + lane;
        const u32 g0 = __hip_atomic_load(qp, RLX, SYS), g1 = __hip_atomic_load(qp + 64, RLX, SYS);
        ctl[C_LIST2 + lane] = tile_off(rank ^ 4, g0);
        ctl[C_LIST2 + 64 + lane] = tile_off(rank ^ 4, g1);
        open = late || !__ballot(((g0 >> 8) != tag && lane < RP) || ((g1 >> 8) != tag && 64 + lane < RP));
      }
      asm volatile("" ::: "memory");
      *(volatile LAS u32x4*)(ctl + C_AVAIL) = (u32x4){staged, cf, open, 0u};
      if (cf == 128 && open && (!pair || (m2 & m3) == ~0ull)) break;
      if (!pair && staged == 32) __builtin_amdgcn_s_sleep(8);
    }
  } else if (wv == 7 && !pair) {
    GAS u32* const flag = (GAS u32*)(ws + OFF_INFLAG + xl * 16 + j * 4);
    const GAS u32* lf = (const GAS u32*)(ws + OFF_SFLAG + (xl * 4 + j) * 128) + (lane & 31);
    while (__ballot(__hip_atomic_load(lf, RLX, SYS) < seq && lane < NP0)) {
      if (expired(t0)) { lds_set(ctl + C_BAD, 1); break; }
      __builtin_amdgcn_s_sleep(1);
    }
    if (lane < NP0) __hip_atomic_store(flag + lane * 64, seq, RLX, AGT);
#pragma unroll 1
    for (u32 pi = NP0; pi < 32; ++pi) {
      while (lds_get(ctl + C_CNT + pi) < 4) __builtin_amdgcn_s_sleep(1);
      if (lane == 0) __hip_atomic_store(flag + pi * 64, seq, RLX, AGT);
    }
  }
}

__device__ __forceinline__ void announcer(const Args& a, u32 wv, u32 lane, u32 seq, u64 t0) {
  const GAS u32* src = (const GAS u32*)(a.ws[0] + OFF_QUEUE + (wv ? 0 : 512)) + lane;
  GAS u32* dst = (GAS u32*)(a.ws[wv ? wv : 4] + OFF_QUEUE + (wv ? wv * 1024 : 4 * 1024 + 512)) + lane;
  const u32 sh = wv ? 8 : 0, tag = wv ? seq & 0xffffff : seq;
  bool s0 = false, s1 = false;
  while (__ballot(!s0 || !s1) && !expired(t0)) {
    const u32 e0 = __hip_atomic_load(src, RLX, AGT), e1 = __hip_atomic_load(src + 64, RLX, AGT);
    if (!s0 && (e0 >> sh) == tag) { __hip_atomic_store(dst, e0, RLX, SYS); s0 = true; }
    if (!s1 && (e1 >> sh) == tag) { __hip_atomic_store(dst + 64, e1, RLX, SYS); s1 = true; }
    __builtin_amdgcn_s_sleep(ANN_SLEEP);
  }
}

__device__ __forceinline__ u32 pack2(float lo, float hi) {
  bf16x2 v = {(__bf16)lo, (__bf16)hi};
  return __builtin_bit_cast(u32, v);
}
__device__ __forceinline__ float silu(float y) {
  const float c = __builtin_fminf(__builtin_fmaxf(y, -60.f), 60.f);
  return y * __builtin_amdgcn_rcpf(1.f + __builtin_amdgcn_exp2f(c * -1.44269504088896341f));
}

__global__ void __launch_bounds__(512, 2) fused_gemm_ar_silu(const Args a) {
  __shared__ __attribute__((aligned(1024))) char shm[LDS_BYTES];
  const u32 tid = threadIdx.x, lane = tid & 63;
  const u32 wv = __builtin_amdgcn_readfirstlane(tid >> 6);
  const u32 b = blockIdx.x, slot = b >> 3;
  const u32 xl = (a.xmap >> (4 * (b & 7))) & 7;
  LAS u32* lctr = (LAS u32*)(shm + LDS_CTL);
  const u32 lds0 = (u32)(size_t)(LAS char*)shm;
  const bool engine = xl != 0 && slot < 4;
  const u64 t0 = __builtin_amdgcn_s_memrealtime();
  const u32 rank = a.rank & 7;
  char* const ws = a.ws[0];
  GAS u32* ctr = (GAS u32*)(ws + OFF_CTR) + b;
  u32 calls = __hip_atomic_load(ctr, RLX, AGT);
  char* pws = a.ws[lane & 7];
  if (wv == 7) { lctr[lane] = 0; lctr[64 + lane] = 0; lctr[128 + lane] = 0; }
  const u32 lead = (slot - LEAD0) >> 2;
  const bool is_lead = (xl & 3) && wv < 4 && lead < NP0;
  if (is_lead) push_leading(a, rank ^ xl, xl, lead, slot & 3, wv & 3, lane, t0);
  if (!(xl & 3)) {
    if (slot >= 32 - STAGE_SPLIT) {
      for (u32 u = (slot - (32 - STAGE_SPLIT)) * 2 + (xl >> 2); u < STAGE_QUIET; u += 2 * STAGE_SPLIT) {
        stage_unit(a, rank, u, wv, lane, calls, pws);
        if (wv == 0) STAMP(u < 2 * STAGE_SPLIT ? 11 : 10);
      }
    }
  } else if (slot >= STAGE0) {
    while (__builtin_amdgcn_s_memrealtime() - t0 < STAGE_DELAY) __builtin_amdgcn_s_sleep(2);
    for (u32 u = STAGE_QUIET + (slot - STAGE0) * 6 + (xl < 4 ? xl - 1 : xl - 2); u < 256; u += (32 - STAGE0) * 6) {
      stage_unit(a, rank, u, wv, lane, calls, pws);
      if (wv == 0) STAMP(10);
    }
  }
  asm volatile("" : "+v"(calls));
  const u32 seq = __builtin_amdgcn_readfirstlane(calls) + 1;
  WG_BARRIER();
  if (is_lead && tid == 0)
    __hip_atomic_store((GAS u32*)(a.ws[xl] + OFF_SFLAG + (xl * 4 + (slot & 3)) * 128) + lead, seq, RLX, SYS);
  if (is_lead && wv == 0) STAMP(11);
  if (tid == 0) __hip_atomic_store(ctr, seq, RLX, AGT);
  if (engine) {
    link_engine(a, rank, xl, slot, wv, lane, seq, t0, (LAS char*)shm, lds0);
    return;
  }

  if (xl == 0 && slot == 0) {
    if (wv || ANN == 2) announcer(a, wv, lane, seq, t0);
    return;
  }
#pragma unroll 1
  for (u32 ts = slot;; ts -= 4) {
  const u32 te = a.tile[xl * 32 + ts];
  const u32 lr = te & 15, lc = (te >> 4) & 15, rpos = (te >> 8) & 31, tix = (te >> 13) & 127;
  const u32 kind = lr >> 3;
  const u32 row0 = grow0(rank, lr), col0 = gcol0(rank, lc);
  const u32 vl = lane * 16;

  if (wv >= 4) {
    const u32 w4 = wv - 4;
    const char* xo = a.x + (size_t)(row0 + w4 * 32) * 1024;
    const char* wo = a.w + ((size_t)w4 * 16 * 8192 + col0 * 2);
    const char* xs = ws + (OFF_XS + (size_t)lr * 32 * 16384 + w4 * 4096);
    const char* wsg = ws + (OFF_WS + (size_t)lc * 32 * 16384 + w4 * 4096);
    const u32 lq = lds0 + w4 * 4096;
    const GAS u32x4* inflag = (const GAS u32x4*)(ws + OFF_INFLAG + rpos * 256 + (lane & 7) * 16);
    const GAS u32x4* lastflag = (lane & 7) ? inflag : (const GAS u32x4*)(ws + OFF_PFLAG + tix * 64);
    { const u32 d = START_DELAY + rpos * START_SLOPE; while (__builtin_amdgcn_s_memrealtime() - t0 < d) __builtin_amdgcn_s_sleep(4); }
    if (wv == 7 && ts == slot) {
      __builtin_amdgcn_fence(__ATOMIC_ACQUIRE, "");
      WAIT_VM0();
    }
    __builtin_amdgcn_s_setprio(LOADER_PRIO);
    {
      const u32 x0 = src_x(0, lane), x1 = src_x(1, lane), x2 = src_x(2, lane), x3 = src_x(3, lane);
      const u32 y0 = src_w(0, lane), y1 = src_w(1, lane), y2 = src_w(2, lane), y3 = src_w(3, lane);
#pragma unroll
      for (u32 kb = 0; kb < 8; ++kb) {
        if (kb >= 3) asm volatile("s_waitcnt vmcnt(16)\n\ts_barrier" ::: "memory");
        glds4v(x0, x1, x2, x3, xo + kb * 128, lq + (kb & 3) * STAGE_BYTES);
        glds4v(y0, y1, y2, y3, wo + (size_t)kb * 64 * 8192, lq + (kb & 3) * STAGE_BYTES + 16384);
      }
    }
    const u32 nst = 32 + kind;
    u32 avail = 8, issued = 8, ord = 0, seen = 0;
    auto refresh = [&]() {
      if (w4 == 0) {
        u32x4 f;
        const GAS u32x4* fp = issued >= 20 ? lastflag : inflag;
        asm volatile("global_load_dwordx4 %0, %1, off sc0 sc1\n\ts_waitcnt vmcnt(0)" : "=&v"(f) : "v"(fp) : "memory");
        u32 mask = (u32)__ballot(f.x >= seq && f.y >= seq && f.z >= seq && f.w >= seq);
        if (expired(t0)) { lds_set(lctr + C_BAD, 1); mask = 0xff; }
        ord |= (mask & 1) << 21;
        for (u32 m = mask & 0xee & ~seen; m; m &= m - 1) {
          const u32 l = __builtin_ctz(m);
          ord = (ord | ((l < 4 ? l + 1 : l) << (3 * ((ord >> 18) & 7)))) + (1u << 18);
        }
        seen |= mask;
        lds_set(lctr + C_MASK, ord);
      } else {
        ord = lds_get(lctr + C_MASK);
      }
      const u32 n = (ord >> 18) & 7;
      avail = 8 + 4 * n + (n == 6 ? (ord >> 21) & kind : 0);
    };
#pragma unroll 1
    for (u32 s = 5; s <= 32; ++s) {
      while (issued <= s + 2) {
        if (issued >= avail) {
          if (issued >= nst) break;
          refresh();
          while (issued >= avail && issued == s) {
            if (issued == 32) __builtin_amdgcn_s_sleep(PPOLL_SLEEP); else __builtin_amdgcn_s_sleep(POLL_SLEEP);
            refresh();
          }
          if (issued >= avail) break;
        }
        const u32 l = lq + (issued & 3) * STAGE_BYTES;
        const u32 kk = ((ord >> (3 * ((issued - 8) >> 2))) & 7) * 4 + (issued & 3);
        const char* sx = xs + kk * 16384;
        const char* sw = wsg + kk * 16384;
        if (issued == 32) { sx = ws + (OFF_PS + (size_t)tix * 32768 + w4 * 4096); sw = sx + 16384; }
        glds4v(vl, vl, vl, vl, sx, l);
        glds4v(vl, vl, vl, vl, sw, l + 16384);
        ++issued;
      }
      const u32 ahead = issued - s;
      if (ahead >= 3) asm volatile("s_waitcnt vmcnt(16)\n\ts_barrier" ::: "memory");
      else if (ahead == 2) asm volatile("s_waitcnt vmcnt(8)\n\ts_barrier" ::: "memory");
      else asm volatile("s_waitcnt vmcnt(0) lgkmcnt(0)\n\ts_barrier" ::: "memory");
    }
    __builtin_amdgcn_s_setprio(0);
    u32 pos = 0;
    if (kind && wv == 4 && lane == 0) pos = __hip_atomic_fetch_add((GAS u32*)(ws + OFF_TICK), 1u, RLX, AGT);
    pos = __builtin_amdgcn_readfirstlane(pos);
    if (wv == 4) lds_set(lctr + C_MASK, 0);
    WG_BARRIER();
    if (wv == 4) {
      const u32 e = (seq << 8) | ((lr & 7) * 16 + lc);
      if (kind) {
        if (lane == 0) __hip_atomic_store((GAS u32*)(ws + OFF_QUEUE) + (pos & 127), e, RLX, AGT);
      } else if (lane == 0) {
        __hip_atomic_store((GAS u32*)(a.ws[4] + OFF_QUEUE + 4 * 1024 + 512) + tix, seq, RLX, SYS);
      }
      TSTAMP(7);
    }
  } else {
    const u32 wn = wv & 1, wm = wv >> 1;
    const u32 g = lane >> 4, q = (lane >> 2) & 3, pp = lane & 3;
    u32 ax[2];
#pragma unroll
    for (int ks = 0; ks < 2; ++ks) ax[ks] = (wm * 64 + (lane & 15)) * 128 + (((4 * ks + g) ^ ((lane & 15) >> 1)) * 16);
    u32 aw[4];
#pragma unroll
    for (int nt = 0; nt < 4; ++nt)
      aw[nt] = 16384 + (8 * g + q) * 256 + (((2 * (wn * 4 + nt) + (pp >> 1)) ^ ((q | ((g & 1) << 2)) << 1)) * 16) + (pp & 1) * 8;

    f32x4 acc[4][4];
#pragma unroll
    for (int i = 0; i < 4; ++i)
#pragma unroll
      for (int jj = 0; jj < 4; ++jj) acc[i][jj] = (f32x4){0.f, 0.f, 0.f, 0.f};
    bf16x8 xa[4], wa[4], xb[4], wb[4];
    auto rd = [&](bf16x8* fx, bf16x8* fw, u32 t, int ks) {
      const LAS char* st = (const LAS char*)shm + (t & (NSTAGE - 1)) * STAGE_BYTES;
#pragma unroll
      for (int i = 0; i < 4; ++i) {
        const bf16x4 lo = __builtin_amdgcn_ds_read_tr16_b64_v4bf16((LAS bf16x4*)(st + aw[i] + ks * 8192));
        const bf16x4 hi = __builtin_amdgcn_ds_read_tr16_b64_v4bf16((LAS bf16x4*)(st + aw[i] + ks * 8192 + 1024));
        fw[i] = __builtin_shufflevector(lo, hi, 0, 1, 2, 3, 4, 5, 6, 7);
        fx[i] = *(const LAS bf16x8*)(st + ax[ks] + i * 2048);
      }
    };
    auto mm = [&](const bf16x8* fx, const bf16x8* fw) {
#pragma unroll
      for (int mt = 0; mt < 4; ++mt)
#pragma unroll
        for (int nt = 0; nt < 4; ++nt)
          acc[nt][mt] = __builtin_amdgcn_mfma_f32_16x16x32_bf16(fw[nt], fx[mt], acc[nt][mt], 0, 0, 0);
    };
#define INTERLEAVE() do { _Pragma("unroll") for (int i_ = 0; i_ < 12; ++i_) { \
      __builtin_amdgcn_sched_group_barrier(0x8, 1, 0); __builtin_amdgcn_sched_group_barrier(0x100, 1, 0); } \
      __builtin_amdgcn_sched_group_barrier(0x8, 4, 0); } while (0)
    WG_BARRIER();
    rd(xa, wa, 0, 0);
    __builtin_amdgcn_sched_barrier(0);
    for (u32 t = 0; t < 31; ++t) {
      rd(xb, wb, t, 1);
      mm(xa, wa);
      INTERLEAVE();
      __builtin_amdgcn_sched_barrier(0);
      WG_BARRIER();
      __builtin_amdgcn_sched_barrier(0);
      rd(xa, wa, t + 1, 0);
      mm(xb, wb);
      INTERLEAVE();
      __builtin_amdgcn_sched_barrier(0);
    }
    rd(xb, wb, 31, 1);
    mm(xa, wa);
    INTERLEAVE();
    __builtin_amdgcn_sched_barrier(0);
    mm(xb, wb);
    if (wv == 0) TSTAMP(5);
    WG_BARRIER();
    const bool bad = lds_get(lctr + C_BAD) != 0;
    const float qnan = __builtin_nanf("");
    if (kind == 0) {
      char* db = ws + OFF_PO + (size_t)tix * 32768 + wv * 8192;
#pragma unroll
      for (int mt = 0; mt < 4; ++mt)
#pragma unroll
        for (int np = 0; np < 2; ++np) {
          f32x4 u = acc[2 * np][mt], v = acc[2 * np + 1][mt];
          if (bad) u[0] = qnan;
          const u32x4 o = {pack2(u[0], u[1]), pack2(u[2], u[3]), pack2(v[0], v[1]), pack2(v[2], v[3])};
          st16_sys(vl, o, db + (mt * 2 + np) * 1024);
        }
    } else {
      const LAS u32x4* pr = (const LAS u32x4*)(shm + (32 & (NSTAGE - 1)) * STAGE_BYTES + wv * 8192) + lane;
      char* ob = a.out[0] + (size_t)(row0 + wm * 64) * 8192 + (col0 + wn * 64) * 2;
      const u32 vo = (lane & 15) * 8192 + (g & 1) * 32 + (g >> 1) * 16;
#pragma unroll
      for (int mt = 0; mt < 4; ++mt)
#pragma unroll
        for (int np = 0; np < 2; ++np) {
          const u32x4 pv = pr[(mt * 2 + np) * 64];
          f32x4 u = acc[2 * np][mt], v = acc[2 * np + 1][mt];
          u[0] += __uint_as_float(pv.x << 16); u[1] += __uint_as_float(pv.x & 0xffff0000u);
          u[2] += __uint_as_float(pv.y << 16); u[3] += __uint_as_float(pv.y & 0xffff0000u);
          v[0] += __uint_as_float(pv.z << 16); v[1] += __uint_as_float(pv.z & 0xffff0000u);
          v[2] += __uint_as_float(pv.w << 16); v[3] += __uint_as_float(pv.w & 0xffff0000u);
          if (bad) u[0] = qnan;
          const u32 a0 = pack2(silu(u[0]), silu(u[1])), a1 = pack2(silu(u[2]), silu(u[3]));
          const u32 b0 = pack2(silu(v[0]), silu(v[1])), b1 = pack2(silu(v[2]), silu(v[3]));
          const auto s0 = __builtin_amdgcn_permlane16_swap(a0, b0, false, false);
          const auto s1 = __builtin_amdgcn_permlane16_swap(a1, b1, false, false);
          const u32x4 o = {s0[0], s1[0], s0[1], s1[1]};
          st16_sys(vo, o, ob + (size_t)mt * 16 * 8192 + np * 64);
        }
    }
    WAIT_VM0();
    WG_BARRIER();
  }

  if (ts < 4 || ts >= (xl ? 8u : ANN ? 5u : 4u)) break;
  }
}

static void build_tables(Args& a) {
  int pos[32];
  for (int i = 0; i < 32; ++i) pos[PIECES[i]] = i;
  auto rdy = [&](int lr, int lc) { return pos[lr] > pos[16 + lc] ? pos[lr] : pos[16 + lc]; };
  int prank[128];
  for (int t = 0; t < 128; ++t) {
    const int kt = (rdy(8 + (t >> 4), t & 15) * 32 + rdy(t >> 4, t & 15)) * 128 + t;
    prank[t] = 0;
    for (int u = 0; u < 128; ++u) prank[t] += (rdy(8 + (u >> 4), u & 15) * 32 + rdy(u >> 4, u & 15)) * 128 + u < kt;
  }
  for (int xl = 0; xl < 8; ++xl) {
    int key[32], idx[32], ent[32];
    for (int t = 0; t < 32; ++t) {
      const int lr = (t >> 2) * 2 + (xl >> 2), lc = (t & 3) * 4 + (xl & 3), pr = prank[(lr & 7) * 16 + lc];
      key[t] = lr < 8 ? rdy(lr, lc) * 128 + pr : 10000 + pr;
      ent[t] = lr | (lc << 4) | (rdy(lr, lc) << 8) | (pr << 13);
      idx[t] = t;
    }
    for (int i = 1; i < 32; ++i)
      for (int j = i; j > 0 && key[idx[j]] < key[idx[j - 1]]; --j) { int s = idx[j]; idx[j] = idx[j - 1]; idx[j - 1] = s; }
    for (int slot = 0; slot < 32; ++slot) a.tile[xl * 32 + slot] = ent[idx[slot < 4 ? 28 + slot : slot - 4]];
  }
}

extern "C" void kernel_launch(int rank, int nranks, ncclComm_t comm, hipStream_t stream,
                              void* const* d_in, const int* in_sizes, int n_in,
                              void* d_out, int out_size, const int* in_dtypes, int out_dtype,
                              void* const* d_out_all, void* const* d_ws_all, size_t ws_bytes) {
  if (nranks != 8 || (unsigned)rank >= 8u || n_in < 2 || ws_bytes < WS_NEED || out_size != 4096 * 4096 ||
      in_sizes[0] != 4096 * 512 || in_sizes[1] != 512 * 4096) {
    fprintf(stderr, "kernel_launch: unexpected problem shape\n");
    return;
  }
  Args a;
  a.x = (const char*)d_in[0];
  a.w = (const char*)d_in[1];
  for (int l = 0; l < 8; ++l) { a.out[l] = (char*)d_out_all[rank ^ l]; a.ws[l] = (char*)d_ws_all[rank ^ l]; }
  a.rank = (u32)rank;
  static const unsigned char near_xcd[8][8] = {{0, 4, 2, 3, 6, 5, 1, 7}, {4, 0, 7, 5, 2, 3, 6, 1}, {2, 7, 0, 1, 4, 5, 3, 6}, {3, 5, 1, 0, 7, 4, 2, 6},
                                               {5, 2, 4, 7, 0, 1, 6, 3}, {6, 3, 5, 4, 1, 0, 7, 2}, {1, 6, 3, 2, 5, 7, 0, 4}, {7, 1, 6, 5, 3, 2, 4, 0}};
  a.xmap = 0;
  for (int p = 0; p < 8; ++p) a.xmap |= (u32)(rank ^ p) << (4 * near_xcd[rank][p]);
  build_tables(a);
  fused_gemm_ar_silu<<<256, 512, 0, stream>>>(a);
  hipError_t e = hipGetLastError();
  if (e != hipSuccess) fprintf(stderr, "launch failed: %s\n", hipGetErrorString(e));
}
```

```cpp
#include <hip/hip_runtime.h>
#include <rccl/rccl.h>
#include <cstdio>
#include <cstdint>

typedef unsigned int u32;
typedef unsigned long long u64;
typedef __attribute__((ext_vector_type(4))) float f32x4;
typedef __attribute__((ext_vector_type(4))) unsigned int u32x4;
typedef __attribute__((ext_vector_type(2))) unsigned int u32x2;
typedef __attribute__((ext_vector_type(4))) __bf16 bf16x4;
typedef __attribute__((ext_vector_type(8))) __bf16 bf16x8;
typedef __attribute__((ext_vector_type(2))) __bf16 bf16x2;
#define LAS __attribute__((address_space(3)))
#define GAS __attribute__((address_space(1)))
#define RLX __ATOMIC_RELAXED
#define SYS __HIP_MEMORY_SCOPE_SYSTEM
#define AGT __HIP_MEMORY_SCOPE_AGENT
#define WGP __HIP_MEMORY_SCOPE_WORKGROUP

constexpr size_t OFF_CTR = 0;
constexpr size_t OFF_TICK = 5120;
constexpr size_t OFF_POOL = 5376;
constexpr size_t OFF_HOLD = 6144;
constexpr size_t OFF_INFLAG = 8192;
constexpr size_t OFF_PFLAG = 16384;
constexpr size_t OFF_QUEUE = 32768;
constexpr size_t OFF_SFLAG = 45056;
constexpr size_t OFF_XS = 1u << 20;
constexpr size_t OFF_WS = 9u << 20;
constexpr size_t OFF_PS = 17u << 20;
constexpr size_t OFF_PO = 21u << 20;
constexpr size_t OFF_SX = 25u << 20;
constexpr size_t OFF_SW = 29u << 20;
constexpr size_t WS_NEED = 33u << 20;

constexpr u64 DEADLINE = 100000000ull;
constexpr int NSTAGE = 4;
constexpr int STAGE_BYTES = 32768;
constexpr int LDS_CTL = NSTAGE * STAGE_BYTES;
constexpr int LDS_BYTES = LDS_CTL + 2048;

#define PIECE_ORDER 0, 16, 1, 17, 2, 18, 3, 19, 8, 20, 21, 9, 22, 23, 10, 24, 25, 11, 26, 27, 4, 28, 29, 30, 31, 5, 12, 6, 13, 7, 14, 15
constexpr u32 PIECES[32] = {PIECE_ORDER};
constexpr u64 pack_pieces(int first) {
  u64 v = 0;
  for (int i = 0; i < 8; ++i) v |= (u64)PIECES[first + i] << (8 * i);
  return v;
}
__device__ __forceinline__ u32 piece_at(u32 i) {
  constexpr u64 p0 = pack_pieces(0), p1 = pack_pieces(8), p2 = pack_pieces(16), p3 = pack_pieces(24);
  const u64 v = i < 16 ? (i < 8 ? p0 : p1) : (i < 24 ? p2 : p3);
  return (u32)(v >> (8 * (i & 7))) & 31u;
}

constexpr u64 pack_pos(int first) {
  u64 v = 0;
  for (int i = 0; i < 32; ++i) if ((int)PIECES[i] >= first && (int)PIECES[i] < first + 8) v |= (u64)i << (8 * (PIECES[i] - first));
  return v;
}
__device__ __forceinline__ u32 pos_of(u32 c) {
  constexpr u64 p0 = pack_pos(0), p1 = pack_pos(8), p2 = pack_pos(16), p3 = pack_pos(24);
  const u64 v = c < 16 ? (c < 8 ? p0 : p1) : (c < 24 ? p2 : p3);
  return (u32)(v >> (8 * (c & 7))) & 31u;
}
#define NP0 3
#define RP 100
#define ANN_SLEEP 32
#define PAIR_OPEN 64
#define FINE 28
constexpr u32 NBIG = RP - FINE, NPOOL = NBIG + 2 * FINE;
struct StageOrder { u64 v[4]; };
constexpr StageOrder make_stage_order() {
  StageOrder o = {{0, 0, 0, 0}};
  bool seen[24] = {};
  int n = 0;
  for (int k = NP0; k < 32 + NP0; ++k) {
    const int c = (int)PIECES[k & 31], cl = c < 16 ? (c & 7) : c - 8;
    if (seen[cl]) continue;
    seen[cl] = true;
    for (int e = 0; e < (c < 16 ? 2 : 1); ++e, ++n) o.v[n >> 3] |= (u64)(c < 16 ? cl * 2 + e : c) << (8 * (n & 7));
  }
  return o;
}
__device__ __forceinline__ u32 stage_class(u32 i) {
  constexpr StageOrder o = make_stage_order();
  constexpr u64 p0 = o.v[0], p1 = o.v[1], p2 = o.v[2], p3 = o.v[3];
  const u64 v = i < 16 ? (i < 8 ? p0 : p1) : (i < 24 ? p2 : p3);
  return (u32)(v >> (8 * (i & 7))) & 31u;
}

struct Args {
  const char* x;
  const char* w;
  u32 rank;
  u32 xmap;
  char* ws[8];
  char* out[8];
  u32 tile[256];
};

__device__ __forceinline__ u32 grow0(u32 r, u32 lr) {
  return ((r >> 1) & 1) * 2048 + ((lr >> 3) ^ (r >> 2) ^ 1) * 1024 + (lr & 7) * 128;
}
__device__ __forceinline__ u32 gcol0(u32 r, u32 lc) { return (r & 1) * 2048 + lc * 128; }
__device__ __forceinline__ u32 tile_off(u32 r, u32 id) {
  return grow0(r, 8 + ((id >> 4) & 7)) * 8192 + gcol0(r, id & 15) * 2;
}

__device__ __forceinline__ void st16_sys(u32 voff, u32x4 v, char* sbase) {
  asm volatile("global_store_dwordx4 %0, %1, %2 sc0 sc1\n\ts_nop 1" :: "v"(voff), "v"(v), "s"(sbase) : "memory");
}
__device__ __forceinline__ void st16_agent(u32 voff, u32x4 v, char* sbase) {
  asm volatile("global_store_dwordx4 %0, %1, %2 sc1\n\ts_nop 1" :: "v"(voff), "v"(v), "s"(sbase) : "memory");
}
__device__ __forceinline__ bool expired(u64 t0) { return __builtin_amdgcn_s_memrealtime() - t0 > DEADLINE; }
#define WAIT_VM0() asm volatile("s_waitcnt vmcnt(0)" ::: "memory")
#define WG_BARRIER() asm volatile("s_waitcnt lgkmcnt(0)\n\ts_barrier" ::: "memory")

#define WIN 8
#define NST 4
constexpr u32 RING = 24, RING0 = 16384;
#define AHEAD 7
#define START_DELAY 100
#define START_SLOPE 100
#define LEAD0 4
#define STAGE0 (LEAD0 + 4 * NP0)
#define STAGE_SPLIT 24
#define STAGE_DELAY 80
constexpr u32 C_CNT = 0;
constexpr u32 C_AVAIL = 128;
constexpr u32 C_LISTED = 136, C_TOTAL = 137, C_TICKET = 138, C_BAD = 139, C_MASK = 140, C_LANDED = 141, C_TAKEN = 142, C_HELD = 143;
constexpr u32 C_FREE = 160;
constexpr u32 C_LIST = 192, C_LIST2 = 320;

__device__ __forceinline__ void glds4v(u32 v0, u32 v1, u32 v2, u32 v3, const char* sbase, u32 lds) {
  asm volatile(
      "s_mov_b32 m0, %5\n\ts_nop 0\n\t"
      "global_load_lds_dwordx4 %0, %4\n\t"
      "global_load_lds_dwordx4 %1, %4 offset:1024\n\t"
      "global_load_lds_dwordx4 %2, %4 offset:2048\n\t"
      "global_load_lds_dwordx4 %3, %4 offset:3072"
      :: "v"(v0), "v"(v1), "v"(v2), "v"(v3), "s"(sbase), "s"(lds) : "memory");
}
__device__ __forceinline__ void pull4l(u32 v0, u32 v1, u32 v2, u32 v3, const char* sbase, u32 lds) {
  asm volatile(
      "s_mov_b32 m0, %5\n\ts_waitcnt vmcnt(%6)\n\t"
      "global_load_lds_dwordx4 %0, %4 sc0 sc1\n\t"
      "global_load_lds_dwordx4 %1, %4 offset:1024 sc0 sc1\n\t"
      "global_load_lds_dwordx4 %2, %4 offset:2048 sc0 sc1\n\t"
      "global_load_lds_dwordx4 %3, %4 offset:3072 sc0 sc1"
      :: "v"(v0), "v"(v1), "v"(v2), "v"(v3), "s"(sbase), "s"(lds), "n"(WIN) : "memory");
}
__device__ __forceinline__ u32 src_x(u32 s, u32 lane) {
  return (8 * s + (lane >> 3)) * 1024 + (((lane & 7) ^ (4 * (s & 1) + (lane >> 4))) * 16) - s * 1024;
}
__device__ __forceinline__ u32 src_w(u32 s, u32 lane) {
  return (4 * s + (lane >> 4)) * 8192 + (((lane & 15) ^ (((lane >> 4) | ((s >> 1) << 2)) << 1)) * 16) - s * 1024;
}
__device__ __forceinline__ u32 lds_get(LAS u32* p) {
  asm volatile("" ::: "memory");
  const u32 v = __builtin_amdgcn_readfirstlane(__hip_atomic_load(p, RLX, WGP));
  asm volatile("" ::: "memory");
  return v;
}
__device__ __forceinline__ void lds_set(LAS u32* p, u32 v) {
  asm volatile("" ::: "memory");
  __hip_atomic_store(p, v, RLX, WGP);
  asm volatile("" ::: "memory");
}
__device__ __forceinline__ u32 lds_add(LAS u32* p, u32 lane) {
  u32 t = 0;
  if (lane == 0) t = __hip_atomic_fetch_add(p, 1u, RLX, WGP);
  return __builtin_amdgcn_readfirstlane(t);
}

__device__ __forceinline__ void ld4(u32x4& d0, u32x4& d1, u32x4& d2, u32x4& d3, u32 v0, u32 v1, u32 v2, u32 v3, const char* sb) {
  asm volatile("global_load_dwordx4 %0, %4, %8\n\tglobal_load_dwordx4 %1, %5, %8 offset:1024\n\t"
               "global_load_dwordx4 %2, %6, %8 offset:2048\n\tglobal_load_dwordx4 %3, %7, %8 offset:3072\n\ts_waitcnt vmcnt(0)"
               : "=&v"(d0), "=&v"(d1), "=&v"(d2), "=&v"(d3) : "v"(v0), "v"(v1), "v"(v2), "v"(v3), "s"(sb) : "memory");
}
__device__ __forceinline__ u32 kb_first(u32 xl) { return xl < 4 ? 4 + 4 * xl : 4 * xl; }

__device__ __forceinline__ void push_leading(const Args& a, u32 p, u32 xl, u32 i, u32 j, u32 kb, u32 lane) {
  const u32 pc = piece_at(i), k0 = (p >> 2) * 256 + kb * 64;
  u32x4 d0, d1, d2, d3;
  if (pc >> 4) ld4(d0, d1, d2, d3, src_w(0, lane), src_w(1, lane), src_w(2, lane), src_w(3, lane),
                   a.w + ((k0 + j * 16) * 8192 + gcol0(p, pc & 15) * 2));
  else ld4(d0, d1, d2, d3, src_x(0, lane), src_x(1, lane), src_x(2, lane), src_x(3, lane),
           a.x + ((grow0(p, pc) + j * 32) * 1024 + k0 * 2));
  char* db = a.ws[xl] + (OFF_XS + (pc * 32 + kb_first(xl) + kb) * 16384 + j * 4096);
  const u32 vd = lane * 16;
  st16_sys(vd, d0, db); st16_sys(vd, d1, db + 1024); st16_sys(vd, d2, db + 2048); st16_sys(vd, d3, db + 3072);
  WAIT_VM0();
}

__device__ __forceinline__ void stage_unit(const Args& a, u32 rank, u32 unit, u32 wv, u32 lane, u32& calls, char*& pws) {
  const u32 j = unit & 3, sub = (unit >> 2) & 1, cls = stage_class(unit >> 3);
  const bool isw = cls >= 16;
  const u32 ch = isw ? sub * 16 + (cls & 15) : (cls & 1) * 16 + sub * 8 + (cls >> 1);
  u32x4 d0, d1, d2, d3;
  if (isw) ld4(d0, d1, d2, d3, src_w(0, lane), src_w(1, lane), src_w(2, lane), src_w(3, lane), a.w + ((wv * 64 + j * 16) * 8192 + ch * 256));
  else ld4(d0, d1, d2, d3, src_x(0, lane), src_x(1, lane), src_x(2, lane), src_x(3, lane), a.x + ((ch * 128 + j * 32) * 1024 + wv * 128));
  char* db = a.ws[0] + ((isw ? OFF_SW : OFF_SX) + (ch * 8 + wv) * 16384 + j * 4096);
  const u32 vd = lane * 16;
  st16_sys(vd, d0, db); st16_sys(vd, d1, db + 1024); st16_sys(vd, d2, db + 2048); st16_sys(vd, d3, db + 3072);
  asm volatile("s_waitcnt vmcnt(0)\n\ts_barrier" : "+v"(calls), "+v"(pws) :: "memory");
  if (wv == 0 && lane < 8 && (lane & 3)) {
    const u32 p = rank ^ lane;
    const u32 pos = pos_of(isw ? cls : (sub ^ (p >> 2) ^ 1) * 8 + (cls >> 1));
    if ((isw ? p & 1 : (p >> 1) & 1) == ch >> 4 && pos >= NP0)
      __hip_atomic_store((GAS u32*)(pws + OFF_SFLAG + (lane * 4 + j) * 128) + pos, calls + 1, RLX, SYS);
  }
}

__device__ __forceinline__ void link_engine(const Args& a, u32 rank, u32 xl, u32 j, u32 wv, u32 lane, u32 seq, u64 t0,
                                            LAS char* shm, u32 lds0) {
  LAS u32* ctl = (LAS u32*)(shm + LDS_CTL);
  const bool pair = xl == 4;
  const u32 f0 = (lane >> 4) * 8192 + (lane & 15) * 16, vl = lane * 16;
  char* const ws = a.ws[0];
  volatile LAS u32x2* list = (volatile LAS u32x2*)shm;
  if (wv == 5) {
    const u32 isrc = OFF_SX + (rank >> 2) * 4 * 16384 + j * 4096;
    const u32 idst = OFF_XS + kb_first(xl) * 16384 + j * 4096;
    u32 gi = 0, ni = pair ? 32u : NP0, nf = pair ? RP : 0u;
    u64 left0 = pair ? ~0ull : 0ull, left1 = left0;
    bool pool = true;
#pragma unroll 1
    while (pool || nf < 128 || (left0 | left1)) {
      asm volatile("" ::: "memory");
      const u32x4 av = *(volatile LAS u32x4*)(ctl + C_AVAIL), pm = *(volatile LAS u32x4*)(ctl + C_AVAIL + 4);
      const bool near = gi < lds_get(ctl + C_TICKET) + AHEAD;
      const u64 a0 = left0 & (((u64)__builtin_amdgcn_readfirstlane(pm.y) << 32) | (u32)__builtin_amdgcn_readfirstlane(pm.x));
      const u64 a1 = left1 & (((u64)__builtin_amdgcn_readfirstlane(pm.w) << 32) | (u32)__builtin_amdgcn_readfirstlane(pm.z));
      const u32 cf = __builtin_amdgcn_readfirstlane(av.y);
      u32 src = 0, dst = 0, n = 0, step = 16 * 8192;
      if (!near) {
      } else if (a0 | a1) {
        const u32 id = a0 ? __builtin_ctzll(a0) : 64 + __builtin_ctzll(a1);
        if (a0) left0 &= ~(1ull << id); else left1 &= ~(1ull << (id - 64));
        src = OFF_PO + id * 32768 + j * 8192; dst = OFF_PS + id * 32768 + j * 8192 + id * 2 + 1; n = 2; step = 4096;
      } else if (ni < 32) {
        if (ni < __builtin_amdgcn_readfirstlane(av.x)) {
          const u32 pc = piece_at(ni);
          src = isrc + (pc >> 4 ? 4096 + gcol0(rank, pc & 15) : grow0(rank, pc)) * 1024;
          dst = idst + pc * 32 * 16384 + ni * 2 + 1; n = 4; step = 16384;
          ++ni;
        }
      } else if (nf < cf) {
        src = lds_get(ctl + C_LIST + nf) + j * 32 * 8192; dst = src + (nf < RP ? 2 : 0); n = 2;
        ++nf;
      } else if (pool && __builtin_amdgcn_readfirstlane(av.z) && (!(left0 | left1) || cf >= RP)) {
        u32 t = 0;
        if (lane == 0) t = __hip_atomic_fetch_add((GAS u32*)(ws + OFF_POOL), 1u, RLX, AGT);
        t = __builtin_amdgcn_readfirstlane(t) - (seq - 1) * (NPOOL + 28);
        if (t < NPOOL) {
          const u32 ht = t < NBIG ? 2 * t : t + NBIG;
          if (pair) while (lds_get(ctl + C_AVAIL + 1) <= (ht >> 1)) __builtin_amdgcn_s_sleep(1);
          src = dst = lds_get(ctl + (pair ? C_LIST : C_LIST2) + (ht >> 1)) + (ht & 1) * 64 * 8192;
          n = t < NBIG ? 8 : 4;
        } else {
          pool = false;
        }
      }
      if (n) {
        if (lane < n) list[gi + lane] = (u32x2){src + lane * step, dst + lane * step};
        gi += n;
        lds_set(ctl + C_LISTED, gi);
      } else {
        __builtin_amdgcn_s_sleep(1);
      }
    }
    lds_set(ctl + C_TOTAL, gi + 1);
  } else if (wv == 0) {
    constexpr u32 BACK = (WIN + 3) / 4;
    const char* const pws = a.ws[xl];
    const char* const fsrc = a.out[xl];
    u32 slot = 0, pub = 0, gi = 0;
    __builtin_amdgcn_s_setprio(3);
#pragma unroll 1
    for (;; ++gi) {
      u32 src, lin;
      bool end = false;
      for (u32 tries = 0;; ++tries) {
        asm volatile("" ::: "memory");
        const u32x2 lt = *(volatile LAS u32x2*)(ctl + C_LISTED);
        const u32 fr = __hip_atomic_load(ctl + C_FREE + slot, RLX, WGP);
        asm volatile("" ::: "memory");
        const u32x2 d = list[gi];
        src = __builtin_amdgcn_readfirstlane(d.x); lin = __builtin_amdgcn_readfirstlane(d.y) & 1;
        if (__builtin_amdgcn_readfirstlane(lt.x) > gi && __builtin_amdgcn_readfirstlane(fr) + RING > gi) break;
        if (__builtin_amdgcn_readfirstlane(lt.y) == gi + 1) { end = true; break; }
        if (tries == 3) { WAIT_VM0(); lds_set(ctl + C_LANDED, gi); pub = gi; }
        __builtin_amdgcn_s_sleep(1);
      }
      if (end) break;
      if (lin) pull4l(vl, vl, vl, vl, pws + src, lds0 + RING0 + slot * 4096);
      else pull4l(f0, f0 + 31744, f0 + 63488, f0 + 95232, fsrc + src, lds0 + RING0 + slot * 4096);
      if (gi > pub + BACK) { pub = gi - BACK; lds_set(ctl + C_LANDED, pub); }
      lds_set(ctl + C_TICKET, gi + 1);
      slot = slot + 1 == RING ? 0 : slot + 1;
    }
    WAIT_VM0();
    lds_set(ctl + C_LANDED, gi);
  } else if (wv <= NST) {
    char* const fdst = a.out[0];
    u32 landed = 0, prev = 0;
    auto done = [&]() {
      if (prev & 1) {
        const u32 c = (prev >> 1) & 127;
        if (lds_add(ctl + C_CNT + c, lane) == 1 && pair && lane == 0)
          __hip_atomic_store((GAS u32*)(ws + OFF_PFLAG + c * 64) + j, seq, RLX, AGT);
      } else if (prev & 2) {
        if (lds_add(ctl + C_HELD, lane) == 2 * RP - 1 && lane == 0)
          __hip_atomic_store((GAS u32*)(a.ws[xl ^ 4] + OFF_HOLD + (xl ^ 4) * 128) + j, seq, RLX, SYS);
      }
      prev = 0;
    };
#pragma unroll 1
    for (;;) {
      const u32 gi = lds_add(ctl + C_TAKEN, lane);
      if (landed <= gi) {
        landed = lds_get(ctl + C_LANDED);
        if (landed <= gi) {
          WAIT_VM0();
          done();
          bool end = false;
          do {
            end = lds_get(ctl + C_TOTAL) - 1 <= gi;
            if (end) break;
            __builtin_amdgcn_s_sleep(1);
            landed = lds_get(ctl + C_LANDED);
          } while (landed <= gi);
          if (end) break;
        }
      }
      const u32 slot = gi % RING;
      const u32 desc = __builtin_amdgcn_readfirstlane(list[gi].y);
      const LAS u32x4* rs = (const LAS u32x4*)(shm + RING0 + slot * 4096) + lane;
      u32x4 d0 = rs[0];
      const u32x4 d1 = rs[64], d2 = rs[128], d3 = rs[192];
      lds_set(ctl + C_FREE + slot, gi + 1);
      if (lds_get(ctl + C_BAD)) d0.x = 0x7fc07fc0u;
      WAIT_VM0();
      if (desc & 1) {
        char* db = ws + (desc & ~4095u);
        st16_agent(vl, d0, db); st16_agent(vl, d1, db + 1024); st16_agent(vl, d2, db + 2048); st16_agent(vl, d3, db + 3072);
      } else {
        char* db = fdst + (desc & ~255u);
        st16_sys(f0, d0, db); st16_sys(f0, d1, db + 32768); st16_sys(f0, d2, db + 65536); st16_sys(f0, d3, db + 98304);
      }
      done();
      prev = desc;
    }
  } else if (wv == 6) {
    const u32 tag = seq & 0xffffff;
    const GAS u32* qf = (const GAS u32*)(ws + OFF_QUEUE + xl * 1024) + lane;
    const GAS u32* qs = pair ? qf + 128 : lane < 32 ? (const GAS u32*)(ws + OFF_SFLAG + (xl * 4 + j) * 128) + lane
                                                    : (const GAS u32*)(ws + OFF_HOLD + xl * 128) + (lane & 3);
    u32 nf = 0, open = 0;
    for (;;) {
      u32 e0 = __hip_atomic_load(qf, RLX, SYS), e1 = __hip_atomic_load(qf + 64, RLX, SYS), e2 = __hip_atomic_load(qs, RLX, SYS);
      u32 e3 = seq;
      if (pair) e3 = __hip_atomic_load(qs + 64, RLX, SYS);
      const bool late = expired(t0);
      if (late) { lds_set(ctl + C_BAD, 1); e0 = (seq << 8) | lane; e1 = (seq << 8) | (64 + lane); e2 = e3 = seq; }
      const u64 n0 = __ballot((e0 >> 8) != tag), n1 = __ballot((e1 >> 8) != tag);
      const u64 m2 = __ballot(e2 >= seq || (!pair && lane < NP0)), m3 = __ballot(e3 >= seq);
      const u32 cf = n0 ? (u32)__builtin_ctzll(n0) : (n1 ? 64u + (u32)__builtin_ctzll(n1) : 128u);
      const u32 staged = (u32)m2 == ~0u ? 32u : (u32)__builtin_ctz(~(u32)m2);
      if (cf != nf) {
        if (lane >= nf) ctl[C_LIST + lane] = tile_off(rank ^ xl, e0);
        if (64 + lane >= nf) ctl[C_LIST + 64 + lane] = tile_off(rank ^ xl, e1);
        nf = cf;
      }
      if (pair) {
        open = cf >= PAIR_OPEN;
        *(volatile LAS u32x4*)(ctl + C_AVAIL + 4) = (u32x4){(u32)m2, (u32)(m2 >> 32), (u32)m3, (u32)(m3 >> 32)};
      } else if (!open && (u32)(m2 >> 32) == ~0u) {
        const GAS u32* qp = (const GAS u32*)(ws + OFF_QUEUE + 4 * 1024) + lane;
        const u32 g0 = __hip_atomic_load(qp, RLX, SYS), g1 = __hip_atomic_load(qp + 64, RLX, SYS);
        ctl[C_LIST2 + lane] = tile_off(rank ^ 4, g0);
        ctl[C_LIST2 + 64 + lane] = tile_off(rank ^ 4, g1);
        open = late || !__ballot(((g0 >> 8) != tag && lane < RP) || ((g1 >> 8) != tag && 64 + lane < RP));
      }
      asm volatile("" ::: "memory");
      *(volatile LAS u32x4*)(ctl + C_AVAIL) = (u32x4){staged, cf, open, 0u};
      if (cf == 128 && open && (!pair || (m2 & m3) == ~0ull)) break;
      if (!pair && staged == 32) __builtin_amdgcn_s_sleep(8);
    }
  } else if (wv == 7 && !pair) {
    GAS u32* const flag = (GAS u32*)(ws + OFF_INFLAG + xl * 16 + j * 4);
    const GAS u32* lf = (const GAS u32*)(ws + OFF_SFLAG + (xl * 4 + j) * 128) + (lane & 31);
    while (__ballot(__hip_atomic_load(lf, RLX, SYS) < seq && lane < NP0)) {
      if (expired(t0)) { lds_set(ctl + C_BAD, 1); break; }
      __builtin_amdgcn_s_sleep(1);
    }
    if (lane < NP0) __hip_atomic_store(flag + lane * 64, seq, RLX, AGT);
#pragma unroll 1
    for (u32 pi = NP0; pi < 32; ++pi) {
      while (lds_get(ctl + C_CNT + pi) < 4) __builtin_amdgcn_s_sleep(1);
      if (lane == 0) __hip_atomic_store(flag + pi * 64, seq, RLX, AGT);
    }
  }
}

__device__ __forceinline__ void announcer(const Args& a, u32 l, u32 lane, u32 seq, u64 t0) {
  const GAS u32* src = (const GAS u32*)(a.ws[0] + OFF_QUEUE) + lane;
  GAS u32* dst = (GAS u32*)(a.ws[l] + OFF_QUEUE + l * 1024) + lane;
  const u32 tag = seq & 0xffffff;
  bool s0 = false, s1 = false;
  while (__ballot(!s0 || !s1) && !expired(t0)) {
    const u32 e0 = __hip_atomic_load(src, RLX, AGT), e1 = __hip_atomic_load(src + 64, RLX, AGT);
    if (!s0 && (e0 >> 8) == tag) { __hip_atomic_store(dst, e0, RLX, SYS); s0 = true; }
    if (!s1 && (e1 >> 8) == tag) { __hip_atomic_store(dst + 64, e1, RLX, SYS); s1 = true; }
    __builtin_amdgcn_s_sleep(ANN_SLEEP);
  }
}

__device__ __forceinline__ u32 pack2(float lo, float hi) {
  bf16x2 v = {(__bf16)lo, (__bf16)hi};
  return __builtin_bit_cast(u32, v);
}
__device__ __forceinline__ float silu(float y) {
  const float c = __builtin_fminf(__builtin_fmaxf(y, -60.f), 60.f);
  return y * __builtin_amdgcn_rcpf(1.f + __builtin_amdgcn_exp2f(c * -1.44269504088896341f));
}

__global__ void __launch_bounds__(512, 2) fused_gemm_ar_silu(const Args a) {
  __shared__ __attribute__((aligned(1024))) char shm[LDS_BYTES];
  const u32 tid = threadIdx.x, lane = tid & 63;
  const u32 wv = __builtin_amdgcn_readfirstlane(tid >> 6);
  const u32 b = blockIdx.x, slot = b >> 3;
  const u32 xl = (a.xmap >> (4 * (b & 7))) & 7;
  LAS u32* lctr = (LAS u32*)(shm + LDS_CTL);
  const u32 lds0 = (u32)(size_t)(LAS char*)shm;
  const bool engine = xl != 0 && slot < 4;
  const u64 t0 = __builtin_amdgcn_s_memrealtime();
  const u32 rank = a.rank & 7;
  char* const ws = a.ws[0];
  GAS u32* ctr = (GAS u32*)(ws + OFF_CTR) + b;
  u32 calls = __hip_atomic_load(ctr, RLX, AGT);
  char* pws = a.ws[lane & 7];
  if (wv == 7) { lctr[lane] = 0; lctr[64 + lane] = 0; lctr[128 + lane] = 0; }
  const u32 lead = (slot - LEAD0) >> 2;
  const bool is_lead = (xl & 3) && wv < 4 && lead < NP0;
  if (is_lead) push_leading(a, rank ^ xl, xl, lead, slot & 3, wv, lane);
  if (!(xl & 3)) {
    if (slot >= 32 - STAGE_SPLIT) {
      stage_unit(a, rank, (slot - (32 - STAGE_SPLIT)) * 2 + (xl >> 2), wv, lane, calls, pws);
    }
  } else if (slot >= STAGE0) {
    while (__builtin_amdgcn_s_memrealtime() - t0 < STAGE_DELAY) __builtin_amdgcn_s_sleep(2);
    for (u32 u = 2 * STAGE_SPLIT + (slot - STAGE0) * 6 + (xl < 4 ? xl - 1 : xl - 2); u < 256; u += (32 - STAGE0) * 6) {
      stage_unit(a, rank, u, wv, lane, calls, pws);
    }
  }
  asm volatile("" : "+v"(calls));
  const u32 seq = __builtin_amdgcn_readfirstlane(calls) + 1;
  WG_BARRIER();
  if (is_lead && tid == 0)
    __hip_atomic_store((GAS u32*)(a.ws[xl] + OFF_SFLAG + (xl * 4 + (slot & 3)) * 128) + lead, seq, RLX, SYS);
  if (tid == 0) __hip_atomic_store(ctr, seq, RLX, AGT);
  if (engine) {
    link_engine(a, rank, xl, slot, wv, lane, seq, t0, (LAS char*)shm, lds0);
    return;
  }

  if (xl == 0 && slot == 0) {
    if (wv) announcer(a, wv, lane, seq, t0);
    return;
  }
#pragma unroll 1
  for (u32 ts = slot;; ts -= 4) {
    const u32 te = a.tile[xl * 32 + ts];
    const u32 lr = te & 15, lc = (te >> 4) & 15, rpos = (te >> 8) & 31, tix = (te >> 13) & 127;
    const u32 kind = lr >> 3;
    const u32 row0 = grow0(rank, lr), col0 = gcol0(rank, lc);
    const u32 vl = lane * 16;

    if (wv >= 4) {
      const u32 w4 = wv - 4;
      const char* xo = a.x + (size_t)(row0 + w4 * 32) * 1024;
      const char* wo = a.w + ((size_t)w4 * 16 * 8192 + col0 * 2);
      const char* xs = ws + (OFF_XS + (size_t)lr * 32 * 16384 + w4 * 4096);
      const char* wsg = ws + (OFF_WS + (size_t)lc * 32 * 16384 + w4 * 4096);
      const u32 lq = lds0 + w4 * 4096;
      const GAS u32x4* inflag = (const GAS u32x4*)(ws + OFF_INFLAG + rpos * 256 + (lane & 7) * 16);
      const GAS u32x4* lastflag = (lane & 7) ? inflag : (const GAS u32x4*)(ws + OFF_PFLAG + tix * 64);
      { const u32 d = START_DELAY + rpos * START_SLOPE; while (__builtin_amdgcn_s_memrealtime() - t0 < d) __builtin_amdgcn_s_sleep(4); }
      if (wv == 7 && ts == slot) {
        __builtin_amdgcn_fence(__ATOMIC_ACQUIRE, "");
        WAIT_VM0();
      }
      __builtin_amdgcn_s_setprio(2);
      {
        const u32 x0 = src_x(0, lane), x1 = src_x(1, lane), x2 = src_x(2, lane), x3 = src_x(3, lane);
        const u32 y0 = src_w(0, lane), y1 = src_w(1, lane), y2 = src_w(2, lane), y3 = src_w(3, lane);
#pragma unroll
        for (u32 kb = 0; kb < 8; ++kb) {
          if (kb >= 3) asm volatile("s_waitcnt vmcnt(16)\n\ts_barrier" ::: "memory");
          glds4v(x0, x1, x2, x3, xo + kb * 128, lq + (kb & 3) * STAGE_BYTES);
          glds4v(y0, y1, y2, y3, wo + (size_t)kb * 64 * 8192, lq + (kb & 3) * STAGE_BYTES + 16384);
        }
      }
      const u32 nst = 32 + kind;
      u32 avail = 8, issued = 8, ord = 0, seen = 0;
      auto refresh = [&]() {
        if (w4 == 0) {
          u32x4 f;
          const GAS u32x4* fp = issued >= 20 ? lastflag : inflag;
          asm volatile("global_load_dwordx4 %0, %1, off sc0 sc1\n\ts_waitcnt vmcnt(0)" : "=&v"(f) : "v"(fp) : "memory");
          u32 mask = (u32)__ballot(f.x >= seq && f.y >= seq && f.z >= seq && f.w >= seq);
          if (expired(t0)) { lds_set(lctr + C_BAD, 1); mask = 0xff; }
          ord |= (mask & 1) << 21;
          for (u32 m = mask & 0xee & ~seen; m; m &= m - 1) {
            const u32 l = __builtin_ctz(m);
            ord = (ord | ((l < 4 ? l + 1 : l) << (3 * ((ord >> 18) & 7)))) + (1u << 18);
          }
          seen |= mask;
          lds_set(lctr + C_MASK, ord);
        } else {
          ord = lds_get(lctr + C_MASK);
        }
        const u32 n = (ord >> 18) & 7;
        avail = 8 + 4 * n + (n == 6 ? (ord >> 21) & kind : 0);
      };
#pragma unroll 1
      for (u32 s = 5; s <= 32; ++s) {
        if (issued == s + 2 && !(issued & 3) && issued + 4 <= avail) {
          const u32 off = ((ord >> (3 * ((issued - 8) >> 2))) & 7) * 65536;
          const char* sx = xs + off;
          const char* sw = wsg + off;
#pragma unroll
          for (u32 i = 0; i < 4; ++i) {
            glds4v(vl, vl, vl, vl, sx + i * 16384, lq + i * STAGE_BYTES);
            glds4v(vl, vl, vl, vl, sw + i * 16384, lq + i * STAGE_BYTES + 16384);
            asm volatile("s_waitcnt vmcnt(16)\n\ts_barrier" ::: "memory");
          }
          issued += 4; s += 3;
          continue;
        }
        while (issued <= s + 2) {
          if (issued >= avail) {
            if (issued >= nst) break;
            refresh();
            while (issued >= avail && issued == s) {
              if (issued == 32) __builtin_amdgcn_s_sleep(8); else __builtin_amdgcn_s_sleep(2);
              refresh();
            }
            if (issued >= avail) break;
          }
          const u32 l = lq + (issued & 3) * STAGE_BYTES;
          const u32 kk = ((ord >> (3 * ((issued - 8) >> 2))) & 7) * 4 + (issued & 3);
          const char* sx = xs + kk * 16384;
          const char* sw = wsg + kk * 16384;
          if (issued == 32) { sx = ws + (OFF_PS + (size_t)tix * 32768 + w4 * 4096); sw = sx + 16384; }
          glds4v(vl, vl, vl, vl, sx, l);
          glds4v(vl, vl, vl, vl, sw, l + 16384);
          ++issued;
        }
        const u32 ahead = issued - s;
        if (ahead >= 3) asm volatile("s_waitcnt vmcnt(16)\n\ts_barrier" ::: "memory");
        else if (ahead == 2) asm volatile("s_waitcnt vmcnt(8)\n\ts_barrier" ::: "memory");
        else asm volatile("s_waitcnt vmcnt(0) lgkmcnt(0)\n\ts_barrier" ::: "memory");
      }
      __builtin_amdgcn_s_setprio(0);
      u32 pos = 0;
      if (kind && wv == 4 && lane == 0) pos = __hip_atomic_fetch_add((GAS u32*)(ws + OFF_TICK), 1u, RLX, AGT);
      pos = __builtin_amdgcn_readfirstlane(pos);
      if (wv == 4) lds_set(lctr + C_MASK, 0);
      WG_BARRIER();
      if (wv == 4) {
        const u32 e = (seq << 8) | ((lr & 7) * 16 + lc);
        if (kind) {
          if (lane == 0) __hip_atomic_store((GAS u32*)(ws + OFF_QUEUE) + (pos & 127), e, RLX, AGT);
        } else if (lane == 0) {
          __hip_atomic_store((GAS u32*)(a.ws[4] + OFF_QUEUE + 4 * 1024 + 512) + tix, seq, RLX, SYS);
        }
      }
    } else {
      const u32 wn = wv & 1, wm = wv >> 1;
      const u32 g = lane >> 4, q = (lane >> 2) & 3, pp = lane & 3;
      u32 ax[2];
#pragma unroll
      for (int ks = 0; ks < 2; ++ks) ax[ks] = (wm * 64 + (lane & 15)) * 128 + (((4 * ks + g) ^ ((lane & 15) >> 1)) * 16);
      u32 aw[4];
#pragma unroll
      for (int nt = 0; nt < 4; ++nt)
        aw[nt] = 16384 + (8 * g + q) * 256 + (((2 * (wn * 4 + nt) + (pp >> 1)) ^ ((q | ((g & 1) << 2)) << 1)) * 16) + (pp & 1) * 8;

      f32x4 acc[4][4];
#pragma unroll
      for (int i = 0; i < 4; ++i)
#pragma unroll
        for (int jj = 0; jj < 4; ++jj) acc[i][jj] = (f32x4){0.f, 0.f, 0.f, 0.f};
      bf16x8 xa[4], wa[4], xb[4], wb[4];
      auto rd = [&](bf16x8* fx, bf16x8* fw, u32 t, int ks) {
        const LAS char* st = (const LAS char*)shm + (t & (NSTAGE - 1)) * STAGE_BYTES;
#pragma unroll
        for (int i = 0; i < 4; ++i) {
          const bf16x4 lo = __builtin_amdgcn_ds_read_tr16_b64_v4bf16((LAS bf16x4*)(st + aw[i] + ks * 8192));
          const bf16x4 hi = __builtin_amdgcn_ds_read_tr16_b64_v4bf16((LAS bf16x4*)(st + aw[i] + ks * 8192 + 1024));
          fw[i] = __builtin_shufflevector(lo, hi, 0, 1, 2, 3, 4, 5, 6, 7);
          fx[i] = *(const LAS bf16x8*)(st + ax[ks] + i * 2048);
        }
      };
      auto mm = [&](const bf16x8* fx, const bf16x8* fw) {
#pragma unroll
        for (int mt = 0; mt < 4; ++mt)
#pragma unroll
          for (int nt = 0; nt < 4; ++nt)
            acc[nt][mt] = __builtin_amdgcn_mfma_f32_16x16x32_bf16(fw[nt], fx[mt], acc[nt][mt], 0, 0, 0);
      };
#define INTERLEAVE() do { _Pragma("unroll") for (int i_ = 0; i_ < 12; ++i_) { \
        __builtin_amdgcn_sched_group_barrier(0x8, 1, 0); __builtin_amdgcn_sched_group_barrier(0x100, 1, 0); } \
        __builtin_amdgcn_sched_group_barrier(0x8, 4, 0); } while (0)
      WG_BARRIER();
      rd(xa, wa, 0, 0);
      __builtin_amdgcn_sched_barrier(0);
      for (u32 t = 0; t < 31; ++t) {
        rd(xb, wb, t, 1);
        mm(xa, wa);
        INTERLEAVE();
        __builtin_amdgcn_sched_barrier(0);
        WG_BARRIER();
        __builtin_amdgcn_sched_barrier(0);
        rd(xa, wa, t + 1, 0);
        mm(xb, wb);
        INTERLEAVE();
        __builtin_amdgcn_sched_barrier(0);
      }
      rd(xb, wb, 31, 1);
      mm(xa, wa);
      INTERLEAVE();
      __builtin_amdgcn_sched_barrier(0);
      mm(xb, wb);
      WG_BARRIER();
      const bool bad = lds_get(lctr + C_BAD) != 0;
      const float qnan = __builtin_nanf("");
      if (kind == 0) {
        char* db = ws + OFF_PO + (size_t)tix * 32768 + wv * 8192;
#pragma unroll
        for (int mt = 0; mt < 4; ++mt)
#pragma unroll
          for (int np = 0; np < 2; ++np) {
            f32x4 u = acc[2 * np][mt], v = acc[2 * np + 1][mt];
            if (bad) u[0] = qnan;
            const u32x4 o = {pack2(u[0], u[1]), pack2(u[2], u[3]), pack2(v[0], v[1]), pack2(v[2], v[3])};
            st16_sys(vl, o, db + (mt * 2 + np) * 1024);
          }
      } else {
        const LAS u32x4* pr = (const LAS u32x4*)(shm + (32 & (NSTAGE - 1)) * STAGE_BYTES + wv * 8192) + lane;
        char* ob = a.out[0] + (size_t)(row0 + wm * 64) * 8192 + (col0 + wn * 64) * 2;
        const u32 vo = (lane & 15) * 8192 + (g & 1) * 32 + (g >> 1) * 16;
#pragma unroll
        for (int mt = 0; mt < 4; ++mt)
#pragma unroll
          for (int np = 0; np < 2; ++np) {
            const u32x4 pv = pr[(mt * 2 + np) * 64];
            f32x4 u = acc[2 * np][mt], v = acc[2 * np + 1][mt];
            u[0] += __uint_as_float(pv.x << 16); u[1] += __uint_as_float(pv.x & 0xffff0000u);
            u[2] += __uint_as_float(pv.y << 16); u[3] += __uint_as_float(pv.y & 0xffff0000u);
            v[0] += __uint_as_float(pv.z << 16); v[1] += __uint_as_float(pv.z & 0xffff0000u);
            v[2] += __uint_as_float(pv.w << 16); v[3] += __uint_as_float(pv.w & 0xffff0000u);
            if (bad) u[0] = qnan;
            const u32 a0 = pack2(silu(u[0]), silu(u[1])), a1 = pack2(silu(u[2]), silu(u[3]));
            const u32 b0 = pack2(silu(v[0]), silu(v[1])), b1 = pack2(silu(v[2]), silu(v[3]));
            const auto s0 = __builtin_amdgcn_permlane16_swap(a0, b0, false, false);
            const auto s1 = __builtin_amdgcn_permlane16_swap(a1, b1, false, false);
            const u32x4 o = {s0[0], s1[0], s0[1], s1[1]};
            st16_sys(vo, o, ob + (size_t)mt * 16 * 8192 + np * 64);
          }
      }
      WAIT_VM0();
      WG_BARRIER();
    }

    if (ts < 4 || ts >= (xl ? 8u : 5u)) break;
  }
}

static void build_tables(Args& a) {
  int pos[32];
  for (int i = 0; i < 32; ++i) pos[PIECES[i]] = i;
  auto rdy = [&](int lr, int lc) { return pos[lr] > pos[16 + lc] ? pos[lr] : pos[16 + lc]; };
  int prank[128];
  for (int t = 0; t < 128; ++t) {
    const int kt = (rdy(8 + (t >> 4), t & 15) * 32 + rdy(t >> 4, t & 15)) * 128 + t;
    prank[t] = 0;
    for (int u = 0; u < 128; ++u) prank[t] += (rdy(8 + (u >> 4), u & 15) * 32 + rdy(u >> 4, u & 15)) * 128 + u < kt;
  }
  for (int xl = 0; xl < 8; ++xl) {
    int key[32], idx[32], ent[32];
    for (int t = 0; t < 32; ++t) {
      const int lr = (t >> 2) * 2 + (xl >> 2), lc = (t & 3) * 4 + (xl & 3), pr = prank[(lr & 7) * 16 + lc];
      key[t] = lr < 8 ? rdy(lr, lc) * 128 + pr : 10000 + pr;
      ent[t] = lr | (lc << 4) | (rdy(lr, lc) << 8) | (pr << 13);
      idx[t] = t;
    }
    for (int i = 1; i < 32; ++i)
      for (int j = i; j > 0 && key[idx[j]] < key[idx[j - 1]]; --j) { int s = idx[j]; idx[j] = idx[j - 1]; idx[j - 1] = s; }
    for (int slot = 0; slot < 32; ++slot) a.tile[xl * 32 + slot] = ent[idx[slot < 4 ? 28 + slot : slot - 4]];
  }
}

extern "C" void kernel_launch(int rank, int nranks, ncclComm_t comm, hipStream_t stream,
                              void* const* d_in, const int* in_sizes, int n_in,
                              void* d_out, int out_size, const int* in_dtypes, int out_dtype,
                              void* const* d_out_all, void* const* d_ws_all, size_t ws_bytes) {
  if (nranks != 8 || (unsigned)rank >= 8u || n_in < 2 || ws_bytes < WS_NEED || out_size != 4096 * 4096 ||
      in_sizes[0] != 4096 * 512 || in_sizes[1] != 512 * 4096) {
    fprintf(stderr, "kernel_launch: unexpected problem shape\n");
    return;
  }
  Args a;
  a.x = (const char*)d_in[0];
  a.w = (const char*)d_in[1];
  for (int l = 0; l < 8; ++l) { a.out[l] = (char*)d_out_all[rank ^ l]; a.ws[l] = (char*)d_ws_all[rank ^ l]; }
  a.rank = (u32)rank;
  static const unsigned char near_xcd[8][8] = {{0, 4, 2, 3, 6, 5, 1, 7}, {4, 0, 7, 5, 2, 3, 6, 1}, {2, 7, 0, 1, 4, 5, 3, 6}, {3, 5, 1, 0, 7, 4, 2, 6},
                                               {5, 2, 4, 7, 0, 1, 6, 3}, {6, 3, 5, 4, 1, 0, 7, 2}, {1, 6, 3, 2, 5, 7, 0, 4}, {7, 1, 6, 5, 3, 2, 4, 0}};
  a.xmap = 0;
  for (int p = 0; p < 8; ++p) a.xmap |= (u32)(rank ^ p) << (4 * near_xcd[rank][p]);
  build_tables(a);
  fused_gemm_ar_silu<<<256, 512, 0, stream>>>(a);
  hipError_t e = hipGetLastError();
  if (e != hipSuccess) fprintf(stderr, "launch failed: %s\n", hipGetErrorString(e));
}
```

```cpp
#include <hip/hip_runtime.h>
#include <rccl/rccl.h>
#include <cstdio>
#include <cstdint>

typedef unsigned int u32;
typedef unsigned long long u64;
typedef __attribute__((ext_vector_type(4))) float f32x4;
typedef __attribute__((ext_vector_type(4))) unsigned int u32x4;
typedef __attribute__((ext_vector_type(2))) unsigned int u32x2;
typedef __attribute__((ext_vector_type(4))) __bf16 bf16x4;
typedef __attribute__((ext_vector_type(8))) __bf16 bf16x8;
typedef __attribute__((ext_vector_type(2))) __bf16 bf16x2;
#define LAS __attribute__((address_space(3)))
#define GAS __attribute__((address_space(1)))
#define RLX __ATOMIC_RELAXED
#define SYS __HIP_MEMORY_SCOPE_SYSTEM
#define AGT __HIP_MEMORY_SCOPE_AGENT
#define WGP __HIP_MEMORY_SCOPE_WORKGROUP

constexpr size_t OFF_CTR = 0;
constexpr size_t OFF_ROLE = 1024;
constexpr size_t OFF_TICK = 5120;
constexpr size_t OFF_POOL = 5376;
constexpr size_t OFF_HOLD = 6144;
constexpr size_t OFF_INFLAG = 8192;
constexpr size_t OFF_PFLAG = 16384;
constexpr size_t OFF_QUEUE = 32768;
constexpr size_t OFF_SFLAG = 45056;
constexpr size_t OFF_XS = 1u << 20;
constexpr size_t OFF_WS = 9u << 20;
constexpr size_t OFF_PS = 17u << 20;
constexpr size_t OFF_PO = 21u << 20;
constexpr size_t OFF_SX = 25u << 20;
constexpr size_t OFF_SW = 29u << 20;
constexpr size_t WS_NEED = 33u << 20;

constexpr u64 DEADLINE = 100000000ull;
constexpr int NSTAGE = 4;
constexpr int STAGE_BYTES = 32768;
constexpr int LDS_CTL = NSTAGE * STAGE_BYTES;
constexpr int LDS_BYTES = LDS_CTL + 2048;

#define PIECE_ORDER 0, 16, 1, 17, 2, 18, 3, 19, 8, 20, 21, 9, 22, 23, 10, 24, 25, 11, 26, 27, 4, 28, 29, 30, 31, 5, 12, 6, 13, 7, 14, 15
constexpr u32 PIECES[32] = {PIECE_ORDER};
constexpr u64 pack_pieces(int first) {
  u64 v = 0;
  for (int i = 0; i < 8; ++i) v |= (u64)PIECES[first + i] << (8 * i);
  return v;
}
__device__ __forceinline__ u32 piece_at(u32 i) {
  constexpr u64 p0 = pack_pieces(0), p1 = pack_pieces(8), p2 = pack_pieces(16), p3 = pack_pieces(24);
  const u64 v = i < 16 ? (i < 8 ? p0 : p1) : (i < 24 ? p2 : p3);
  return (u32)(v >> (8 * (i & 7))) & 31u;
}

constexpr u64 pack_pos(int first) {
  u64 v = 0;
  for (int i = 0; i < 32; ++i) if ((int)PIECES[i] >= first && (int)PIECES[i] < first + 8) v |= (u64)i << (8 * (PIECES[i] - first));
  return v;
}
__device__ __forceinline__ u32 pos_of(u32 c) {
  constexpr u64 p0 = pack_pos(0), p1 = pack_pos(8), p2 = pack_pos(16), p3 = pack_pos(24);
  const u64 v = c < 16 ? (c < 8 ? p0 : p1) : (c < 24 ? p2 : p3);
  return (u32)(v >> (8 * (c & 7))) & 31u;
}
#define NP0 3
#define RP 100
#define ANN_SLEEP 32
#define PAIR_OPEN 64
#define FINE 28
constexpr u32 NBIG = RP - FINE, NPOOL = NBIG + 2 * FINE;
struct StageOrder { u64 v[4]; };
constexpr StageOrder make_stage_order() {
  StageOrder o = {{0, 0, 0, 0}};
  bool seen[24] = {};
  int n = 0;
  for (int k = NP0; k < 32 + NP0; ++k) {
    const int c = (int)PIECES[k & 31], cl = c < 16 ? (c & 7) : c - 8;
    if (seen[cl]) continue;
    seen[cl] = true;
    for (int e = 0; e < (c < 16 ? 2 : 1); ++e, ++n) o.v[n >> 3] |= (u64)(c < 16 ? cl * 2 + e : c) << (8 * (n & 7));
  }
  return o;
}
__device__ __forceinline__ u32 stage_class(u32 i) {
  constexpr StageOrder o = make_stage_order();
  constexpr u64 p0 = o.v[0], p1 = o.v[1], p2 = o.v[2], p3 = o.v[3];
  const u64 v = i < 16 ? (i < 8 ? p0 : p1) : (i < 24 ? p2 : p3);
  return (u32)(v >> (8 * (i & 7))) & 31u;
}

struct Args {
  const char* x;
  const char* w;
  u32 rank;
  u32 xmap;
  char* ws[8];
  char* out[8];
  char* wsx[8];
  u32 tile[256];
};

__device__ __forceinline__ u32 grow0(u32 r, u32 lr) {
  return ((r >> 1) & 1) * 2048 + ((lr >> 3) ^ (r >> 2) ^ 1) * 1024 + (lr & 7) * 128;
}
__device__ __forceinline__ u32 gcol0(u32 r, u32 lc) { return (r & 1) * 2048 + lc * 128; }
__device__ __forceinline__ u32 tile_off(u32 r, u32 id) {
  return grow0(r, 8 + ((id >> 4) & 7)) * 8192 + gcol0(r, id & 15) * 2;
}

__device__ __forceinline__ void st16_sys(u32 voff, u32x4 v, char* sbase) {
  asm volatile("global_store_dwordx4 %0, %1, %2 sc0 sc1\n\ts_nop 1" :: "v"(voff), "v"(v), "s"(sbase) : "memory");
}
__device__ __forceinline__ void st16_agent(u32 voff, u32x4 v, char* sbase) {
  asm volatile("global_store_dwordx4 %0, %1, %2 sc1\n\ts_nop 1" :: "v"(voff), "v"(v), "s"(sbase) : "memory");
}
__device__ __forceinline__ bool expired(u64 t0) { return __builtin_amdgcn_s_memrealtime() - t0 > DEADLINE; }
#define WAIT_VM0() asm volatile("s_waitcnt vmcnt(0)" ::: "memory")
#define WG_BARRIER() asm volatile("s_waitcnt lgkmcnt(0)\n\ts_barrier" ::: "memory")

#define WIN 8
#define NST 4
constexpr u32 RING = 24, RING0 = 16384;
#define AHEAD 7
#define START_DELAY 100
#define START_SLOPE 100
#define LEAD0 4
#define STAGE0 (LEAD0 + 4 * NP0)
#define STAGE_SPLIT 24
#define STAGE_DELAY 80
constexpr u32 C_CNT = 0;
constexpr u32 C_AVAIL = 128;
constexpr u32 C_LISTED = 136, C_TOTAL = 137, C_TICKET = 138, C_BAD = 139, C_MASK = 140, C_LANDED = 141, C_TAKEN = 142, C_HELD = 143, C_ROLE = 144;
constexpr u32 C_FREE = 160;
constexpr u32 C_LIST = 192, C_LIST2 = 320;

__device__ __forceinline__ void glds4v(u32 v0, u32 v1, u32 v2, u32 v3, const char* sbase, u32 lds) {
  asm volatile(
      "s_mov_b32 m0, %5\n\ts_nop 0\n\t"
      "global_load_lds_dwordx4 %0, %4\n\t"
      "global_load_lds_dwordx4 %1, %4 offset:1024\n\t"
      "global_load_lds_dwordx4 %2, %4 offset:2048\n\t"
      "global_load_lds_dwordx4 %3, %4 offset:3072"
      :: "v"(v0), "v"(v1), "v"(v2), "v"(v3), "s"(sbase), "s"(lds) : "memory");
}
__device__ __forceinline__ void pull4l(u32 v0, u32 v1, u32 v2, u32 v3, const char* sbase, u32 lds) {
  asm volatile(
      "s_mov_b32 m0, %5\n\ts_waitcnt vmcnt(%6)\n\t"
      "global_load_lds_dwordx4 %0, %4 sc0 sc1\n\t"
      "global_load_lds_dwordx4 %1, %4 offset:1024 sc0 sc1\n\t"
      "global_load_lds_dwordx4 %2, %4 offset:2048 sc0 sc1\n\t"
      "global_load_lds_dwordx4 %3, %4 offset:3072 sc0 sc1"
      :: "v"(v0), "v"(v1), "v"(v2), "v"(v3), "s"(sbase), "s"(lds), "n"(WIN) : "memory");
}
__device__ __forceinline__ u32 src_x(u32 s, u32 lane) {
  return (8 * s + (lane >> 3)) * 1024 + (((lane & 7) ^ (4 * (s & 1) + (lane >> 4))) * 16) - s * 1024;
}
__device__ __forceinline__ u32 src_w(u32 s, u32 lane) {
  return (4 * s + (lane >> 4)) * 8192 + (((lane & 15) ^ (((lane >> 4) | ((s >> 1) << 2)) << 1)) * 16) - s * 1024;
}
__device__ __forceinline__ u32 lds_get(LAS u32* p) {
  asm volatile("" ::: "memory");
  const u32 v = __builtin_amdgcn_readfirstlane(__hip_atomic_load(p, RLX, WGP));
  asm volatile("" ::: "memory");
  return v;
}
__device__ __forceinline__ void lds_set(LAS u32* p, u32 v) {
  asm volatile("" ::: "memory");
  __hip_atomic_store(p, v, RLX, WGP);
  asm volatile("" ::: "memory");
}
__device__ __forceinline__ u32 lds_add(LAS u32* p, u32 lane) {
  u32 t = 0;
  if (lane == 0) t = __hip_atomic_fetch_add(p, 1u, RLX, WGP);
  return __builtin_amdgcn_readfirstlane(t);
}

__device__ __forceinline__ void ld4(u32x4& d0, u32x4& d1, u32x4& d2, u32x4& d3, u32 v0, u32 v1, u32 v2, u32 v3, const char* sb) {
  asm volatile("global_load_dwordx4 %0, %4, %8\n\tglobal_load_dwordx4 %1, %5, %8 offset:1024\n\t"
               "global_load_dwordx4 %2, %6, %8 offset:2048\n\tglobal_load_dwordx4 %3, %7, %8 offset:3072\n\ts_waitcnt vmcnt(0)"
               : "=&v"(d0), "=&v"(d1), "=&v"(d2), "=&v"(d3) : "v"(v0), "v"(v1), "v"(v2), "v"(v3), "s"(sb) : "memory");
}
__device__ __forceinline__ u32 kb_first(u32 xl) { return xl < 4 ? 4 + 4 * xl : 4 * xl; }

__device__ __forceinline__ void push_leading(const Args& a, char* db, u32 p, u32 xl, u32 i, u32 j, u32 kb, u32 lane) {
  const u32 pc = piece_at(i), k0 = (p >> 2) * 256 + kb * 64;
  u32x4 d0, d1, d2, d3;
  if (pc >> 4) ld4(d0, d1, d2, d3, src_w(0, lane), src_w(1, lane), src_w(2, lane), src_w(3, lane),
                   a.w + ((k0 + j * 16) * 8192 + gcol0(p, pc & 15) * 2));
  else ld4(d0, d1, d2, d3, src_x(0, lane), src_x(1, lane), src_x(2, lane), src_x(3, lane),
           a.x + ((grow0(p, pc) + j * 32) * 1024 + k0 * 2));
  db += OFF_XS + (pc * 32 + kb_first(xl) + kb) * 16384 + j * 4096;
  const u32 vd = lane * 16;
  st16_sys(vd, d0, db); st16_sys(vd, d1, db + 1024); st16_sys(vd, d2, db + 2048); st16_sys(vd, d3, db + 3072);
}

__device__ __forceinline__ void stage_unit(const Args& a, u32 rank, u32 unit, u32 wv, u32 lane, u32 calls) {
  const u32 j = unit & 3, sub = (unit >> 2) & 1, cls = stage_class(unit >> 3);
  const bool isw = cls >= 16;
  const u32 ch = isw ? sub * 16 + (cls & 15) : (cls & 1) * 16 + sub * 8 + (cls >> 1);
  u32x4 d0, d1, d2, d3;
  if (isw) ld4(d0, d1, d2, d3, src_w(0, lane), src_w(1, lane), src_w(2, lane), src_w(3, lane), a.w + ((wv * 64 + j * 16) * 8192 + ch * 256));
  else ld4(d0, d1, d2, d3, src_x(0, lane), src_x(1, lane), src_x(2, lane), src_x(3, lane), a.x + ((ch * 128 + j * 32) * 1024 + wv * 128));
  char* db = a.ws[0] + ((isw ? OFF_SW : OFF_SX) + (ch * 8 + wv) * 16384 + j * 4096);
  const u32 vd = lane * 16;
  st16_sys(vd, d0, db); st16_sys(vd, d1, db + 1024); st16_sys(vd, d2, db + 2048); st16_sys(vd, d3, db + 3072);
  const u32 p = rank ^ (lane & 7);
  const u32 pos = pos_of(isw ? cls : (sub ^ (p >> 2) ^ 1) * 8 + (cls >> 1));
  u32 tell = wv == 0 && lane < 8 && (lane & 3) && (isw ? p & 1 : (p >> 1) & 1) == ch >> 4 && pos >= NP0;
  char *w1 = a.ws[1], *w2 = a.ws[2], *w3 = a.ws[3], *w5 = a.ws[5], *w6 = a.ws[6], *w7 = a.ws[7];
  asm volatile("" : "+s"(w1), "+s"(w2), "+s"(w3), "+s"(w5), "+s"(w6), "+s"(w7));
  char* pw = lane & 4 ? (lane & 2 ? (lane & 1 ? w7 : w6) : w5) : (lane & 2 ? (lane & 1 ? w3 : w2) : w1);
  pw += OFF_SFLAG + ((lane & 7) * 4 + j) * 128 + pos * 4;
  asm volatile("s_waitcnt vmcnt(0)\n\ts_barrier" : "+v"(pw), "+v"(tell), "+s"(calls) :: "memory");
  if (tell) __hip_atomic_store((GAS u32*)pw, calls + 1, RLX, SYS);
}

__device__ __forceinline__ void link_engine(const Args& a, u32 rank, u32 xl, u32 j, u32 wv, u32 lane, u32 seq, u64 t0,
                                            LAS char* shm, u32 lds0) {
  LAS u32* ctl = (LAS u32*)(shm + LDS_CTL);
  const bool pair = xl == 4;
  const u32 f0 = (lane >> 4) * 8192 + (lane & 15) * 16, vl = lane * 16;
  char* const ws = a.ws[0];
  volatile LAS u32x2* list = (volatile LAS u32x2*)shm;
  if (wv == 5) {
    const u32 isrc = OFF_SX + (rank >> 2) * 4 * 16384 + j * 4096;
    const u32 idst = OFF_XS + kb_first(xl) * 16384 + j * 4096;
    u32 gi = 0, ni = pair ? 32u : NP0, nf = pair ? RP : 0u;
    u64 left0 = pair ? ~0ull : 0ull, left1 = left0;
    bool pool = true;
#pragma unroll 1
    while (pool || nf < 128 || (left0 | left1)) {
      asm volatile("" ::: "memory");
      const u32x4 av = *(volatile LAS u32x4*)(ctl + C_AVAIL), pm = *(volatile LAS u32x4*)(ctl + C_AVAIL + 4);
      const bool near = gi < lds_get(ctl + C_TICKET) + AHEAD;
      const u64 a0 = left0 & (((u64)__builtin_amdgcn_readfirstlane(pm.y) << 32) | (u32)__builtin_amdgcn_readfirstlane(pm.x));
      const u64 a1 = left1 & (((u64)__builtin_amdgcn_readfirstlane(pm.w) << 32) | (u32)__builtin_amdgcn_readfirstlane(pm.z));
      const u32 cf = __builtin_amdgcn_readfirstlane(av.y);
      u32 src = 0, dst = 0, n = 0, step = 16 * 8192;
      if (!near) {
      } else if (a0 | a1) {
        const u32 id = a0 ? __builtin_ctzll(a0) : 64 + __builtin_ctzll(a1);
        if (a0) left0 &= ~(1ull << id); else left1 &= ~(1ull << (id - 64));
        src = OFF_PO + id * 32768 + j * 8192; dst = OFF_PS + id * 32768 + j * 8192 + id * 2 + 1; n = 2; step = 4096;
      } else if (ni < 32) {
        if (ni < __builtin_amdgcn_readfirstlane(av.x)) {
          const u32 pc = piece_at(ni);
          src = isrc + (pc >> 4 ? 4096 + gcol0(rank, pc & 15) : grow0(rank, pc)) * 1024;
          dst = idst + pc * 32 * 16384 + ni * 2 + 1; n = 4; step = 16384;
          ++ni;
        }
      } else if (nf < cf) {
        src = lds_get(ctl + C_LIST + nf) + j * 32 * 8192; dst = src + (nf < RP ? 2 : 0); n = 2;
        ++nf;
      } else if (pool && __builtin_amdgcn_readfirstlane(av.z) && (!(left0 | left1) || cf >= RP)) {
        u32 t = 0;
        if (lane == 0) t = __hip_atomic_fetch_add((GAS u32*)(ws + OFF_POOL), 1u, RLX, AGT);
        t = __builtin_amdgcn_readfirstlane(t) - (seq - 1) * (NPOOL + 28);
        if (t < NPOOL) {
          const u32 ht = t < NBIG ? 2 * t : t + NBIG;
          if (pair) while (lds_get(ctl + C_AVAIL + 1) <= (ht >> 1)) __builtin_amdgcn_s_sleep(1);
          src = dst = lds_get(ctl + (pair ? C_LIST : C_LIST2) + (ht >> 1)) + (ht & 1) * 64 * 8192;
          n = t < NBIG ? 8 : 4;
        } else {
          pool = false;
        }
      }
      if (n) {
        if (lane < n) list[gi + lane] = (u32x2){src + lane * step, dst + lane * step};
        gi += n;
        lds_set(ctl + C_LISTED, gi);
      } else {
        __builtin_amdgcn_s_sleep(1);
      }
    }
    lds_set(ctl + C_TOTAL, gi + 1);
  } else if (wv == 0) {
    constexpr u32 BACK = (WIN + 3) / 4;
    const char* const pws = a.ws[xl];
    const char* const fsrc = a.out[xl];
    u32 slot = 0, pub = 0, gi = 0;
    __builtin_amdgcn_s_setprio(3);
#pragma unroll 1
    for (;; ++gi) {
      u32 src, lin;
      bool end = false;
      for (u32 tries = 0;; ++tries) {
        asm volatile("" ::: "memory");
        const u32x2 lt = *(volatile LAS u32x2*)(ctl + C_LISTED);
        const u32 fr = __hip_atomic_load(ctl + C_FREE + slot, RLX, WGP);
        asm volatile("" ::: "memory");
        const u32x2 d = list[gi];
        src = __builtin_amdgcn_readfirstlane(d.x); lin = __builtin_amdgcn_readfirstlane(d.y) & 1;
        if (__builtin_amdgcn_readfirstlane(lt.x) > gi && __builtin_amdgcn_readfirstlane(fr) + RING > gi) break;
        if (__builtin_amdgcn_readfirstlane(lt.y) == gi + 1) { end = true; break; }
        if (tries == 3) { WAIT_VM0(); lds_set(ctl + C_LANDED, gi); pub = gi; }
        __builtin_amdgcn_s_sleep(1);
      }
      if (end) break;
      if (lin) pull4l(vl, vl, vl, vl, pws + src, lds0 + RING0 + slot * 4096);
      else pull4l(f0, f0 + 31744, f0 + 63488, f0 + 95232, fsrc + src, lds0 + RING0 + slot * 4096);
      if (gi > pub + BACK) { pub = gi - BACK; lds_set(ctl + C_LANDED, pub); }
      lds_set(ctl + C_TICKET, gi + 1);
      slot = slot + 1 == RING ? 0 : slot + 1;
    }
    WAIT_VM0();
    lds_set(ctl + C_LANDED, gi);
  } else if (wv <= NST) {
    char* const fdst = a.out[0];
    u32 landed = 0, prev = 0;
    auto done = [&]() {
      if (prev & 1) {
        const u32 c = (prev >> 1) & 127;
        if (lds_add(ctl + C_CNT + c, lane) == 1 && pair && lane == 0)
          __hip_atomic_store((GAS u32*)(ws + OFF_PFLAG + c * 64) + j, seq, RLX, AGT);
      } else if (prev & 2) {
        if (lds_add(ctl + C_HELD, lane) == 2 * RP - 1 && lane == 0)
          __hip_atomic_store((GAS u32*)(a.ws[xl ^ 4] + OFF_HOLD + (xl ^ 4) * 128) + j, seq, RLX, SYS);
      }
      prev = 0;
    };
#pragma unroll 1
    for (;;) {
      const u32 gi = lds_add(ctl + C_TAKEN, lane);
      if (landed <= gi) {
        landed = lds_get(ctl + C_LANDED);
        if (landed <= gi) {
          WAIT_VM0();
          done();
          bool end = false;
          do {
            end = lds_get(ctl + C_TOTAL) - 1 <= gi;
            if (end) break;
            __builtin_amdgcn_s_sleep(1);
            landed = lds_get(ctl + C_LANDED);
          } while (landed <= gi);
          if (end) break;
        }
      }
      const u32 slot = gi % RING;
      const u32 desc = __builtin_amdgcn_readfirstlane(list[gi].y);
      const LAS u32x4* rs = (const LAS u32x4*)(shm + RING0 + slot * 4096) + lane;
      u32x4 d0 = rs[0];
      const u32x4 d1 = rs[64], d2 = rs[128], d3 = rs[192];
      lds_set(ctl + C_FREE + slot, gi + 1);
      if (lds_get(ctl + C_BAD)) d0.x = 0x7fc07fc0u;
      WAIT_VM0();
      if (desc & 1) {
        char* db = ws + (desc & ~4095u);
        st16_agent(vl, d0, db); st16_agent(vl, d1, db + 1024); st16_agent(vl, d2, db + 2048); st16_agent(vl, d3, db + 3072);
      } else {
        char* db = fdst + (desc & ~255u);
        st16_sys(f0, d0, db); st16_sys(f0, d1, db + 32768); st16_sys(f0, d2, db + 65536); st16_sys(f0, d3, db + 98304);
      }
      done();
      prev = desc;
    }
  } else if (wv == 6) {
    const u32 tag = seq & 0xffffff;
    const GAS u32* qf = (const GAS u32*)(ws + OFF_QUEUE + xl * 1024) + lane;
    const GAS u32* qs = pair ? qf + 128 : lane < 32 ? (const GAS u32*)(ws + OFF_SFLAG + (xl * 4 + j) * 128) + lane
                                                    : (const GAS u32*)(ws + OFF_HOLD + xl * 128) + (lane & 3);
    u32 nf = 0, open = 0;
    for (;;) {
      u32 e0 = __hip_atomic_load(qf, RLX, SYS), e1 = __hip_atomic_load(qf + 64, RLX, SYS), e2 = __hip_atomic_load(qs, RLX, SYS);
      u32 e3 = seq;
      if (pair) e3 = __hip_atomic_load(qs + 64, RLX, SYS);
      const bool late = expired(t0);
      if (late) { lds_set(ctl + C_BAD, 1); e0 = (seq << 8) | lane; e1 = (seq << 8) | (64 + lane); e2 = e3 = seq; }
      const u64 n0 = __ballot((e0 >> 8) != tag), n1 = __ballot((e1 >> 8) != tag);
      const u64 m2 = __ballot(e2 >= seq || (!pair && lane < NP0)), m3 = __ballot(e3 >= seq);
      const u32 cf = n0 ? (u32)__builtin_ctzll(n0) : (n1 ? 64u + (u32)__builtin_ctzll(n1) : 128u);
      const u32 staged = (u32)m2 == ~0u ? 32u : (u32)__builtin_ctz(~(u32)m2);
      if (cf != nf) {
        if (lane >= nf) ctl[C_LIST + lane] = tile_off(rank ^ xl, e0);
        if (64 + lane >= nf) ctl[C_LIST + 64 + lane] = tile_off(rank ^ xl, e1);
        nf = cf;
      }
      if (pair) {
        open = cf >= PAIR_OPEN;
        *(volatile LAS u32x4*)(ctl + C_AVAIL + 4) = (u32x4){(u32)m2, (u32)(m2 >> 32), (u32)m3, (u32)(m3 >> 32)};
      } else if (!open && (u32)(m2 >> 32) == ~0u) {
        const GAS u32* qp = (const GAS u32*)(ws + OFF_QUEUE + 4 * 1024) + lane;
        const u32 g0 = __hip_atomic_load(qp, RLX, SYS), g1 = __hip_atomic_load(qp + 64, RLX, SYS);
        ctl[C_LIST2 + lane] = tile_off(rank ^ 4, g0);
        ctl[C_LIST2 + 64 + lane] = tile_off(rank ^ 4, g1);
        open = late || !__ballot(((g0 >> 8) != tag && lane < RP) || ((g1 >> 8) != tag && 64 + lane < RP));
      }
      asm volatile("" ::: "memory");
      *(volatile LAS u32x4*)(ctl + C_AVAIL) = (u32x4){staged, cf, open, 0u};
      if (cf == 128 && open && (!pair || (m2 & m3) == ~0ull)) break;
      if (!pair && staged == 32) __builtin_amdgcn_s_sleep(8);
    }
  } else if (wv == 7 && !pair) {
    GAS u32* const flag = (GAS u32*)(ws + OFF_INFLAG + xl * 16 + j * 4);
    const GAS u32* lf = (const GAS u32*)(ws + OFF_SFLAG + (xl * 4 + j) * 128) + (lane & 31);
    while (__ballot(__hip_atomic_load(lf, RLX, SYS) < seq && lane < NP0)) {
      if (expired(t0)) { lds_set(ctl + C_BAD, 1); break; }
      __builtin_amdgcn_s_sleep(1);
    }
    if (lane < NP0) __hip_atomic_store(flag + lane * 64, seq, RLX, AGT);
#pragma unroll 1
    for (u32 pi = NP0; pi < 32; ++pi) {
      while (lds_get(ctl + C_CNT + pi) < 4) __builtin_amdgcn_s_sleep(1);
      if (lane == 0) __hip_atomic_store(flag + pi * 64, seq, RLX, AGT);
    }
  }
}

__device__ __forceinline__ void announcer(const Args& a, u32 l, u32 lane, u32 seq, u64 t0) {
  const GAS u32* src = (const GAS u32*)(a.ws[0] + OFF_QUEUE) + lane;
  GAS u32* dst = (GAS u32*)(a.ws[l] + OFF_QUEUE + l * 1024) + lane;
  const u32 tag = seq & 0xffffff;
  bool s0 = false, s1 = false;
  while (__ballot(!s0 || !s1) && !expired(t0)) {
    const u32 e0 = __hip_atomic_load(src, RLX, AGT), e1 = __hip_atomic_load(src + 64, RLX, AGT);
    if (!s0 && (e0 >> 8) == tag) { __hip_atomic_store(dst, e0, RLX, SYS); s0 = true; }
    if (!s1 && (e1 >> 8) == tag) { __hip_atomic_store(dst + 64, e1, RLX, SYS); s1 = true; }
    __builtin_amdgcn_s_sleep(ANN_SLEEP);
  }
}

__device__ __forceinline__ u32 pack2(float lo, float hi) {
  bf16x2 v = {(__bf16)lo, (__bf16)hi};
  return __builtin_bit_cast(u32, v);
}
__device__ __forceinline__ float silu(float y) {
  const float c = __builtin_fminf(__builtin_fmaxf(y, -60.f), 60.f);
  return y * __builtin_amdgcn_rcpf(1.f + __builtin_amdgcn_exp2f(c * -1.44269504088896341f));
}

__global__ void __launch_bounds__(512, 2) fused_gemm_ar_silu(const Args a) {
  __shared__ __attribute__((aligned(1024))) char shm[LDS_BYTES];
  const u32 tid = threadIdx.x, lane = tid & 63;
  const u32 wv = __builtin_amdgcn_readfirstlane(tid >> 6);
  const u32 b = blockIdx.x;
  u32 role;
  asm volatile("s_getreg_b32 %0, hwreg(HW_REG_XCC_ID)" : "=s"(role));
  role = (role & 7) | (b & ~7u);
  char* wsp = a.wsx[role & 7];
  asm volatile("" : "+s"(wsp) : "s"(a.x), "s"(a.w), "s"(a.rank), "s"(a.xmap), "s"(a.ws[0]));
  LAS u32* lctr = (LAS u32*)(shm + LDS_CTL);
  const u32 lds0 = (u32)(size_t)(LAS char*)shm;
  const u64 t0 = __builtin_amdgcn_s_memrealtime();
  const u32 rank = a.rank & 7;
  char* const ws = a.ws[0];
  GAS u32* ctr = (GAS u32*)(ws + OFF_CTR) + b;
  u32 calls = *(const __attribute__((address_space(4))) u32*)(size_t)ctr;
  if (wv == 7) { lctr[lane] = 0; lctr[64 + lane] = 0; lctr[128 + lane] = 0; }
  u32 xl, slot, seq;
  auto startup = [&](bool claim) {
    xl = (a.xmap >> (4 * (role & 7))) & 7;
    slot = role >> 3;
    const u32 lead = (slot - LEAD0) >> 2;
    const bool is_lead = (xl & 3) && wv < 4 && lead < NP0;
    if (is_lead) push_leading(a, wsp, rank ^ xl, xl, lead, slot & 3, wv, lane);
    if (!(xl & 3)) {
      if (slot >= 32 - STAGE_SPLIT) stage_unit(a, rank, (slot - (32 - STAGE_SPLIT)) * 2 + (xl >> 2), wv, lane, calls);
    } else if (slot >= STAGE0) {
      while (__builtin_amdgcn_s_memrealtime() - t0 < STAGE_DELAY) __builtin_amdgcn_s_sleep(2);
      for (u32 u = 2 * STAGE_SPLIT + (slot - STAGE0) * 6 + (xl < 4 ? xl - 1 : xl - 2); u < 256; u += (32 - STAGE0) * 6)
        stage_unit(a, rank, u, wv, lane, calls);
    }
    asm volatile("s_waitcnt vmcnt(0)" : "+s"(calls) :: "memory");
    seq = calls + 1;
    if (claim && tid == 448) {
      u32 r = role;
      for (u32 i = 0; i < 256 && __hip_atomic_exchange((GAS u32*)(ws + OFF_ROLE) + r, seq, RLX, AGT) == seq; ++i) r = (r + 1) & 255;
      lctr[C_ROLE] = r;
    }
    WG_BARRIER();
    if (is_lead && tid == 0)
      __hip_atomic_store((GAS u32*)(wsp + OFF_SFLAG + (xl * 4 + (slot & 3)) * 128) + lead, seq, RLX, SYS);
  };
  startup(true);
  const u32 got = lds_get(lctr + C_ROLE);
  if (__builtin_expect(got != role, 0)) { role = got; wsp = a.wsx[role & 7]; startup(false); }
  const bool engine = xl != 0 && slot < 4;
  if (tid == 0) __hip_atomic_store(ctr, seq, RLX, AGT);
  if (engine) {
    link_engine(a, rank, xl, slot, wv, lane, seq, t0, (LAS char*)shm, lds0);
    return;
  }

  if (xl == 0 && slot == 0) {
    if (wv) announcer(a, wv, lane, seq, t0);
    return;
  }
#pragma unroll 1
  for (u32 ts = slot;; ts -= 4) {
    const u32 te = a.tile[xl * 32 + ts];
    const u32 lr = te & 15, lc = (te >> 4) & 15, rpos = (te >> 8) & 31, tix = (te >> 13) & 127;
    const u32 kind = lr >> 3;
    const u32 row0 = grow0(rank, lr), col0 = gcol0(rank, lc);
    const u32 vl = lane * 16;

    if (wv >= 4) {
      const u32 w4 = wv - 4;
      const char* xo = a.x + (size_t)(row0 + w4 * 32) * 1024;
      const char* wo = a.w + ((size_t)w4 * 16 * 8192 + col0 * 2);
      const char* xs = ws + (OFF_XS + (size_t)lr * 32 * 16384 + w4 * 4096);
      const char* wsg = ws + (OFF_WS + (size_t)lc * 32 * 16384 + w4 * 4096);
      const u32 lq = lds0 + w4 * 4096;
      const GAS u32x4* inflag = (const GAS u32x4*)(ws + OFF_INFLAG + rpos * 256 + (lane & 7) * 16);
      const GAS u32x4* lastflag = (lane & 7) ? inflag : (const GAS u32x4*)(ws + OFF_PFLAG + tix * 64);
      { const u32 d = START_DELAY + rpos * START_SLOPE; while (__builtin_amdgcn_s_memrealtime() - t0 < d) __builtin_amdgcn_s_sleep(4); }
      if (wv == 7 && ts == slot) {
        __builtin_amdgcn_fence(__ATOMIC_ACQUIRE, "");
        WAIT_VM0();
      }
      __builtin_amdgcn_s_setprio(2);
      {
        const u32 x0 = src_x(0, lane), x1 = src_x(1, lane), x2 = src_x(2, lane), x3 = src_x(3, lane);
        const u32 y0 = src_w(0, lane), y1 = src_w(1, lane), y2 = src_w(2, lane), y3 = src_w(3, lane);
#pragma unroll
        for (u32 kb = 0; kb < 8; ++kb) {
          if (kb >= 3) asm volatile("s_waitcnt vmcnt(16)\n\ts_barrier" ::: "memory");
          glds4v(x0, x1, x2, x3, xo + kb * 128, lq + (kb & 3) * STAGE_BYTES);
          glds4v(y0, y1, y2, y3, wo + (size_t)kb * 64 * 8192, lq + (kb & 3) * STAGE_BYTES + 16384);
        }
      }
      const u32 nst = 32 + kind;
      u32 avail = 8, issued = 8, ord = 0, seen = 0;
      auto refresh = [&]() {
        if (w4 == 0) {
          u32x4 f;
          const GAS u32x4* fp = issued >= 20 ? lastflag : inflag;
          asm volatile("global_load_dwordx4 %0, %1, off sc0 sc1\n\ts_waitcnt vmcnt(0)" : "=&v"(f) : "v"(fp) : "memory");
          u32 mask = (u32)__ballot(f.x >= seq && f.y >= seq && f.z >= seq && f.w >= seq);
          if (expired(t0)) { lds_set(lctr + C_BAD, 1); mask = 0xff; }
          ord |= (mask & 1) << 21;
          for (u32 m = mask & 0xee & ~seen; m; m &= m - 1) {
            const u32 l = __builtin_ctz(m);
            ord = (ord | ((l < 4 ? l + 1 : l) << (3 * ((ord >> 18) & 7)))) + (1u << 18);
          }
          seen |= mask;
          lds_set(lctr + C_MASK, ord);
        } else {
          ord = lds_get(lctr + C_MASK);
        }
        const u32 n = (ord >> 18) & 7;
        avail = 8 + 4 * n + (n == 6 ? (ord >> 21) & kind : 0);
      };
#pragma unroll 1
      for (u32 s = 5; s <= 32; ++s) {
        if (issued == s + 2 && !(issued & 3) && issued + 4 <= avail) {
          const u32 off = ((ord >> (3 * ((issued - 8) >> 2))) & 7) * 65536;
          const char* sx = xs + off;
          const char* sw = wsg + off;
#pragma unroll
          for (u32 i = 0; i < 4; ++i) {
            glds4v(vl, vl, vl, vl, sx + i * 16384, lq + i * STAGE_BYTES);
            glds4v(vl, vl, vl, vl, sw + i * 16384, lq + i * STAGE_BYTES + 16384);
            asm volatile("s_waitcnt vmcnt(16)\n\ts_barrier" ::: "memory");
          }
          issued += 4; s += 3;
          continue;
        }
        while (issued <= s + 2) {
          if (issued >= avail) {
            if (issued >= nst) break;
            refresh();
            while (issued >= avail && issued == s) {
              if (issued == 32) __builtin_amdgcn_s_sleep(8); else __builtin_amdgcn_s_sleep(2);
              refresh();
            }
            if (issued >= avail) break;
          }
          const u32 l = lq + (issued & 3) * STAGE_BYTES;
          const u32 kk = ((ord >> (3 * ((issued - 8) >> 2))) & 7) * 4 + (issued & 3);
          const char* sx = xs + kk * 16384;
          const char* sw = wsg + kk * 16384;
          if (issued == 32) { sx = ws + (OFF_PS + (size_t)tix * 32768 + w4 * 4096); sw = sx + 16384; }
          glds4v(vl, vl, vl, vl, sx, l);
          glds4v(vl, vl, vl, vl, sw, l + 16384);
          ++issued;
        }
        const u32 ahead = issued - s;
        if (ahead >= 3) asm volatile("s_waitcnt vmcnt(16)\n\ts_barrier" ::: "memory");
        else if (ahead == 2) asm volatile("s_waitcnt vmcnt(8)\n\ts_barrier" ::: "memory");
        else asm volatile("s_waitcnt vmcnt(0) lgkmcnt(0)\n\ts_barrier" ::: "memory");
      }
      __builtin_amdgcn_s_setprio(0);
      u32 pos = 0;
      if (kind && wv == 4 && lane == 0) pos = __hip_atomic_fetch_add((GAS u32*)(ws + OFF_TICK), 1u, RLX, AGT);
      pos = __builtin_amdgcn_readfirstlane(pos);
      if (wv == 4) lds_set(lctr + C_MASK, 0);
      WG_BARRIER();
      if (wv == 4) {
        const u32 e = (seq << 8) | ((lr & 7) * 16 + lc);
        if (kind) {
          if (lane == 0) __hip_atomic_store((GAS u32*)(ws + OFF_QUEUE) + (pos & 127), e, RLX, AGT);
        } else if (lane == 0) {
          __hip_atomic_store((GAS u32*)(a.ws[4] + OFF_QUEUE + 4 * 1024 + 512) + tix, seq, RLX, SYS);
        }
      }
    } else {
      const u32 wn = wv & 1, wm = wv >> 1;
      const u32 g = lane >> 4, q = (lane >> 2) & 3, pp = lane & 3;
      u32 ax[2];
#pragma unroll
      for (int ks = 0; ks < 2; ++ks) ax[ks] = (wm * 64 + (lane & 15)) * 128 + (((4 * ks + g) ^ ((lane & 15) >> 1)) * 16);
      u32 aw[4];
#pragma unroll
      for (int nt = 0; nt < 4; ++nt)
        aw[nt] = 16384 + (8 * g + q) * 256 + (((2 * (wn * 4 + nt) + (pp >> 1)) ^ ((q | ((g & 1) << 2)) << 1)) * 16) + (pp & 1) * 8;

      f32x4 acc[4][4];
#pragma unroll
      for (int i = 0; i < 4; ++i)
#pragma unroll
        for (int jj = 0; jj < 4; ++jj) acc[i][jj] = (f32x4){0.f, 0.f, 0.f, 0.f};
      bf16x8 xa[4], wa[4], xb[4], wb[4];
      auto rd = [&](bf16x8* fx, bf16x8* fw, u32 t, int ks) {
        const LAS char* st = (const LAS char*)shm + (t & (NSTAGE - 1)) * STAGE_BYTES;
#pragma unroll
        for (int i = 0; i < 4; ++i) {
          const bf16x4 lo = __builtin_amdgcn_ds_read_tr16_b64_v4bf16((LAS bf16x4*)(st + aw[i] + ks * 8192));
          const bf16x4 hi = __builtin_amdgcn_ds_read_tr16_b64_v4bf16((LAS bf16x4*)(st + aw[i] + ks * 8192 + 1024));
          fw[i] = __builtin_shufflevector(lo, hi, 0, 1, 2, 3, 4, 5, 6, 7);
          fx[i] = *(const LAS bf16x8*)(st + ax[ks] + i * 2048);
        }
      };
      auto mm = [&](const bf16x8* fx, const bf16x8* fw) {
#pragma unroll
        for (int mt = 0; mt < 4; ++mt)
#pragma unroll
          for (int nt = 0; nt < 4; ++nt)
            acc[nt][mt] = __builtin_amdgcn_mfma_f32_16x16x32_bf16(fw[nt], fx[mt], acc[nt][mt], 0, 0, 0);
      };
#define INTERLEAVE() do { _Pragma("unroll") for (int i_ = 0; i_ < 12; ++i_) { \
        __builtin_amdgcn_sched_group_barrier(0x8, 1, 0); __builtin_amdgcn_sched_group_barrier(0x100, 1, 0); } \
        __builtin_amdgcn_sched_group_barrier(0x8, 4, 0); } while (0)
      WG_BARRIER();
      rd(xa, wa, 0, 0);
      __builtin_amdgcn_sched_barrier(0);
      for (u32 t = 0; t < 31; ++t) {
        rd(xb, wb, t, 1);
        mm(xa, wa);
        INTERLEAVE();
        __builtin_amdgcn_sched_barrier(0);
        WG_BARRIER();
        __builtin_amdgcn_sched_barrier(0);
        rd(xa, wa, t + 1, 0);
        mm(xb, wb);
        INTERLEAVE();
        __builtin_amdgcn_sched_barrier(0);
      }
      rd(xb, wb, 31, 1);
      mm(xa, wa);
      INTERLEAVE();
      __builtin_amdgcn_sched_barrier(0);
      mm(xb, wb);
      WG_BARRIER();
      const bool bad = lds_get(lctr + C_BAD) != 0;
      const float qnan = __builtin_nanf("");
      if (kind == 0) {
        char* db = ws + OFF_PO + (size_t)tix * 32768 + wv * 8192;
#pragma unroll
        for (int mt = 0; mt < 4; ++mt)
#pragma unroll
          for (int np = 0; np < 2; ++np) {
            f32x4 u = acc[2 * np][mt], v = acc[2 * np + 1][mt];
            if (bad) u[0] = qnan;
            const u32x4 o = {pack2(u[0], u[1]), pack2(u[2], u[3]), pack2(v[0], v[1]), pack2(v[2], v[3])};
            st16_sys(vl, o, db + (mt * 2 + np) * 1024);
          }
      } else {
        const LAS u32x4* pr = (const LAS u32x4*)(shm + (32 & (NSTAGE - 1)) * STAGE_BYTES + wv * 8192) + lane;
        char* ob = a.out[0] + (size_t)(row0 + wm * 64) * 8192 + (col0 + wn * 64) * 2;
        const u32 vo = (lane & 15) * 8192 + (g & 1) * 32 + (g >> 1) * 16;
#pragma unroll
        for (int mt = 0; mt < 4; ++mt)
#pragma unroll
          for (int np = 0; np < 2; ++np) {
            const u32x4 pv = pr[(mt * 2 + np) * 64];
            f32x4 u = acc[2 * np][mt], v = acc[2 * np + 1][mt];
            u[0] += __uint_as_float(pv.x << 16); u[1] += __uint_as_float(pv.x & 0xffff0000u);
            u[2] += __uint_as_float(pv.y << 16); u[3] += __uint_as_float(pv.y & 0xffff0000u);
            v[0] += __uint_as_float(pv.z << 16); v[1] += __uint_as_float(pv.z & 0xffff0000u);
            v[2] += __uint_as_float(pv.w << 16); v[3] += __uint_as_float(pv.w & 0xffff0000u);
            if (bad) u[0] = qnan;
            const u32 a0 = pack2(silu(u[0]), silu(u[1])), a1 = pack2(silu(u[2]), silu(u[3]));
            const u32 b0 = pack2(silu(v[0]), silu(v[1])), b1 = pack2(silu(v[2]), silu(v[3]));
            const auto s0 = __builtin_amdgcn_permlane16_swap(a0, b0, false, false);
            const auto s1 = __builtin_amdgcn_permlane16_swap(a1, b1, false, false);
            const u32x4 o = {s0[0], s1[0], s0[1], s1[1]};
            st16_sys(vo, o, ob + (size_t)mt * 16 * 8192 + np * 64);
          }
      }
      WAIT_VM0();
      WG_BARRIER();
    }

    if (ts < 4 || ts >= (xl ? 8u : 5u)) break;
  }
}

static void build_tables(Args& a) {
  int pos[32];
  for (int i = 0; i < 32; ++i) pos[PIECES[i]] = i;
  auto rdy = [&](int lr, int lc) { return pos[lr] > pos[16 + lc] ? pos[lr] : pos[16 + lc]; };
  int prank[128];
  for (int t = 0; t < 128; ++t) {
    const int kt = (rdy(8 + (t >> 4), t & 15) * 32 + rdy(t >> 4, t & 15)) * 128 + t;
    prank[t] = 0;
    for (int u = 0; u < 128; ++u) prank[t] += (rdy(8 + (u >> 4), u & 15) * 32 + rdy(u >> 4, u & 15)) * 128 + u < kt;
  }
  for (int xl = 0; xl < 8; ++xl) {
    int key[32], idx[32], ent[32];
    for (int t = 0; t < 32; ++t) {
      const int lr = (t >> 2) * 2 + (xl >> 2), lc = (t & 3) * 4 + (xl & 3), pr = prank[(lr & 7) * 16 + lc];
      key[t] = lr < 8 ? rdy(lr, lc) * 128 + pr : 10000 + pr;
      ent[t] = lr | (lc << 4) | (rdy(lr, lc) << 8) | (pr << 13);
      idx[t] = t;
    }
    for (int i = 1; i < 32; ++i)
      for (int j = i; j > 0 && key[idx[j]] < key[idx[j - 1]]; --j) { int s = idx[j]; idx[j] = idx[j - 1]; idx[j - 1] = s; }
    for (int slot = 0; slot < 32; ++slot) a.tile[xl * 32 + slot] = ent[idx[slot < 4 ? 28 + slot : slot - 4]];
  }
}

extern "C" void kernel_launch(int rank, int nranks, ncclComm_t comm, hipStream_t stream,
                              void* const* d_in, const int* in_sizes, int n_in,
                              void* d_out, int out_size, const int* in_dtypes, int out_dtype,
                              void* const* d_out_all, void* const* d_ws_all, size_t ws_bytes) {
  if (nranks != 8 || (unsigned)rank >= 8u || n_in < 2 || ws_bytes < WS_NEED || out_size != 4096 * 4096 ||
      in_sizes[0] != 4096 * 512 || in_sizes[1] != 512 * 4096) {
    fprintf(stderr, "kernel_launch: unexpected problem shape\n");
    return;
  }
  Args a;
  a.x = (const char*)d_in[0];
  a.w = (const char*)d_in[1];
  for (int l = 0; l < 8; ++l) { a.out[l] = (char*)d_out_all[rank ^ l]; a.ws[l] = (char*)d_ws_all[rank ^ l]; }
  a.rank = (u32)rank;
  static const unsigned char near_xcd[8][8] = {{0, 4, 2, 3, 6, 5, 1, 7}, {4, 0, 7, 5, 2, 3, 6, 1}, {2, 7, 0, 1, 4, 5, 3, 6}, {3, 5, 1, 0, 7, 4, 2, 6},
                                               {5, 2, 4, 7, 0, 1, 6, 3}, {6, 3, 5, 4, 1, 0, 7, 2}, {1, 6, 3, 2, 5, 7, 0, 4}, {7, 1, 6, 5, 3, 2, 4, 0}};
  a.xmap = 0;
  for (int p = 0; p < 8; ++p) {
    a.xmap |= (u32)(rank ^ p) << (4 * near_xcd[rank][p]);
    a.wsx[near_xcd[rank][p]] = (char*)d_ws_all[p];
  }
  build_tables(a);
  fused_gemm_ar_silu<<<256, 512, 0, stream>>>(a);
  hipError_t e = hipGetLastError();
  if (e != hipSuccess) fprintf(stderr, "launch failed: %s\n", hipGetErrorString(e));
}
```

```cpp
#include <hip/hip_runtime.h>
#include <rccl/rccl.h>
#include <cstdio>
#include <cstdint>

typedef unsigned int u32;
typedef unsigned long long u64;
typedef __attribute__((ext_vector_type(4))) float f32x4;
typedef __attribute__((ext_vector_type(4))) unsigned int u32x4;
typedef __attribute__((ext_vector_type(2))) unsigned int u32x2;
typedef __attribute__((ext_vector_type(4))) __bf16 bf16x4;
typedef __attribute__((ext_vector_type(8))) __bf16 bf16x8;
typedef __attribute__((ext_vector_type(2))) __bf16 bf16x2;
#define LAS __attribute__((address_space(3)))
#define GAS __attribute__((address_space(1)))
#define RLX __ATOMIC_RELAXED
#define SYS __HIP_MEMORY_SCOPE_SYSTEM
#define AGT __HIP_MEMORY_SCOPE_AGENT
#define WGP __HIP_MEMORY_SCOPE_WORKGROUP

constexpr size_t OFF_CTR = 0;
constexpr size_t OFF_ROLE = 1024;
constexpr size_t OFF_TICK = 5120;
constexpr size_t OFF_POOL = 5376;
constexpr size_t OFF_HOLD = 6144;
constexpr size_t OFF_INFLAG = 8192;
constexpr size_t OFF_PFLAG = 16384;
constexpr size_t OFF_QUEUE = 32768;
constexpr size_t OFF_SFLAG = 45056;
constexpr size_t OFF_XS = 1u << 20;
constexpr size_t OFF_WS = 9u << 20;
constexpr size_t OFF_PS = 17u << 20;
constexpr size_t OFF_PO = 21u << 20;
constexpr size_t OFF_SX = 25u << 20;
constexpr size_t OFF_SW = 29u << 20;
constexpr size_t WS_NEED = 33u << 20;

constexpr u64 DEADLINE = 150000000ull;
constexpr int NSTAGE = 4;
constexpr int STAGE_BYTES = 32768;
constexpr int LDS_CTL = NSTAGE * STAGE_BYTES;
constexpr int LDS_BYTES = LDS_CTL + 2048;

#define PIECE_ORDER 0, 16, 1, 17, 2, 18, 3, 19, 8, 20, 21, 9, 22, 23, 10, 24, 25, 11, 26, 27, 4, 28, 29, 30, 31, 5, 12, 6, 13, 7, 14, 15
constexpr u32 PIECES[32] = {PIECE_ORDER};
constexpr u64 pack_pieces(int first) {
  u64 v = 0;
  for (int i = 0; i < 8; ++i) v |= (u64)PIECES[first + i] << (8 * i);
  return v;
}
__device__ __forceinline__ u32 piece_at(u32 i) {
  constexpr u64 p0 = pack_pieces(0), p1 = pack_pieces(8), p2 = pack_pieces(16), p3 = pack_pieces(24);
  const u64 v = i < 16 ? (i < 8 ? p0 : p1) : (i < 24 ? p2 : p3);
  return (u32)(v >> (8 * (i & 7))) & 31u;
}

constexpr u64 pack_pos(int first) {
  u64 v = 0;
  for (int i = 0; i < 32; ++i) if ((int)PIECES[i] >= first && (int)PIECES[i] < first + 8) v |= (u64)i << (8 * (PIECES[i] - first));
  return v;
}
__device__ __forceinline__ u32 pos_of(u32 c) {
  constexpr u64 p0 = pack_pos(0), p1 = pack_pos(8), p2 = pack_pos(16), p3 = pack_pos(24);
  const u64 v = c < 16 ? (c < 8 ? p0 : p1) : (c < 24 ? p2 : p3);
  return (u32)(v >> (8 * (c & 7))) & 31u;
}
#define NP0 3
#define RP 100
#define ANN_SLEEP 32
#define PAIR_OPEN 64
#define FINE 28
constexpr u32 NBIG = RP - FINE, NPOOL = NBIG + 2 * FINE;
struct StageOrder { u64 v[4]; };
constexpr StageOrder make_stage_order() {
  StageOrder o = {{0, 0, 0, 0}};
  bool seen[24] = {};
  int n = 0;
  for (int k = NP0; k < 32 + NP0; ++k) {
    const int c = (int)PIECES[k & 31], cl = c < 16 ? (c & 7) : c - 8;
    if (seen[cl]) continue;
    seen[cl] = true;
    for (int e = 0; e < (c < 16 ? 2 : 1); ++e, ++n) o.v[n >> 3] |= (u64)(c < 16 ? cl * 2 + e : c) << (8 * (n & 7));
  }
  return o;
}
__device__ __forceinline__ u32 stage_class(u32 i) {
  constexpr StageOrder o = make_stage_order();
  constexpr u64 p0 = o.v[0], p1 = o.v[1], p2 = o.v[2], p3 = o.v[3];
  const u64 v = i < 16 ? (i < 8 ? p0 : p1) : (i < 24 ? p2 : p3);
  return (u32)(v >> (8 * (i & 7))) & 31u;
}

struct Args {
  const char* x;
  const char* w;
  u32 rank;
  u32 xmap;
  char* ws[8];
  char* out[8];
  char* wsx[8];
  u32 tile[256];
};

__device__ __forceinline__ u32 grow0(u32 r, u32 lr) {
  return ((r >> 1) & 1) * 2048 + ((lr >> 3) ^ (r >> 2) ^ 1) * 1024 + (lr & 7) * 128;
}
__device__ __forceinline__ u32 gcol0(u32 r, u32 lc) { return (r & 1) * 2048 + lc * 128; }
__device__ __forceinline__ u32 tile_off(u32 r, u32 id) {
  return grow0(r, 8 + ((id >> 4) & 7)) * 8192 + gcol0(r, id & 15) * 2;
}

__device__ __forceinline__ void st16_sys(u32 voff, u32x4 v, char* sbase) {
  asm volatile("global_store_dwordx4 %0, %1, %2 sc0 sc1\n\ts_nop 1" :: "v"(voff), "v"(v), "s"(sbase) : "memory");
}
__device__ __forceinline__ void st16_agent(u32 voff, u32x4 v, char* sbase) {
  asm volatile("global_store_dwordx4 %0, %1, %2 sc1\n\ts_nop 1" :: "v"(voff), "v"(v), "s"(sbase) : "memory");
}
__device__ __forceinline__ bool expired(u64 t0) { return __builtin_amdgcn_s_memrealtime() - t0 > DEADLINE; }
#define WAIT_VM0() asm volatile("s_waitcnt vmcnt(0)" ::: "memory")
#define WG_BARRIER() asm volatile("s_waitcnt lgkmcnt(0)\n\ts_barrier" ::: "memory")

#define WIN 8
#define NST 4
constexpr u32 RING = 24, RING0 = 16384;
#define AHEAD 7
#define START_DELAY 100
#define START_SLOPE 100
#define LEAD0 4
#define STAGE0 (LEAD0 + 4 * NP0)
#define STAGE_SPLIT 24
#define STAGE_DELAY 80
constexpr u32 C_CNT = 0;
constexpr u32 C_AVAIL = 128;
constexpr u32 C_LISTED = 136, C_TOTAL = 137, C_TICKET = 138, C_BAD = 139, C_MASK = 140, C_LANDED = 141, C_TAKEN = 142, C_HELD = 143, C_ROLE = 144;
constexpr u32 C_FREE = 160;
constexpr u32 C_LIST = 192, C_LIST2 = 320;

__device__ __forceinline__ void glds4v(u32 v0, u32 v1, u32 v2, u32 v3, const char* sbase, u32 lds) {
  asm volatile(
      "s_mov_b32 m0, %5\n\ts_nop 0\n\t"
      "global_load_lds_dwordx4 %0, %4\n\t"
      "global_load_lds_dwordx4 %1, %4 offset:1024\n\t"
      "global_load_lds_dwordx4 %2, %4 offset:2048\n\t"
      "global_load_lds_dwordx4 %3, %4 offset:3072"
      :: "v"(v0), "v"(v1), "v"(v2), "v"(v3), "s"(sbase), "s"(lds) : "memory");
}
__device__ __forceinline__ void pull4l(u32 v0, u32 v1, u32 v2, u32 v3, const char* sbase, u32 lds) {
  asm volatile(
      "s_mov_b32 m0, %5\n\ts_waitcnt vmcnt(%6)\n\t"
      "global_load_lds_dwordx4 %0, %4 sc0 sc1\n\t"
      "global_load_lds_dwordx4 %1, %4 offset:1024 sc0 sc1\n\t"
      "global_load_lds_dwordx4 %2, %4 offset:2048 sc0 sc1\n\t"
      "global_load_lds_dwordx4 %3, %4 offset:3072 sc0 sc1"
      :: "v"(v0), "v"(v1), "v"(v2), "v"(v3), "s"(sbase), "s"(lds), "n"(WIN) : "memory");
}
__device__ __forceinline__ u32 src_x(u32 s, u32 lane) {
  return (8 * s + (lane >> 3)) * 1024 + (((lane & 7) ^ (4 * (s & 1) + (lane >> 4))) * 16) - s * 1024;
}
__device__ __forceinline__ u32 src_w(u32 s, u32 lane) {
  return (4 * s + (lane >> 4)) * 8192 + (((lane & 15) ^ (((lane >> 4) | ((s >> 1) << 2)) << 1)) * 16) - s * 1024;
}
__device__ __forceinline__ u32 lds_get(LAS u32* p) {
  asm volatile("" ::: "memory");
  const u32 v = __builtin_amdgcn_readfirstlane(__hip_atomic_load(p, RLX, WGP));
  asm volatile("" ::: "memory");
  return v;
}
__device__ __forceinline__ void lds_set(LAS u32* p, u32 v) {
  asm volatile("" ::: "memory");
  __hip_atomic_store(p, v, RLX, WGP);
  asm volatile("" ::: "memory");
}
__device__ __forceinline__ u32 lds_add(LAS u32* p, u32 lane) {
  u32 t = 0;
  if (lane == 0) t = __hip_atomic_fetch_add(p, 1u, RLX, WGP);
  return __builtin_amdgcn_readfirstlane(t);
}

__device__ __forceinline__ void ld4(u32x4& d0, u32x4& d1, u32x4& d2, u32x4& d3, u32 v0, u32 v1, u32 v2, u32 v3, const char* sb) {
  asm volatile("global_load_dwordx4 %0, %4, %8\n\tglobal_load_dwordx4 %1, %5, %8 offset:1024\n\t"
               "global_load_dwordx4 %2, %6, %8 offset:2048\n\tglobal_load_dwordx4 %3, %7, %8 offset:3072\n\ts_waitcnt vmcnt(0)"
               : "=&v"(d0), "=&v"(d1), "=&v"(d2), "=&v"(d3) : "v"(v0), "v"(v1), "v"(v2), "v"(v3), "s"(sb) : "memory");
}
__device__ __forceinline__ u32 kb_first(u32 xl) { return xl < 4 ? 4 + 4 * xl : 4 * xl; }

__device__ __forceinline__ void push_leading(const Args& a, char* db, u32 p, u32 xl, u32 i, u32 j, u32 kb, u32 lane) {
  const u32 pc = piece_at(i), k0 = (p >> 2) * 256 + kb * 64;
  u32x4 d0, d1, d2, d3;
  if (pc >> 4) ld4(d0, d1, d2, d3, src_w(0, lane), src_w(1, lane), src_w(2, lane), src_w(3, lane),
                   a.w + ((k0 + j * 16) * 8192 + gcol0(p, pc & 15) * 2));
  else ld4(d0, d1, d2, d3, src_x(0, lane), src_x(1, lane), src_x(2, lane), src_x(3, lane),
           a.x + ((grow0(p, pc) + j * 32) * 1024 + k0 * 2));
  db += OFF_XS + (pc * 32 + kb_first(xl) + kb) * 16384 + j * 4096;
  const u32 vd = lane * 16;
  st16_sys(vd, d0, db); st16_sys(vd, d1, db + 1024); st16_sys(vd, d2, db + 2048); st16_sys(vd, d3, db + 3072);
}

__device__ __forceinline__ void stage_unit(const Args& a, u32 rank, u32 unit, u32 wv, u32 lane, u32 calls) {
  const u32 j = unit & 3, sub = (unit >> 2) & 1, cls = stage_class(unit >> 3);
  const bool isw = cls >= 16;
  const u32 ch = isw ? sub * 16 + (cls & 15) : (cls & 1) * 16 + sub * 8 + (cls >> 1);
  u32x4 d0, d1, d2, d3;
  if (isw) ld4(d0, d1, d2, d3, src_w(0, lane), src_w(1, lane), src_w(2, lane), src_w(3, lane), a.w + ((wv * 64 + j * 16) * 8192 + ch * 256));
  else ld4(d0, d1, d2, d3, src_x(0, lane), src_x(1, lane), src_x(2, lane), src_x(3, lane), a.x + ((ch * 128 + j * 32) * 1024 + wv * 128));
  char* db = a.ws[0] + ((isw ? OFF_SW : OFF_SX) + (ch * 8 + wv) * 16384 + j * 4096);
  const u32 vd = lane * 16;
  st16_sys(vd, d0, db); st16_sys(vd, d1, db + 1024); st16_sys(vd, d2, db + 2048); st16_sys(vd, d3, db + 3072);
  const u32 p = rank ^ (lane & 7);
  const u32 pos = pos_of(isw ? cls : (sub ^ (p >> 2) ^ 1) * 8 + (cls >> 1));
  u32 tell = wv == 0 && lane < 8 && (lane & 3) && (isw ? p & 1 : (p >> 1) & 1) == ch >> 4 && pos >= NP0;
  char *w1 = a.ws[1], *w2 = a.ws[2], *w3 = a.ws[3], *w5 = a.ws[5], *w6 = a.ws[6], *w7 = a.ws[7];
  asm volatile("" : "+s"(w1), "+s"(w2), "+s"(w3), "+s"(w5), "+s"(w6), "+s"(w7));
  char* pw = lane & 4 ? (lane & 2 ? (lane & 1 ? w7 : w6) : w5) : (lane & 2 ? (lane & 1 ? w3 : w2) : w1);
  pw += OFF_SFLAG + ((lane & 7) * 4 + j) * 128 + pos * 4;
  asm volatile("s_waitcnt vmcnt(0)\n\ts_barrier" : "+v"(pw), "+v"(tell), "+s"(calls) :: "memory");
  if (tell) __hip_atomic_store((GAS u32*)pw, calls + 1, RLX, SYS);
}

__device__ __forceinline__ void link_engine(const Args& a, u32 rank, u32 xl, u32 j, u32 wv, u32 lane, u32 seq, u64 t0,
                                            LAS char* shm, u32 lds0) {
  LAS u32* ctl = (LAS u32*)(shm + LDS_CTL);
  const bool pair = xl == 4;
  const u32 f0 = (lane >> 4) * 8192 + (lane & 15) * 16, vl = lane * 16;
  char* const ws = a.ws[0];
  volatile LAS u32x2* list = (volatile LAS u32x2*)shm;
  if (wv == 5) {
    const u32 isrc = OFF_SX + (rank >> 2) * 4 * 16384 + j * 4096;
    const u32 idst = OFF_XS + kb_first(xl) * 16384 + j * 4096;
    u32 gi = 0, ni = pair ? 32u : NP0, nf = pair ? RP : 0u;
    u64 left0 = pair ? ~0ull : 0ull, left1 = left0;
    bool pool = true;
#pragma unroll 1
    while (pool || nf < 128 || (left0 | left1)) {
      asm volatile("" ::: "memory");
      const u32x4 av = *(volatile LAS u32x4*)(ctl + C_AVAIL), pm = *(volatile LAS u32x4*)(ctl + C_AVAIL + 4);
      const bool near = gi < lds_get(ctl + C_TICKET) + AHEAD;
      const u64 a0 = left0 & (((u64)__builtin_amdgcn_readfirstlane(pm.y) << 32) | (u32)__builtin_amdgcn_readfirstlane(pm.x));
      const u64 a1 = left1 & (((u64)__builtin_amdgcn_readfirstlane(pm.w) << 32) | (u32)__builtin_amdgcn_readfirstlane(pm.z));
      const u32 cf = __builtin_amdgcn_readfirstlane(av.y);
      u32 src = 0, dst = 0, n = 0, step = 16 * 8192;
      if (!near) {
      } else if (a0 | a1) {
        const u32 id = a0 ? __builtin_ctzll(a0) : 64 + __builtin_ctzll(a1);
        if (a0) left0 &= ~(1ull << id); else left1 &= ~(1ull << (id - 64));
        src = OFF_PO + id * 32768 + j * 8192; dst = OFF_PS + id * 32768 + j * 8192 + id * 2 + 1; n = 2; step = 4096;
      } else if (ni < 32) {
        if (ni < __builtin_amdgcn_readfirstlane(av.x)) {
          const u32 pc = piece_at(ni);
          src = isrc + (pc >> 4 ? 4096 + gcol0(rank, pc & 15) : grow0(rank, pc)) * 1024;
          dst = idst + pc * 32 * 16384 + ni * 2 + 1; n = 4; step = 16384;
          ++ni;
        }
      } else if (nf < cf) {
        src = lds_get(ctl + C_LIST + nf) + j * 32 * 8192; dst = src + (nf < RP ? 2 : 0); n = 2;
        ++nf;
      } else if (pool && __builtin_amdgcn_readfirstlane(av.z) && (!(left0 | left1) || cf >= RP)) {
        u32 t = 0;
        if (lane == 0) t = __hip_atomic_fetch_add((GAS u32*)(ws + OFF_POOL), 1u, RLX, AGT);
        t = __builtin_amdgcn_readfirstlane(t) - (seq - 1) * (NPOOL + 28);
        if (t < NPOOL) {
          const u32 ht = t < NBIG ? 2 * t : t + NBIG;
          if (pair) while (lds_get(ctl + C_AVAIL + 1) <= (ht >> 1)) __builtin_amdgcn_s_sleep(1);
          src = dst = lds_get(ctl + (pair ? C_LIST : C_LIST2) + (ht >> 1)) + (ht & 1) * 64 * 8192;
          n = t < NBIG ? 8 : 4;
        } else {
          pool = false;
        }
      }
      if (n) {
        if (lane < n) list[gi + lane] = (u32x2){src + lane * step, dst + lane * step};
        gi += n;
        lds_set(ctl + C_LISTED, gi);
      } else {
        __builtin_amdgcn_s_sleep(1);
      }
    }
    lds_set(ctl + C_TOTAL, gi + 1);
  } else if (wv == 0) {
    constexpr u32 BACK = (WIN + 3) / 4;
    const char* const pws = a.ws[xl];
    const char* const fsrc = a.out[xl];
    u32 slot = 0, pub = 0, gi = 0;
    __builtin_amdgcn_s_setprio(3);
#pragma unroll 1
    for (;; ++gi) {
      u32 src, lin;
      bool end = false;
      for (u32 tries = 0;; ++tries) {
        asm volatile("" ::: "memory");
        const u32x2 lt = *(volatile LAS u32x2*)(ctl + C_LISTED);
        const u32 fr = __hip_atomic_load(ctl + C_FREE + slot, RLX, WGP);
        asm volatile("" ::: "memory");
        const u32x2 d = list[gi];
        src = __builtin_amdgcn_readfirstlane(d.x); lin = __builtin_amdgcn_readfirstlane(d.y) & 1;
        if (__builtin_amdgcn_readfirstlane(lt.x) > gi && __builtin_amdgcn_readfirstlane(fr) + RING > gi) break;
        if (__builtin_amdgcn_readfirstlane(lt.y) == gi + 1) { end = true; break; }
        if (tries == 3) { WAIT_VM0(); lds_set(ctl + C_LANDED, gi); pub = gi; }
        __builtin_amdgcn_s_sleep(1);
      }
      if (end) break;
      if (lin) pull4l(vl, vl, vl, vl, pws + src, lds0 + RING0 + slot * 4096);
      else pull4l(f0, f0 + 31744, f0 + 63488, f0 + 95232, fsrc + src, lds0 + RING0 + slot * 4096);
      if (gi > pub + BACK) { pub = gi - BACK; lds_set(ctl + C_LANDED, pub); }
      lds_set(ctl + C_TICKET, gi + 1);
      slot = slot + 1 == RING ? 0 : slot + 1;
    }
    asm volatile("s_waitcnt vmcnt(8)" ::: "memory");
    if (gi > pub + 2) lds_set(ctl + C_LANDED, gi - 2);
    asm volatile("s_waitcnt vmcnt(4)" ::: "memory");
    if (gi > pub + 1) lds_set(ctl + C_LANDED, gi - 1);
    WAIT_VM0();
    lds_set(ctl + C_LANDED, gi);
  } else if (wv <= NST) {
    char* const fdst = a.out[0];
    u32 landed = 0, prev = 0;
    auto done = [&]() {
      if (prev & 1) {
        const u32 c = (prev >> 1) & 127;
        if (lds_add(ctl + C_CNT + c, lane) == 1 && pair && lane == 0)
          __hip_atomic_store((GAS u32*)(ws + OFF_PFLAG + c * 64) + j, seq, RLX, AGT);
      } else if (prev & 2) {
        if (lds_add(ctl + C_HELD, lane) == 2 * RP - 1 && lane == 0)
          __hip_atomic_store((GAS u32*)(a.ws[xl ^ 4] + OFF_HOLD + (xl ^ 4) * 128) + j, seq, RLX, SYS);
      }
      prev = 0;
    };
#pragma unroll 1
    for (;;) {
      const u32 gi = lds_add(ctl + C_TAKEN, lane);
      if (landed <= gi) {
        landed = lds_get(ctl + C_LANDED);
        if (landed <= gi) {
          WAIT_VM0();
          done();
          bool end = false;
          do {
            end = lds_get(ctl + C_TOTAL) - 1 <= gi;
            if (end) break;
            __builtin_amdgcn_s_sleep(1);
            landed = lds_get(ctl + C_LANDED);
          } while (landed <= gi);
          if (end) break;
        }
      }
      const u32 slot = gi % RING;
      const u32 desc = __builtin_amdgcn_readfirstlane(list[gi].y);
      const LAS u32x4* rs = (const LAS u32x4*)(shm + RING0 + slot * 4096) + lane;
      u32x4 d0 = rs[0];
      const u32x4 d1 = rs[64], d2 = rs[128], d3 = rs[192];
      lds_set(ctl + C_FREE + slot, gi + 1);
      if (lds_get(ctl + C_BAD)) d0.x = 0x7fc07fc0u;
      WAIT_VM0();
      if (desc & 1) {
        char* db = ws + (desc & ~4095u);
        st16_agent(vl, d0, db); st16_agent(vl, d1, db + 1024); st16_agent(vl, d2, db + 2048); st16_agent(vl, d3, db + 3072);
      } else {
        char* db = fdst + (desc & ~255u);
        st16_sys(f0, d0, db); st16_sys(f0, d1, db + 32768); st16_sys(f0, d2, db + 65536); st16_sys(f0, d3, db + 98304);
      }
      done();
      prev = desc;
    }
  } else if (wv == 6) {
    const u32 tag = seq & 0xffffff;
    const GAS u32* qf = (const GAS u32*)(ws + OFF_QUEUE + xl * 1024) + lane;
    const GAS u32* qs = pair ? qf + 128 : lane < 32 ? (const GAS u32*)(ws + OFF_SFLAG + (xl * 4 + j) * 128) + lane
                                                    : (const GAS u32*)(ws + OFF_HOLD + xl * 128) + (lane & 3);
    u32 nf = 0, open = 0;
    for (;;) {
      u32 e0 = __hip_atomic_load(qf, RLX, SYS), e1 = __hip_atomic_load(qf + 64, RLX, SYS), e2 = __hip_atomic_load(qs, RLX, SYS);
      u32 e3 = seq;
      if (pair) e3 = __hip_atomic_load(qs + 64, RLX, SYS);
      const bool late = expired(t0);
      if (late) { lds_set(ctl + C_BAD, 1); e0 = (seq << 8) | lane; e1 = (seq << 8) | (64 + lane); e2 = e3 = seq; }
      const u64 n0 = __ballot((e0 >> 8) != tag), n1 = __ballot((e1 >> 8) != tag);
      const u64 m2 = __ballot(e2 >= seq || (!pair && lane < NP0)), m3 = __ballot(e3 >= seq);
      const u32 cf = n0 ? (u32)__builtin_ctzll(n0) : (n1 ? 64u + (u32)__builtin_ctzll(n1) : 128u);
      const u32 staged = (u32)m2 == ~0u ? 32u : (u32)__builtin_ctz(~(u32)m2);
      if (cf != nf) {
        if (lane >= nf) ctl[C_LIST + lane] = tile_off(rank ^ xl, e0);
        if (64 + lane >= nf) ctl[C_LIST + 64 + lane] = tile_off(rank ^ xl, e1);
        nf = cf;
      }
      if (pair) {
        open = cf >= PAIR_OPEN;
        *(volatile LAS u32x4*)(ctl + C_AVAIL + 4) = (u32x4){(u32)m2, (u32)(m2 >> 32), (u32)m3, (u32)(m3 >> 32)};
      } else if (!open && (u32)(m2 >> 32) == ~0u) {
        const GAS u32* qp = (const GAS u32*)(ws + OFF_QUEUE + 4 * 1024) + lane;
        const u32 g0 = __hip_atomic_load(qp, RLX, SYS), g1 = __hip_atomic_load(qp + 64, RLX, SYS);
        ctl[C_LIST2 + lane] = tile_off(rank ^ 4, g0);
        ctl[C_LIST2 + 64 + lane] = tile_off(rank ^ 4, g1);
        open = late || !__ballot(((g0 >> 8) != tag && lane < RP) || ((g1 >> 8) != tag && 64 + lane < RP));
      }
      asm volatile("" ::: "memory");
      *(volatile LAS u32x4*)(ctl + C_AVAIL) = (u32x4){staged, cf, open, 0u};
      if (cf == 128 && open && (!pair || (m2 & m3) == ~0ull)) break;
      if (!pair && staged == 32) __builtin_amdgcn_s_sleep(8);
    }
  } else if (wv == 7 && !pair) {
    GAS u32* const flag = (GAS u32*)(ws + OFF_INFLAG + xl * 16 + j * 4);
    const GAS u32* lf = (const GAS u32*)(ws + OFF_SFLAG + (xl * 4 + j) * 128) + (lane & 31);
    while (__ballot(__hip_atomic_load(lf, RLX, SYS) < seq && lane < NP0)) {
      if (expired(t0)) { lds_set(ctl + C_BAD, 1); break; }
      __builtin_amdgcn_s_sleep(1);
    }
    if (lane < NP0) __hip_atomic_store(flag + lane * 64, seq, RLX, AGT);
#pragma unroll 1
    for (u32 pi = NP0; pi < 32; ++pi) {
      while (lds_get(ctl + C_CNT + pi) < 4) __builtin_amdgcn_s_sleep(1);
      if (lane == 0) __hip_atomic_store(flag + pi * 64, seq, RLX, AGT);
    }
  }
}

__device__ __forceinline__ void announcer(const Args& a, u32 l, u32 lane, u32 seq, u64 t0) {
  const GAS u32* src = (const GAS u32*)(a.ws[0] + OFF_QUEUE) + lane;
  GAS u32* dst = (GAS u32*)(a.ws[l] + OFF_QUEUE + l * 1024) + lane;
  const u32 tag = seq & 0xffffff;
  bool s0 = false, s1 = false;
  while (__ballot(!s0 || !s1) && !expired(t0)) {
    const u32 e0 = __hip_atomic_load(src, RLX, AGT), e1 = __hip_atomic_load(src + 64, RLX, AGT);
    if (!s0 && (e0 >> 8) == tag) { __hip_atomic_store(dst, e0, RLX, SYS); s0 = true; }
    if (!s1 && (e1 >> 8) == tag) { __hip_atomic_store(dst + 64, e1, RLX, SYS); s1 = true; }
    __builtin_amdgcn_s_sleep(ANN_SLEEP);
  }
}

__device__ __forceinline__ u32 pack2(float lo, float hi) {
  bf16x2 v = {(__bf16)lo, (__bf16)hi};
  return __builtin_bit_cast(u32, v);
}
__device__ __forceinline__ float silu(float y) {
  const float c = __builtin_fminf(__builtin_fmaxf(y, -60.f), 60.f);
  return y * __builtin_amdgcn_rcpf(1.f + __builtin_amdgcn_exp2f(c * -1.44269504088896341f));
}

__global__ void __launch_bounds__(512, 2) fused_gemm_ar_silu(const Args a) {
  __shared__ __attribute__((aligned(1024))) char shm[LDS_BYTES];
  const u32 tid = threadIdx.x, lane = tid & 63;
  const u32 wv = __builtin_amdgcn_readfirstlane(tid >> 6);
  const u32 b = blockIdx.x;
  u32 role;
  asm volatile("s_getreg_b32 %0, hwreg(HW_REG_XCC_ID)" : "=s"(role));
  role = (role & 7) | (b & ~7u);
  char* wsp = a.wsx[role & 7];
  asm volatile("" : "+s"(wsp) : "s"(a.x), "s"(a.w), "s"(a.rank), "s"(a.xmap), "s"(a.ws[0]));
  LAS u32* lctr = (LAS u32*)(shm + LDS_CTL);
  const u32 lds0 = (u32)(size_t)(LAS char*)shm;
  const u64 t0 = __builtin_amdgcn_s_memrealtime();
  const u32 rank = a.rank & 7;
  char* const ws = a.ws[0];
  GAS u32* ctr = (GAS u32*)(ws + OFF_CTR) + b;
  u32 calls = *(const __attribute__((address_space(4))) u32*)(size_t)ctr;
  if (wv == 7) { lctr[lane] = 0; lctr[64 + lane] = 0; lctr[128 + lane] = 0; }
  u32 xl, slot, seq;
  auto startup = [&](bool claim) {
    xl = (a.xmap >> (4 * (role & 7))) & 7;
    slot = role >> 3;
    const u32 lead = (slot - LEAD0) >> 2;
    const bool is_lead = (xl & 3) && wv < 4 && lead < NP0;
    if (is_lead) push_leading(a, wsp, rank ^ xl, xl, lead, slot & 3, wv, lane);
    if (!(xl & 3)) {
      if (slot >= 32 - STAGE_SPLIT) stage_unit(a, rank, (slot - (32 - STAGE_SPLIT)) * 2 + (xl >> 2), wv, lane, calls);
    } else if (slot >= STAGE0) {
      while (__builtin_amdgcn_s_memrealtime() - t0 < STAGE_DELAY) __builtin_amdgcn_s_sleep(2);
      for (u32 u = 2 * STAGE_SPLIT + (slot - STAGE0) * 6 + (xl < 4 ? xl - 1 : xl - 2); u < 256; u += (32 - STAGE0) * 6)
        stage_unit(a, rank, u, wv, lane, calls);
    }
    asm volatile("s_waitcnt vmcnt(0)" : "+s"(calls) :: "memory");
    seq = calls + 1;
    if (claim && tid == 448) {
      u32 r = role;
      for (u32 i = 0; i < 256 && __hip_atomic_exchange((GAS u32*)(ws + OFF_ROLE) + r, seq, RLX, AGT) == seq; ++i) r = (r + 1) & 255;
      lctr[C_ROLE] = r;
    }
    WG_BARRIER();
    if (is_lead && tid == 0)
      __hip_atomic_store((GAS u32*)(wsp + OFF_SFLAG + (xl * 4 + (slot & 3)) * 128) + lead, seq, RLX, SYS);
  };
  startup(true);
  const u32 got = lds_get(lctr + C_ROLE);
  if (__builtin_expect(got != role, 0)) { role = got; wsp = a.wsx[role & 7]; startup(false); }
  const bool engine = xl != 0 && slot < 4;
  if (tid == 0) __hip_atomic_store(ctr, seq, RLX, AGT);
  if (engine) {
    link_engine(a, rank, xl, slot, wv, lane, seq, t0, (LAS char*)shm, lds0);
    return;
  }

  if (xl == 0 && slot == 0) {
    if (wv) announcer(a, wv, lane, seq, t0);
    return;
  }
#pragma unroll 1
  for (u32 ts = slot;; ts -= 4) {
    const u32 te = a.tile[xl * 32 + ts];
    const u32 lr = te & 15, lc = (te >> 4) & 15, rpos = (te >> 8) & 31, tix = (te >> 13) & 127;
    const u32 kind = lr >> 3;
    const u32 row0 = grow0(rank, lr), col0 = gcol0(rank, lc);
    const u32 vl = lane * 16;

    if (wv >= 4) {
      const u32 w4 = wv - 4;
      const char* xo = a.x + (size_t)(row0 + w4 * 32) * 1024;
      const char* wo = a.w + ((size_t)w4 * 16 * 8192 + col0 * 2);
      const char* xs = ws + (OFF_XS + (size_t)lr * 32 * 16384 + w4 * 4096);
      const char* wsg = ws + (OFF_WS + (size_t)lc * 32 * 16384 + w4 * 4096);
      const u32 lq = lds0 + w4 * 4096;
      const GAS u32x4* inflag = (const GAS u32x4*)(ws + OFF_INFLAG + rpos * 256 + (lane & 7) * 16);
      const GAS u32x4* lastflag = (lane & 7) ? inflag : (const GAS u32x4*)(ws + OFF_PFLAG + tix * 64);
      { const u32 d = START_DELAY + rpos * START_SLOPE; while (__builtin_amdgcn_s_memrealtime() - t0 < d) __builtin_amdgcn_s_sleep(4); }
      if (wv == 7 && ts == slot) {
        __builtin_amdgcn_fence(__ATOMIC_ACQUIRE, "");
        WAIT_VM0();
      }
      __builtin_amdgcn_s_setprio(2);
      {
        const u32 x0 = src_x(0, lane), x1 = src_x(1, lane), x2 = src_x(2, lane), x3 = src_x(3, lane);
        const u32 y0 = src_w(0, lane), y1 = src_w(1, lane), y2 = src_w(2, lane), y3 = src_w(3, lane);
#pragma unroll
        for (u32 kb = 0; kb < 8; ++kb) {
          if (kb >= 3) asm volatile("s_waitcnt vmcnt(16)\n\ts_barrier" ::: "memory");
          glds4v(x0, x1, x2, x3, xo + kb * 128, lq + (kb & 3) * STAGE_BYTES);
          glds4v(y0, y1, y2, y3, wo + (size_t)kb * 64 * 8192, lq + (kb & 3) * STAGE_BYTES + 16384);
        }
      }
      const u32 nst = 32 + kind;
      u32 avail = 8, issued = 8, ord = 0, seen = 0;
      auto refresh = [&]() {
        if (w4 == 0) {
          u32x4 f;
          const GAS u32x4* fp = issued >= 20 ? lastflag : inflag;
          asm volatile("global_load_dwordx4 %0, %1, off sc0 sc1\n\ts_waitcnt vmcnt(0)" : "=&v"(f) : "v"(fp) : "memory");
          u32 mask = (u32)__ballot(f.x >= seq && f.y >= seq && f.z >= seq && f.w >= seq);
          if (expired(t0)) { lds_set(lctr + C_BAD, 1); mask = 0xff; }
          ord |= (mask & 1) << 21;
          for (u32 m = mask & 0xee & ~seen; m; m &= m - 1) {
            const u32 l = __builtin_ctz(m);
            ord = (ord | ((l < 4 ? l + 1 : l) << (3 * ((ord >> 18) & 7)))) + (1u << 18);
          }
          seen |= mask;
          lds_set(lctr + C_MASK, ord);
        } else {
          ord = lds_get(lctr + C_MASK);
        }
        const u32 n = (ord >> 18) & 7;
        avail = 8 + 4 * n + (n == 6 ? (ord >> 21) & kind : 0);
      };
#pragma unroll 1
      for (u32 s = 5; s <= 32; ++s) {
        if (issued == s + 2 && !(issued & 3) && issued + 4 <= avail) {
          const u32 off = ((ord >> (3 * ((issued - 8) >> 2))) & 7) * 65536;
          const char* sx = xs + off;
          const char* sw = wsg + off;
#pragma unroll
          for (u32 i = 0; i < 4; ++i) {
            glds4v(vl, vl, vl, vl, sx + i * 16384, lq + i * STAGE_BYTES);
            glds4v(vl, vl, vl, vl, sw + i * 16384, lq + i * STAGE_BYTES + 16384);
            asm volatile("s_waitcnt vmcnt(16)\n\ts_barrier" ::: "memory");
          }
          issued += 4; s += 3;
          continue;
        }
        while (issued <= s + 2) {
          if (issued >= avail) {
            if (issued >= nst) break;
            refresh();
            while (issued >= avail && issued == s) {
              if (issued == 32) __builtin_amdgcn_s_sleep(8); else __builtin_amdgcn_s_sleep(2);
              refresh();
            }
            if (issued >= avail) break;
          }
          const u32 l = lq + (issued & 3) * STAGE_BYTES;
          const u32 kk = ((ord >> (3 * ((issued - 8) >> 2))) & 7) * 4 + (issued & 3);
          const char* sx = xs + kk * 16384;
          const char* sw = wsg + kk * 16384;
          if (issued == 32) { sx = ws + (OFF_PS + (size_t)tix * 32768 + w4 * 4096); sw = sx + 16384; }
          glds4v(vl, vl, vl, vl, sx, l);
          glds4v(vl, vl, vl, vl, sw, l + 16384);
          ++issued;
        }
        const u32 ahead = issued - s;
        if (ahead >= 3) asm volatile("s_waitcnt vmcnt(16)\n\ts_barrier" ::: "memory");
        else if (ahead == 2) asm volatile("s_waitcnt vmcnt(8)\n\ts_barrier" ::: "memory");
        else asm volatile("s_waitcnt vmcnt(0) lgkmcnt(0)\n\ts_barrier" ::: "memory");
      }
      __builtin_amdgcn_s_setprio(0);
      u32 pos = 0;
      if (kind && wv == 4 && lane == 0) pos = __hip_atomic_fetch_add((GAS u32*)(ws + OFF_TICK), 1u, RLX, AGT);
      pos = __builtin_amdgcn_readfirstlane(pos);
      if (wv == 4) lds_set(lctr + C_MASK, 0);
      WG_BARRIER();
      if (wv == 4) {
        const u32 e = (seq << 8) | ((lr & 7) * 16 + lc);
        if (kind) {
          if (lane == 0) __hip_atomic_store((GAS u32*)(ws + OFF_QUEUE) + (pos & 127), e, RLX, AGT);
        } else if (lane == 0) {
          __hip_atomic_store((GAS u32*)(a.ws[4] + OFF_QUEUE + 4 * 1024 + 512) + tix, seq, RLX, SYS);
        }
      }
    } else {
      const u32 wn = wv & 1, wm = wv >> 1;
      const u32 g = lane >> 4, q = (lane >> 2) & 3, pp = lane & 3;
      u32 ax[2];
#pragma unroll
      for (int ks = 0; ks < 2; ++ks) ax[ks] = (wm * 64 + (lane & 15)) * 128 + (((4 * ks + g) ^ ((lane & 15) >> 1)) * 16);
      u32 aw[4];
#pragma unroll
      for (int nt = 0; nt < 4; ++nt)
        aw[nt] = 16384 + (8 * g + q) * 256 + (((2 * (wn * 4 + nt) + (pp >> 1)) ^ ((q | ((g & 1) << 2)) << 1)) * 16) + (pp & 1) * 8;

      f32x4 acc[4][4];
#pragma unroll
      for (int i = 0; i < 4; ++i)
#pragma unroll
        for (int jj = 0; jj < 4; ++jj) acc[i][jj] = (f32x4){0.f, 0.f, 0.f, 0.f};
      bf16x8 xa[4], wa[4], xb[4], wb[4];
      auto rd = [&](bf16x8* fx, bf16x8* fw, u32 t, int ks) {
        const LAS char* st = (const LAS char*)shm + (t & (NSTAGE - 1)) * STAGE_BYTES;
#pragma unroll
        for (int i = 0; i < 4; ++i) {
          const bf16x4 lo = __builtin_amdgcn_ds_read_tr16_b64_v4bf16((LAS bf16x4*)(st + aw[i] + ks * 8192));
          const bf16x4 hi = __builtin_amdgcn_ds_read_tr16_b64_v4bf16((LAS bf16x4*)(st + aw[i] + ks * 8192 + 1024));
          fw[i] = __builtin_shufflevector(lo, hi, 0, 1, 2, 3, 4, 5, 6, 7);
          fx[i] = *(const LAS bf16x8*)(st + ax[ks] + i * 2048);
        }
      };
      auto mm = [&](const bf16x8* fx, const bf16x8* fw) {
#pragma unroll
        for (int mt = 0; mt < 4; ++mt)
#pragma unroll
          for (int nt = 0; nt < 4; ++nt)
            acc[nt][mt] = __builtin_amdgcn_mfma_f32_16x16x32_bf16(fw[nt], fx[mt], acc[nt][mt], 0, 0, 0);
      };
#define INTERLEAVE() do { _Pragma("unroll") for (int i_ = 0; i_ < 12; ++i_) { \
        __builtin_amdgcn_sched_group_barrier(0x8, 1, 0); __builtin_amdgcn_sched_group_barrier(0x100, 1, 0); } \
        __builtin_amdgcn_sched_group_barrier(0x8, 4, 0); } while (0)
      WG_BARRIER();
      rd(xa, wa, 0, 0);
      __builtin_amdgcn_sched_barrier(0);
      for (u32 t = 0; t < 31; ++t) {
        rd(xb, wb, t, 1);
        mm(xa, wa);
        INTERLEAVE();
        __builtin_amdgcn_sched_barrier(0);
        WG_BARRIER();
        __builtin_amdgcn_sched_barrier(0);
        rd(xa, wa, t + 1, 0);
        mm(xb, wb);
        INTERLEAVE();
        __builtin_amdgcn_sched_barrier(0);
      }
      rd(xb, wb, 31, 1);
      mm(xa, wa);
      INTERLEAVE();
      __builtin_amdgcn_sched_barrier(0);
      mm(xb, wb);
      WG_BARRIER();
      const bool bad = lds_get(lctr + C_BAD) != 0;
      const float qnan = __builtin_nanf("");
      if (kind == 0) {
        char* db = ws + OFF_PO + (size_t)tix * 32768 + wv * 8192;
#pragma unroll
        for (int mt = 0; mt < 4; ++mt)
#pragma unroll
          for (int np = 0; np < 2; ++np) {
            f32x4 u = acc[2 * np][mt], v = acc[2 * np + 1][mt];
            if (bad) u[0] = qnan;
            const u32x4 o = {pack2(u[0], u[1]), pack2(u[2], u[3]), pack2(v[0], v[1]), pack2(v[2], v[3])};
            st16_sys(vl, o, db + (mt * 2 + np) * 1024);
          }
      } else {
        const LAS u32x4* pr = (const LAS u32x4*)(shm + (32 & (NSTAGE - 1)) * STAGE_BYTES + wv * 8192) + lane;
        char* ob = a.out[0] + (size_t)(row0 + wm * 64) * 8192 + (col0 + wn * 64) * 2;
        const u32 vo = (lane & 15) * 8192 + (g & 1) * 32 + (g >> 1) * 16;
#pragma unroll
        for (int mt = 0; mt < 4; ++mt)
#pragma unroll
          for (int np = 0; np < 2; ++np) {
            const u32x4 pv = pr[(mt * 2 + np) * 64];
            f32x4 u = acc[2 * np][mt], v = acc[2 * np + 1][mt];
            u[0] += __uint_as_float(pv.x << 16); u[1] += __uint_as_float(pv.x & 0xffff0000u);
            u[2] += __uint_as_float(pv.y << 16); u[3] += __uint_as_float(pv.y & 0xffff0000u);
            v[0] += __uint_as_float(pv.z << 16); v[1] += __uint_as_float(pv.z & 0xffff0000u);
            v[2] += __uint_as_float(pv.w << 16); v[3] += __uint_as_float(pv.w & 0xffff0000u);
            if (bad) u[0] = qnan;
            const u32 a0 = pack2(silu(u[0]), silu(u[1])), a1 = pack2(silu(u[2]), silu(u[3]));
            const u32 b0 = pack2(silu(v[0]), silu(v[1])), b1 = pack2(silu(v[2]), silu(v[3]));
            const auto s0 = __builtin_amdgcn_permlane16_swap(a0, b0, false, false);
            const auto s1 = __builtin_amdgcn_permlane16_swap(a1, b1, false, false);
            const u32x4 o = {s0[0], s1[0], s0[1], s1[1]};
            st16_sys(vo, o, ob + (size_t)mt * 16 * 8192 + np * 64);
          }
      }
      WAIT_VM0();
      WG_BARRIER();
    }

    if (ts < 4 || ts >= (xl ? 8u : 5u)) break;
  }
}

static void build_tables(Args& a) {
  int pos[32];
  for (int i = 0; i < 32; ++i) pos[PIECES[i]] = i;
  auto rdy = [&](int lr, int lc) { return pos[lr] > pos[16 + lc] ? pos[lr] : pos[16 + lc]; };
  int prank[128];
  for (int t = 0; t < 128; ++t) {
    const int kt = (rdy(8 + (t >> 4), t & 15) * 32 + rdy(t >> 4, t & 15)) * 128 + t;
    prank[t] = 0;
    for (int u = 0; u < 128; ++u) prank[t] += (rdy(8 + (u >> 4), u & 15) * 32 + rdy(u >> 4, u & 15)) * 128 + u < kt;
  }
  for (int xl = 0; xl < 8; ++xl) {
    int key[32], idx[32], ent[32];
    for (int t = 0; t < 32; ++t) {
      const int lr = (t >> 2) * 2 + (xl >> 2), lc = (t & 3) * 4 + (xl & 3), pr = prank[(lr & 7) * 16 + lc];
      key[t] = lr < 8 ? rdy(lr, lc) * 128 + pr : 10000 + pr;
      ent[t] = lr | (lc << 4) | (rdy(lr, lc) << 8) | (pr << 13);
      idx[t] = t;
    }
    for (int i = 1; i < 32; ++i)
      for (int j = i; j > 0 && key[idx[j]] < key[idx[j - 1]]; --j) { int s = idx[j]; idx[j] = idx[j - 1]; idx[j - 1] = s; }
    for (int slot = 0; slot < 32; ++slot) a.tile[xl * 32 + slot] = ent[idx[slot < 4 ? 28 + slot : slot - 4]];
  }
}

extern "C" void kernel_launch(int rank, int nranks, ncclComm_t comm, hipStream_t stream,
                              void* const* d_in, const int* in_sizes, int n_in,
                              void* d_out, int out_size, const int* in_dtypes, int out_dtype,
                              void* const* d_out_all, void* const* d_ws_all, size_t ws_bytes) {
  if (nranks != 8 || (unsigned)rank >= 8u || n_in < 2 || ws_bytes < WS_NEED || out_size != 4096 * 4096 ||
      in_sizes[0] != 4096 * 512 || in_sizes[1] != 512 * 4096) {
    fprintf(stderr, "kernel_launch: unexpected problem shape\n");
    return;
  }
  Args a;
  a.x = (const char*)d_in[0];
  a.w = (const char*)d_in[1];
  for (int l = 0; l < 8; ++l) { a.out[l] = (char*)d_out_all[rank ^ l]; a.ws[l] = (char*)d_ws_all[rank ^ l]; }
  a.rank = (u32)rank;
  static const unsigned char near_xcd[8][8] = {{0, 4, 2, 3, 6, 5, 1, 7}, {4, 0, 7, 5, 2, 3, 6, 1}, {2, 7, 0, 1, 4, 5, 3, 6}, {3, 5, 1, 0, 7, 4, 2, 6},
                                               {5, 2, 4, 7, 0, 1, 6, 3}, {6, 3, 5, 4, 1, 0, 7, 2}, {1, 6, 3, 2, 5, 7, 0, 4}, {7, 1, 6, 5, 3, 2, 4, 0}};
  a.xmap = 0;
  for (int p = 0; p < 8; ++p) {
    a.xmap |= (u32)(rank ^ p) << (4 * near_xcd[rank][p]);
    a.wsx[near_xcd[rank][p]] = (char*)d_ws_all[p];
  }
  build_tables(a);
  fused_gemm_ar_silu<<<256, 512, 0, stream>>>(a);
  hipError_t e = hipGetLastError();
  if (e != hipSuccess) fprintf(stderr, "launch failed: %s\n", hipGetErrorString(e));
}
```
